```python
import numpy as np

kernel_name = "gpu_gemm_ar_m4096_k4096_n16384_f32_reluq8"


ABSMAX_RTOL = 0.02

COMPUTE_DTYPE = "float32"
EPILOGUE = "relu_quant_int8"
IN_QUANT = None
OUT_QUANT = "int8"
IN_DTYPES = {"x": "float32", "w_mat": "float32"}
WORLD_SIZE = 8

def setup_inputs():

    x = np.random.randn(4096, 4096).astype(np.float32)
    w_mat = (np.random.randn(4096, 16384) / np.sqrt(4096)).astype(np.float32)
    return {"x": x, "w_mat": w_mat}

def _quant_dequant(y, amax):
    scale = amax / 127.0
    q = np.clip(np.round(y / scale), -127.0, 127.0)
    return (q * scale).astype(np.float32)
def _epilogue(y):
    y = np.maximum(y, 0.0)
    return _quant_dequant(y, np.abs(y).max())

def reference(x, w_mat):
    y = (x @ w_mat).astype(np.float32)
    return _epilogue(y)
```

```cpp
#include <hip/hip_runtime.h>
#include <rccl/rccl.h>
#include <cstdio>

#define HIP_CHECK(expr) do { hipError_t _e = (expr); if (_e != hipSuccess) { \
    fprintf(stderr, "HIP error %s at %s:%d\n", hipGetErrorString(_e), __FILE__, __LINE__); \
    return; } } while (0)

namespace {
#define LAS __attribute__((address_space(3)))
typedef short bf16x8 __attribute__((ext_vector_type(8)));
typedef float f32x4 __attribute__((ext_vector_type(4)));
typedef unsigned u32x4 __attribute__((ext_vector_type(4)));
typedef unsigned u32x2 __attribute__((ext_vector_type(2)));
typedef unsigned long long u64;

constexpr int NR = 8;
constexpr int GM = 4096, GN = 16384;
constexpr int KS = 512;
constexpr int NS = GN / NR;
constexpr int NT = 64;
constexpr int NEP = NT / 2;
#define NP 8
constexpr int NI1 = NEP * 8;
constexpr int NMI = 32;
constexpr int NI2 = NMI + 128;
constexpr size_t XS_T = (size_t)NR * GM * 16;
constexpr size_t WS_T = (size_t)NR * NS * 16;
constexpr size_t XS_OFF = 0, XS_BYTES = XS_T * NT;
constexpr size_t WS_OFF = XS_BYTES, WS_BYTES = WS_T * NT;
constexpr size_t MK_OFF = WS_OFF + WS_BYTES;
constexpr size_t MK_BYTES = (size_t)NR * 128 * 8 * 1024;
constexpr unsigned ST_SLOT = 8192 + 128;
constexpr size_t ST_OFF = MK_OFF + MK_BYTES;
constexpr size_t ST_BYTES = (size_t)NR * 128 * 8 * ST_SLOT;
constexpr size_t CTL_OFF = ST_OFF + ST_BYTES;
constexpr size_t INF_OFF = CTL_OFF;
constexpr size_t AMX_OFF = INF_OFF + 8192;
constexpr size_t OUF_OFF = AMX_OFF + 8192;
constexpr size_t MRD_OFF = OUF_OFF + 8192;
constexpr size_t DSC_OFF = MRD_OFF + 1024;
constexpr size_t TK_OFF = DSC_OFF + 8192;
constexpr size_t CNT_OFF = TK_OFF + 2048;
constexpr size_t TMO_OFF = CNT_OFF + 2048;
constexpr size_t DBG_OFF = TMO_OFF + 2048;
constexpr size_t WS_NEED = DBG_OFF + 256 * 16 * 8;
constexpr u64 DEADLINE = 100000000ull;
constexpr int HTB = 16384;
constexpr int LDS_BYTES = 8 * HTB + 1024;
#define OUT_AUX 0
constexpr int AUX_SYS = 17;
constexpr int AUX_DEV = 16;

struct Params {
    const float* x; const float* w;
    char* ws[NR]; signed char* out;
    int rank;
};

__device__ __forceinline__ unsigned cvt_pk_bf16(float lo, float hi) { unsigned r; asm("v_cvt_pk_bf16_f32 %0, %1, %2" : "=v"(r) : "v"(lo), "v"(hi)); return r; }
__device__ __forceinline__ __amdgpu_buffer_rsrc_t mk_rsrc(const void* p, unsigned bytes) { return __builtin_amdgcn_make_buffer_rsrc((void*)p, (short)0, (int)bytes, 0x00020000); }
__device__ __forceinline__ unsigned ld_sys(const unsigned* p) { return __hip_atomic_load(p, __ATOMIC_RELAXED, __HIP_MEMORY_SCOPE_SYSTEM); }
__device__ __forceinline__ u64 ld_sys64(const u64* p) { return __hip_atomic_load(p, __ATOMIC_RELAXED, __HIP_MEMORY_SCOPE_SYSTEM); }
__device__ __forceinline__ u64 ld_dev64(const u64* p) { return __hip_atomic_load(p, __ATOMIC_RELAXED, __HIP_MEMORY_SCOPE_AGENT); }
__device__ __forceinline__ void st_sys(unsigned* p, unsigned v) { __hip_atomic_store(p, v, __ATOMIC_RELAXED, __HIP_MEMORY_SCOPE_SYSTEM); }
__device__ __forceinline__ void st_sys64(u64* p, u64 v) { __hip_atomic_store(p, v, __ATOMIC_RELAXED, __HIP_MEMORY_SCOPE_SYSTEM); }
__device__ __forceinline__ void st_dev(unsigned* p, unsigned v) { __hip_atomic_store(p, v, __ATOMIC_RELAXED, __HIP_MEMORY_SCOPE_AGENT); }
__device__ __forceinline__ void st_dev64(u64* p, u64 v) { __hip_atomic_store(p, v, __ATOMIC_RELAXED, __HIP_MEMORY_SCOPE_AGENT); }
__device__ __forceinline__ int rfl(int v) { return __builtin_amdgcn_readfirstlane(v); }
__device__ __forceinline__ bool give_up(unsigned* tmo, unsigned call, u64 t0) {
    const unsigned tv = (unsigned)rfl((int)__hip_atomic_load(tmo, __ATOMIC_RELAXED, __HIP_MEMORY_SCOPE_AGENT));
    if (tv == call || __builtin_amdgcn_s_memrealtime() - t0 > DEADLINE) { st_dev(tmo, call); return true; }
    return false;
}
#define STAMP(i) do {} while (0)
#define BSTORE(v, rs, voff, soff, aux) do { __builtin_amdgcn_raw_buffer_store_b128(v, rs, voff, soff, aux); asm volatile("s_nop 1" :: "v"(v) : "memory"); } while (0)
#define TICKET_DRAW(dst, ptr) asm volatile("global_atomic_add %0, %1, %2, %3 sc0" : "=v"(dst) : "v"(0u), "v"(1u), "s"(ptr) : "memory")
#define TICKET_USE(v) asm volatile("" : "+v"(v))
#define WAIT_V(n) asm volatile("s_waitcnt vmcnt(" #n ")" ::: "memory")
#define WAIT_L(n) asm volatile("s_waitcnt lgkmcnt(" #n ")" ::: "memory")
#define BAR do { asm volatile("" ::: "memory"); __builtin_amdgcn_s_barrier(); asm volatile("" ::: "memory"); } while (0)
#define RBAR __builtin_amdgcn_s_barrier()
#define SCHED __builtin_amdgcn_sched_barrier(0)

__global__ __launch_bounds__(512, 2) void fused_kernel(Params P) {
    extern __shared__ __attribute__((aligned(16))) unsigned char shm_raw[];
    LAS unsigned char* lds = (LAS unsigned char*)shm_raw;
    const int tid = threadIdx.x, lane = tid & 63;
    const int wid = rfl(tid >> 6);
    const int bid = blockIdx.x, xcd = bid & 7, slot = bid >> 3;
    const int rank = P.rank;
    char* const myws = P.ws[rank];
    unsigned* const tmo = (unsigned*)(myws + TMO_OFF);
    STAMP(0);
    unsigned* const cntp = (unsigned*)(myws + CNT_OFF) + bid;
    const unsigned prev_v = __hip_atomic_load(cntp, __ATOMIC_RELAXED, __HIP_MEMORY_SCOPE_AGENT);

    if (slot < NP) {
        const int dst = (xcd + slot) & 7;
        char* const dws = P.ws[dst];
        LAS unsigned* tkl = (LAS unsigned*)(lds + 8 * HTB);
        if (tid == 0) tkl[2] = 0u;
        unsigned call;
        {
            const __amdgpu_buffer_rsrc_t rs = mk_rsrc(dws, (unsigned)MK_OFF);
            unsigned* const tk = (unsigned*)(myws + TK_OFF) + dst * 32;
            const float* xsrc = P.x + (size_t)(wid * 64 + (lane >> 2)) * KS + (lane & 3) * 4;
            const unsigned xoff = (unsigned)(XS_OFF + ((size_t)rank * GM + tid) * 16);
            const int r8 = tid & 255, opar = wid >> 2;
            const int col = ((r8 >> 5) & 3) * 64 + ((r8 >> 2) & 3) * 16 + (r8 >> 7) * 8 + ((r8 >> 4) & 1) * 4 + (r8 & 3);
            const float* wsrc = P.w + (size_t)opar * 8 * GN + (size_t)dst * NS + col;
            const unsigned woff = (unsigned)(WS_OFF + (size_t)opar * WS_T + ((size_t)rank * NS + r8) * 16);
            LAS unsigned char* xb = lds + wid * 2304;
            const int xw = ((lane >> 1) & 1) * 1088 + (lane >> 2) * 16 + (lane & 1) * 8;
            f32x4 v[4]; float u[8]; u32x4 o0, o1, ow;
#define P1_LOAD(it) do { const int _e = (it) >> 3, _g = (it) & 7; const float* _x = xsrc + (size_t)_g * 512 * KS + _e * 16; const float* _w = wsrc + (size_t)_e * 16 * GN + _g * 256; \
            _Pragma("unroll") for (int j = 0; j < 4; ++j) v[j] = *(const f32x4*)(_x + (size_t)j * 16 * KS); \
            _Pragma("unroll") for (int j = 0; j < 8; ++j) u[j] = _w[(size_t)j * GN]; asm volatile("" ::: "memory"); } while (0)
#define P1_CVT() do { _Pragma("unroll") for (int j = 0; j < 4; ++j) *(LAS u32x2*)(xb + xw + j * 256) = (u32x2){cvt_pk_bf16(v[j][0], v[j][1]), cvt_pk_bf16(v[j][2], v[j][3])}; \
            ow.x = cvt_pk_bf16(u[0], u[1]); ow.y = cvt_pk_bf16(u[2], u[3]); ow.z = cvt_pk_bf16(u[4], u[5]); ow.w = cvt_pk_bf16(u[6], u[7]); \
            WAIT_L(0); o0 = *(const LAS u32x4*)(xb + lane * 16); o1 = *(const LAS u32x4*)(xb + 1088 + lane * 16); WAIT_L(0); asm volatile("" : "+v"(o0), "+v"(o1), "+v"(ow)); } while (0)
            int it0 = slot, it1 = slot + NP, it2 = slot + 2 * NP, itp = -1;
            P1_LOAD(it0);
            P1_CVT();
            call = (unsigned)rfl((int)prev_v) + 1u;
            const unsigned tbase = (call - 1u) * (unsigned)NI1 - 3u * NP;
            unsigned tkt = 0; int n = 0;
            while (it0 < NI1) {
                if (it1 < NI1) P1_LOAD(it1);
                const unsigned e = (unsigned)it0 >> 3, g = (unsigned)it0 & 7u;
                BSTORE(ow, rs, woff, e * (unsigned)(2 * WS_T) + g * 4096u, AUX_SYS);
                BSTORE(o0, rs, xoff, e * (unsigned)(2 * XS_T) + g * 8192u, AUX_SYS);
                BSTORE(o1, rs, xoff, e * (unsigned)(2 * XS_T) + g * 8192u + (unsigned)XS_T, AUX_SYS);
                WAIT_V(3);
                if (it1 < NI1) P1_CVT();
                TICKET_USE(tkt);
                if (tid == 0) { tkl[n & 1] = tkt; WAIT_L(0); TICKET_DRAW(tkt, tk); }
                WAIT_L(0); BAR;
                if (tid == 0 && itp >= 0) st_sys((unsigned*)(dws + INF_OFF) + (rank * 8 + (itp & 7)) * 32 + (itp >> 3), call);
                itp = it0; it0 = it1; it1 = it2;
                if (n > 0) it1 = rfl((int)min(tkl[n & 1] - tbase, (unsigned)NI1));
                ++n;
            }
            WAIT_V(0); BAR;
            if (tid == 0) st_sys((unsigned*)(dws + INF_OFF) + (rank * 8 + (itp & 7)) * 32 + (itp >> 3), call);
            asm volatile("" :: "v"(tkt));
        }
        STAMP(1);
        if (dst != rank) {
            unsigned* const tk = (unsigned*)(myws + TK_OFF) + (8 + dst) * 32;
            const unsigned tbase = (call - 1u) * (unsigned)NI2 - 3u * NP;
            u64 rdy0 = 0, rdy1 = 0; bool desc_ok = false, dead = false; unsigned dl0 = 0, dl1 = 0;
            u32x4 cur[8], nxt[8]; unsigned coff = 0, clen = 0, noff = 0, nlen = 0;
#define P2_KNOWN(it) ((it) < NMI ? ((((rdy0 & rdy1) >> (2 * (it))) & 3ull) == 3ull) : desc_ok)
#define P2_WAIT(it) do { const u64 _t0 = __builtin_amdgcn_s_memrealtime(); unsigned _n = 0; \
            while (!dead) { \
                if ((it) < NMI) { const u64 _f = ld_dev64((const u64*)(myws + MRD_OFF) + lane); rdy0 = __ballot((unsigned)_f == call); rdy1 = __ballot((unsigned)(_f >> 32) == call); } \
                else { const u64 _a = ld_dev64((const u64*)(myws + DSC_OFF) + wid * 128 + lane), _b = ld_dev64((const u64*)(myws + DSC_OFF) + wid * 128 + 64 + lane); \
                       dl0 = (unsigned)_a; dl1 = (unsigned)_b; desc_ok = __all((unsigned)(_a >> 32) == call && (unsigned)(_b >> 32) == call); } \
                if (P2_KNOWN(it)) break; \
                if ((++_n & 31u) == 0u && give_up(tmo, call, _t0)) dead = true; \
                __builtin_amdgcn_s_sleep(2); } \
            asm volatile("" ::: "memory"); } while (0)
#define P2_LOAD(it) do { if ((it) < NMI) { noff = (unsigned)MK_OFF + (unsigned)(rank * 128 + 4 * (it)) * 8192u + (unsigned)wid * 4096u; nlen = 4096u; } \
            else { const int _t = (it) - NMI; noff = (unsigned)ST_OFF + (unsigned)((rank * 128 + _t) * 8 + wid) * ST_SLOT; \
                   nlen = (unsigned)__builtin_amdgcn_readlane((int)(_t < 64 ? dl0 : dl1), _t & 63); nlen = nlen > 8192u ? 8192u : nlen; } \
            const __amdgpu_buffer_rsrc_t _r = mk_rsrc(myws + noff, nlen); \
            _Pragma("unroll") for (int k = 0; k < 8; ++k) nxt[k] = __builtin_amdgcn_raw_buffer_load_b128(_r, (unsigned)lane * 16u + k * 1024u, 0, AUX_DEV); asm volatile("" ::: "memory"); } while (0)
#define P2_STORE() do { const __amdgpu_buffer_rsrc_t _r = mk_rsrc(dws + (unsigned)rfl((int)coff), (unsigned)rfl((int)clen)); \
            _Pragma("unroll") for (int k = 0; k < 8; ++k) BSTORE(cur[k], _r, (unsigned)lane * 16u + k * 1024u, 0, AUX_SYS); } while (0)
#define P2_ROT() do { _Pragma("unroll") for (int k = 0; k < 8; ++k) { cur[k] = nxt[k]; asm volatile("" : "+v"(cur[k])); } coff = noff; clen = nlen; } while (0)
            int it0 = slot, it1 = slot + NP, it2 = slot + 2 * NP, itp = -1;
            P2_WAIT(it0);
            STAMP(2);
            P2_LOAD(it0); WAIT_V(0); P2_ROT();
            unsigned tkt = 0; int n = 0;
            while (it0 < NI2) {
                if (it1 >= NI2) { P2_STORE(); WAIT_V(8); }
                else if (P2_KNOWN(it1)) { P2_LOAD(it1); P2_STORE(); WAIT_V(8); P2_ROT(); }
                else { P2_STORE(); P2_WAIT(it1); P2_LOAD(it1); WAIT_V(0); P2_ROT(); }
                TICKET_USE(tkt);
                if (tid == 0) { tkl[n & 1] = tkt; WAIT_L(0); TICKET_DRAW(tkt, tk); }
                if (lane == 0 && dead) tkl[2] = call;
                WAIT_L(0); BAR;
                if (tid == 0 && itp >= 0 && tkl[2] != call) st_sys((unsigned*)(dws + OUF_OFF) + rank * 256 + itp, call);
                itp = it0; it0 = it1; it1 = it2;
                if (n > 0) it1 = rfl((int)min(tkl[n & 1] - tbase, (unsigned)NI2));
                ++n;
            }
            WAIT_V(0); BAR;
            if (tid == 0 && tkl[2] != call) st_sys((unsigned*)(dws + OUF_OFF) + rank * 256 + itp, call);
            asm volatile("" :: "v"(tkt));
            STAMP(3);
        }
    } else if (slot >= 16) {
        const unsigned call = (unsigned)rfl((int)prev_v) + 1u;
        const int wr = wid >> 2, wc = wid & 3, fr = lane & 15, fq = lane >> 4;
        const __amdgpu_buffer_rsrc_t ro = mk_rsrc(P.out, (unsigned)((size_t)GM * GN));
        {
        const int ts = slot - 16;
        const int pm = (xcd >> 1) * 4 + (ts & 3), pn = (xcd & 1) * 4 + (ts >> 2);
        const int tile = pm * 8 + pn;
        const char* cA = myws + XS_OFF + ((size_t)wid * GM + pm * 256) * 16 + lane * 16;
        const char* cB = myws + WS_OFF + ((size_t)wid * NS + pn * 256) * 16 + lane * 16;
        const unsigned* fp = (const unsigned*)(myws + INF_OFF) + (wid * 8 + (lane < 32 ? (pm >> 1) : pn)) * 32 + (lane & 31);
        const unsigned ldsw = (unsigned)wid * 2048u;
        const int aoff = fq * 2048 + (wr * 64 + fr) * 16, boff = fq * 2048 + (wc * 32 + fr) * 16;
        unsigned known = 0;
        bool kdead = false;
#define SA(b, h) (((b) * 2 + (h)) * HTB)
#define SB(b, h) ((4 + (b) * 2 + (h)) * HTB)
#define STAGE(bufoff, g) do { _Pragma("unroll") for (int _i = 0; _i < 2; ++_i) \
        __builtin_amdgcn_global_load_lds((const unsigned*)((g) + _i * 1024), (LAS unsigned*)(lds + (bufoff) + ldsw + _i * 1024), 16, 0, AUX_DEV); } while (0)
#define LDA(dst, b, h) do { _Pragma("unroll") for (int m = 0; m < 4; ++m) _Pragma("unroll") for (int k = 0; k < 2; ++k) dst[m][k] = *(const LAS bf16x8*)(lds + SA(b, h) + aoff + m * 256 + k * 8192); } while (0)
#define LDB(dst, b, h) do { _Pragma("unroll") for (int n = 0; n < 2; ++n) _Pragma("unroll") for (int k = 0; k < 2; ++k) dst[n][k] = *(const LAS bf16x8*)(lds + SB(b, h) + boff + n * 256 + k * 8192); } while (0)
#define MMA(ai, bj, At, Bt) do { __builtin_amdgcn_s_setprio(1); _Pragma("unroll") for (int m = 0; m < 4; ++m) _Pragma("unroll") for (int n = 0; n < 2; ++n) _Pragma("unroll") for (int k = 0; k < 2; ++k) \
        acc[ai][bj][m][n] = __builtin_amdgcn_mfma_f32_16x16x32_bf16(Bt[n][k], At[m][k], acc[ai][bj][m][n], 0, 0, 0); __builtin_amdgcn_s_setprio(0); } while (0)
#define WAIT_EPOCH(e) do { if (!((known >> (e)) & 1u)) { const u64 _t0 = __builtin_amdgcn_s_memrealtime(); unsigned _n = 0; \
            for (;;) { const u64 _b = __ballot(ld_sys(fp) == call); known = (unsigned)_b & (unsigned)(_b >> 32); if ((known >> (e)) & 1u) break; \
                if ((++_n & 63u) == 0u && give_up(tmo, call, _t0)) { known = 0xffffffffu; kdead = true; break; } \
                __builtin_amdgcn_s_sleep(1); } \
            asm volatile("" ::: "memory"); } } while (0)
        f32x4 acc[2][2][4][2];
#pragma unroll
        for (int a = 0; a < 2; ++a)
#pragma unroll
            for (int b = 0; b < 2; ++b)
#pragma unroll
                for (int m = 0; m < 4; ++m)
#pragma unroll
                    for (int n = 0; n < 2; ++n) acc[a][b][m][n] = (f32x4){0.f, 0.f, 0.f, 0.f};
        bf16x8 At[4][2], B0[2][2], B1[2][2];
        WAIT_EPOCH(0);
        STAMP(1);
        STAGE(SB(0, 0), cB); STAGE(SA(0, 0), cA); STAGE(SB(0, 1), cB + 2048); STAGE(SA(0, 1), cA + 2048);
        if (wr == 1) RBAR;
        WAIT_V(4); RBAR;
        STAGE(SB(1, 0), cB + WS_T); STAGE(SA(1, 0), cA + XS_T); STAGE(SB(1, 1), cB + WS_T + 2048);
        WAIT_V(6); RBAR;
        for (int e = 0; e < NEP; ++e) {
            const bool last = (e == NEP - 1);
            const char* a1 = cA + (size_t)(2 * e + 1) * XS_T;
            const int t2 = last ? 2 * e : 2 * e + 2;
            const char* a2 = cA + (size_t)t2 * XS_T; const char* b2 = cB + (size_t)t2 * WS_T;
            const char* a3 = a2 + XS_T; const char* b3 = b2 + WS_T;
            if (!last) WAIT_EPOCH(e + 1);
            if (e == 7) STAMP(7); if (e == 15) STAMP(8); if (e == 23) STAMP(9); if (e == 30) STAMP(10);
            LDB(B0, 0, 0); SCHED; LDA(At, 0, 0); STAGE(SA(1, 1), a1 + 2048);
            WAIT_L(8); RBAR; WAIT_L(0); MMA(0, 0, At, B0); RBAR; SCHED;
            LDB(B1, 0, 1); STAGE(SB(0, 0), b2);
            RBAR; WAIT_L(0); MMA(0, 1, At, B1); RBAR;
            LDA(At, 0, 1); STAGE(SA(0, 0), a2);
            RBAR; WAIT_L(0); MMA(1, 0, At, B0); RBAR; SCHED;
            STAGE(SB(0, 1), b2 + 2048);
            WAIT_V(6); RBAR; MMA(1, 1, At, B1); RBAR;
            LDB(B0, 1, 0); SCHED; LDA(At, 1, 0); STAGE(SA(0, 1), a2 + 2048);
            WAIT_L(8); RBAR; WAIT_L(0); MMA(0, 0, At, B0); RBAR; SCHED;
            LDB(B1, 1, 1); STAGE(SB(1, 0), b3);
            RBAR; WAIT_L(0); MMA(0, 1, At, B1); RBAR;
            LDA(At, 1, 1); STAGE(SA(1, 0), a3);
            RBAR; WAIT_L(0); MMA(1, 0, At, B0); RBAR; SCHED;
            STAGE(SB(1, 1), b3 + 2048);
            WAIT_V(6); RBAR; MMA(1, 1, At, B1); RBAR;
        }
        if (wr == 0) RBAR;
        STAMP(2);
        u32x4 mk;
#pragma unroll
        for (int g2 = 0; g2 < 4; ++g2) { unsigned mm = 0;
#pragma unroll
            for (int e = 31; e >= 0; --e) { const int g = g2 * 2 + (e >> 4), d = (e >> 2) & 3; mm = __builtin_amdgcn_alignbit(mm, __float_as_uint(acc[g >> 2][d >> 1][g & 3][d & 1][e & 3]), 31); }
            mk[g2] = ~mm; }
        const unsigned reg_id = (unsigned)((rank * 128 + tile) * 8 + wid);
        BSTORE(mk, mk_rsrc(myws + MK_OFF, (unsigned)MK_BYTES), (unsigned)lane * 16u, reg_id * 1024u, AUX_DEV);
        SCHED;
        LAS float* red = (LAS float*)(lds + 8 * HTB);
        float mx = 0.f;
#pragma unroll
        for (int a = 0; a < 2; ++a)
#pragma unroll
            for (int b = 0; b < 2; ++b)
#pragma unroll
                for (int m = 0; m < 4; ++m)
#pragma unroll
                    for (int n = 0; n < 2; ++n) { const f32x4 v = acc[a][b][m][n]; mx = fmaxf(fmaxf(mx, fmaxf(v[0], v[1])), fmaxf(v[2], v[3])); }
#pragma unroll
        for (int o = 32; o >= 1; o >>= 1) mx = fmaxf(mx, __shfl_xor(mx, o));
        if (lane == 0) red[wid] = kdead ? -1.0f : mx;
        __builtin_amdgcn_s_waitcnt(0x0F70);
        WAIT_L(0); BAR;
        {
            float m8 = red[0], n8 = red[0];
#pragma unroll
            for (int i = 1; i < 8; ++i) { m8 = fmaxf(m8, red[i]); n8 = fminf(n8, red[i]); }
            if (lane == 0 && n8 >= 0.f) {
                if (wid == 0) st_dev((unsigned*)(myws + MRD_OFF) + tile, call);
                st_sys64((u64*)(P.ws[wid] + AMX_OFF) + rank * 128 + tile, ((u64)call << 32) | (u64)__float_as_uint(m8));
            }
        }
        LAS unsigned char* sbuf = lds + wid * 8192;
#pragma unroll
        for (int i = 0; i < 8; ++i) *(LAS u32x4*)(sbuf + i * 1024 + lane * 16) = (u32x4){0u, 0u, 0u, 0u};
        const unsigned cntnz = __builtin_popcount(mk.x) + __builtin_popcount(mk.y) + __builtin_popcount(mk.z) + __builtin_popcount(mk.w);
        {
            const u64* gp = (const u64*)(myws + AMX_OFF) + wid * 128 + lane;
            const u64 t0 = __builtin_amdgcn_s_memrealtime(); unsigned n = 0; float gm; bool dead = false;
            for (;;) {
                const u64 g0 = ld_sys64(gp), g1 = ld_sys64(gp + 64);
                gm = fmaxf(__uint_as_float((unsigned)g0), __uint_as_float((unsigned)g1));
                if (__all((unsigned)(g0 >> 32) == call && (unsigned)(g1 >> 32) == call)) break;
                if ((++n & 63u) == 0u && give_up(tmo, call, t0)) { dead = true; break; }
                __builtin_amdgcn_s_sleep(1);
            }
#pragma unroll
            for (int o = 32; o >= 1; o >>= 1) gm = fmaxf(gm, __shfl_xor(gm, o));
            if (lane == 0) red[8 + wid] = dead ? -1.0f : gm;
        }
        WAIT_L(0); BAR;
        float amax = red[8], amin = red[8];
#pragma unroll
        for (int i = 1; i < 8; ++i) { amax = fmaxf(amax, red[8 + i]); amin = fminf(amin, red[8 + i]); }
        const bool bad = rfl(amin < 0.f ? 1 : 0) != 0;
        const float rinv = amax > 0.f ? 1.0f / amax : 0.f;
        STAMP(3);
        unsigned allor = 0;
        u32x4 qloc[8];
        {
            const float c127 = __uint_as_float(127u);
#pragma unroll
            for (int g = 0; g < 8; ++g) { u32x4& qb = qloc[g];
#pragma unroll
                for (int d = 0; d < 4; ++d) { f32x4& v = acc[g >> 2][d >> 1][g & 3][d & 1];
#pragma unroll
                    for (int j = 0; j < 4; ++j) { float t; asm("v_mul_f32_e64 %0, %1, %2 clamp" : "=v"(t) : "v"(v[j]), "v"(rinv)); asm("v_mul_f32_e32 %0, %1, %2" : "=v"(t) : "v"(c127), "v"(t)); v[j] = t; }
                    const unsigned w = __float_as_uint(v[0]) | (__float_as_uint(v[1]) << 8) | (__float_as_uint(v[2]) << 16) | (__float_as_uint(v[3]) << 24);
                    allor |= w; qb[d] = bad ? 0x80808080u : w; }
            }
        }
        allor |= allor >> 16; allor |= allor >> 8; allor &= 0xffu;
        const unsigned W = 31u - (unsigned)__builtin_clz(2u * allor + 1u);
        unsigned incl = cntnz * W;
#pragma unroll
        for (int o = 1; o < 64; o <<= 1) { const unsigned t = __shfl_up(incl, o); if (lane >= o) incl += t; }
        const unsigned total_bits = 192u + (unsigned)__builtin_amdgcn_readlane((int)incl, 63);
        unsigned Pb = 192u + incl - cntnz * W;
        { const u64 h = (u64)W << ((3u * lane) & 31u); LAS unsigned* hp = (LAS unsigned*)(sbuf + ((3u * lane) >> 5) * 4u);
          __hip_atomic_fetch_or(hp, (unsigned)h, __ATOMIC_RELAXED, __HIP_MEMORY_SCOPE_WORKGROUP); __hip_atomic_fetch_or(hp + 1, (unsigned)(h >> 32), __ATOMIC_RELAXED, __HIP_MEMORY_SCOPE_WORKGROUP); }
#pragma unroll
        for (int g = 0; g < 8; ++g)
#pragma unroll
            for (int d = 0; d < 4; ++d) { const f32x4 v = acc[g >> 2][d >> 1][g & 3][d & 1]; const unsigned mw = mk[g >> 1]; const int b0 = (g & 1) * 16 + d * 4;
                unsigned ch = __float_as_uint(v[0]); unsigned p = ((mw >> b0) & 1u) * W;
                ch |= __float_as_uint(v[1]) << p; p += ((mw >> (b0 + 1)) & 1u) * W;
                ch |= __float_as_uint(v[2]) << p; p += ((mw >> (b0 + 2)) & 1u) * W;
                ch |= __float_as_uint(v[3]) << p; p += ((mw >> (b0 + 3)) & 1u) * W;
                const u64 h = (u64)ch << (Pb & 31u); LAS unsigned* hp = (LAS unsigned*)(sbuf + ((Pb >> 3) & ~3u));
                __hip_atomic_fetch_or(hp, (unsigned)h, __ATOMIC_RELAXED, __HIP_MEMORY_SCOPE_WORKGROUP); __hip_atomic_fetch_or(hp + 1, (unsigned)(h >> 32), __ATOMIC_RELAXED, __HIP_MEMORY_SCOPE_WORKGROUP);
                Pb += p; }
        WAIT_L(0);
        {
            const unsigned nbytes = (total_bits + 127u) / 128u * 16u;
            const __amdgpu_buffer_rsrc_t rd = mk_rsrc(myws + ST_OFF + (size_t)reg_id * ST_SLOT, nbytes);
#pragma unroll
            for (int k = 0; k < 8; ++k) { const u32x4 sv = *(const LAS u32x4*)(sbuf + k * 1024 + lane * 16); BSTORE(sv, rd, (unsigned)lane * 16u + k * 1024u, 0, AUX_DEV); }
            WAIT_V(0);
            if (lane == 0 && !bad) st_dev64((u64*)(myws + DSC_OFF) + wid * 128 + tile, ((u64)call << 32) | nbytes);
        }
        STAMP(4);
        {
            const unsigned voff = (unsigned)((pm * 256 + wr * 64 + fr) * GN + rank * NS + pn * 256 + wc * 64 + fq * 16);
#pragma unroll
            for (int g = 0; g < 8; ++g) BSTORE(qloc[g], ro, voff, (unsigned)(((g >> 2) * 128 + (g & 3) * 16) * GN), OUT_AUX);
        }
        WAIT_L(0); BAR;
        }
        {
        LAS unsigned char* sbuf = lds + wid * 8448;
        const __amdgpu_buffer_rsrc_t rm = mk_rsrc(myws + MK_OFF, (unsigned)MK_BYTES);
        for (int i = 0; i < NR - 1; ++i) {
            const int J = (bid - 128) + 128 * i, tile = J / 7, src = (rank + 1 + (J - tile * 7)) & 7;
            const int pm = tile >> 3, pn = tile & 7;
            bool bad = false;
            {
                const unsigned* fp = (const unsigned*)(myws + OUF_OFF) + src * 256 + ((lane & 1) ? NMI + tile : (tile >> 2));
                const u64 t0 = __builtin_amdgcn_s_memrealtime(); unsigned n = 0;
                while (!__all(ld_sys(fp) == call)) {
                    if ((++n & 15u) == 0u && give_up(tmo, call, t0)) { bad = true; break; }
                    __builtin_amdgcn_s_sleep(4);
                }
                asm volatile("" ::: "memory");
            }
            if (i == 0) STAMP(5);
            if (i == 6) STAMP(11);
            const unsigned reg_id = (unsigned)((src * 128 + tile) * 8 + wid);
            const char* sp = myws + ST_OFF + (size_t)reg_id * ST_SLOT + lane * 16;
#pragma unroll
            for (int k = 0; k < 8; ++k) __builtin_amdgcn_global_load_lds((const unsigned*)(sp + k * 1024), (LAS unsigned*)(sbuf + k * 1024), 16, 0, AUX_DEV);
            const u32x4 mk = __builtin_amdgcn_raw_buffer_load_b128(rm, (unsigned)lane * 16u, reg_id * 1024u, AUX_DEV);
            const unsigned cntnz = __builtin_popcount(mk.x) + __builtin_popcount(mk.y) + __builtin_popcount(mk.z) + __builtin_popcount(mk.w);
            WAIT_V(0);
            unsigned W;
            { const LAS unsigned* hp = (const LAS unsigned*)(sbuf + ((3u * lane) >> 5) * 4u); W = __builtin_amdgcn_alignbit(hp[1], hp[0], (3u * lane) & 31u) & 7u; }
            unsigned incl = cntnz * W;
#pragma unroll
            for (int o = 1; o < 64; o <<= 1) { const unsigned t = __shfl_up(incl, o); if (lane >= o) incl += t; }
            unsigned Pb = 192u + incl - cntnz * W;
            const unsigned voff = (unsigned)((pm * 256 + wr * 64 + fr) * GN + src * NS + pn * 256 + wc * 64 + fq * 16);
#pragma unroll
            for (int g = 0; g < 8; ++g) { u32x4 qb;
#pragma unroll
                for (int d = 0; d < 4; ++d) { const unsigned mw = mk[g >> 1]; const int b0 = (g & 1) * 16 + d * 4;
                    const LAS unsigned* hp = (const LAS unsigned*)(sbuf + ((Pb >> 3) & ~3u));
                    const unsigned win = __builtin_amdgcn_alignbit(hp[1], hp[0], Pb & 31u);
                    const unsigned w0 = ((mw >> b0) & 1u) * W, w1 = ((mw >> (b0 + 1)) & 1u) * W, w2 = ((mw >> (b0 + 2)) & 1u) * W, w3 = ((mw >> (b0 + 3)) & 1u) * W;
                    const unsigned p1 = w0, p2 = p1 + w1, p3 = p2 + w2;
                    const unsigned o = __builtin_amdgcn_ubfe(win, 0u, w0) | (__builtin_amdgcn_ubfe(win, p1, w1) << 8) | (__builtin_amdgcn_ubfe(win, p2, w2) << 16) | (__builtin_amdgcn_ubfe(win, p3, w3) << 24);
                    qb[d] = bad ? 0x80808080u : o; Pb += p3 + w3; }
                BSTORE(qb, ro, voff, (unsigned)(((g >> 2) * 128 + (g & 3) * 16) * GN), OUT_AUX);
            }
            WAIT_L(0);
        }
        STAMP(12);
        }
    }
    __syncthreads();
    STAMP(6);
    if (tid == 0) st_dev(cntp, prev_v + 1u);
}
}

extern "C" void kernel_launch(int rank, int nranks, ncclComm_t comm, hipStream_t stream,
                              void* const* d_in, const int* in_sizes, int n_in,
                              void* d_out, int out_size, const int* in_dtypes, int out_dtype,
                              void* const* d_out_all, void* const* d_ws_all, size_t ws_bytes) {
    if (nranks != NR || ws_bytes < WS_NEED || (unsigned)rank >= (unsigned)NR) { fprintf(stderr, "unexpected configuration\n"); return; }
    HIP_CHECK(hipFuncSetAttribute((const void*)fused_kernel, hipFuncAttributeMaxDynamicSharedMemorySize, LDS_BYTES));
    Params P;
    P.x = (const float*)d_in[0]; P.w = (const float*)d_in[1]; P.rank = rank; P.out = (signed char*)d_out;
    for (int p = 0; p < NR; ++p) P.ws[p] = (char*)d_ws_all[p];
    hipLaunchKernelGGL(fused_kernel, dim3(256), dim3(512), LDS_BYTES, stream, P);
    HIP_CHECK(hipGetLastError());
}
```

```cpp
#include <hip/hip_runtime.h>
#include <rccl/rccl.h>
#include <cstdio>

#define HIP_CHECK(expr) do { hipError_t _e = (expr); if (_e != hipSuccess) { \
    fprintf(stderr, "HIP error %s at %s:%d\n", hipGetErrorString(_e), __FILE__, __LINE__); \
    return; } } while (0)

namespace {
#define LAS __attribute__((address_space(3)))
typedef short bf16x8 __attribute__((ext_vector_type(8)));
typedef float f32x4 __attribute__((ext_vector_type(4)));
typedef unsigned u32x4 __attribute__((ext_vector_type(4)));
typedef unsigned u32x2 __attribute__((ext_vector_type(2)));
typedef unsigned long long u64;

constexpr int NR = 8;
constexpr int GM = 4096, GN = 16384;
constexpr int KS = 512;
constexpr int NS = GN / NR;
constexpr int NT = 64;
constexpr int NEP = NT / 2;
#define NP 8
constexpr int NI1 = NEP * 8;
constexpr int NMI = 32;
constexpr int NI2 = NMI + 128;
constexpr size_t XS_T = (size_t)NR * GM * 16;
constexpr size_t WS_T = (size_t)NR * NS * 16;
constexpr size_t XS_OFF = 0, XS_BYTES = XS_T * NT;
constexpr size_t WS_OFF = XS_BYTES, WS_BYTES = WS_T * NT;
constexpr size_t MK_OFF = WS_OFF + WS_BYTES;
constexpr size_t MK_BYTES = (size_t)NR * 128 * 8 * 1024;
constexpr unsigned ST_SLOT = 8192 + 128;
constexpr size_t ST_OFF = MK_OFF + MK_BYTES;
constexpr size_t ST_BYTES = (size_t)NR * 128 * 8 * ST_SLOT;
constexpr size_t IB_OFF = ST_OFF + ST_BYTES;
constexpr size_t IB_BYTES = (size_t)NR * NI1 * 6 * 3072;
constexpr size_t CTL_OFF = IB_OFF + IB_BYTES;
constexpr size_t HDR_OFF = CTL_OFF;
constexpr size_t INF_OFF = HDR_OFF + (size_t)NR * NI1 * 64;
constexpr size_t AMX_OFF = INF_OFF + 24576;
constexpr size_t OUF_OFF = AMX_OFF + 8192;
constexpr size_t RDY_OFF = OUF_OFF + 8192;
constexpr size_t TK_OFF = RDY_OFF + 1024;
constexpr size_t CNT_OFF = TK_OFF + 2048;
constexpr size_t TMO_OFF = CNT_OFF + 2048;
constexpr size_t DBG_OFF = TMO_OFF + 2048;
constexpr size_t WS_NEED = DBG_OFF + 256 * 16 * 8;
constexpr u64 DEADLINE = 100000000ull;
constexpr int HTB = 16384;
constexpr int LDS_BYTES = 8 * HTB + 1024;
#define POLL_SLEEP 1
#define ND 2
#define P2_SLEEP 4
#define EXP_SLEEP 2
#define OUT_AUX 0
constexpr unsigned HDR_BITS = 32 + 192;
constexpr int AUX_SYS = 17;
constexpr int AUX_DEV = 16;

struct Params {
    const float* x; const float* w;
    char* ws[NR]; signed char* out;
    int rank;
};

__device__ __forceinline__ unsigned cvt_pk_bf16(float lo, float hi) { unsigned r; asm("v_cvt_pk_bf16_f32 %0, %1, %2" : "=v"(r) : "v"(lo), "v"(hi)); return r; }
__device__ __forceinline__ __amdgpu_buffer_rsrc_t mk_rsrc(const void* p, unsigned bytes) { return __builtin_amdgcn_make_buffer_rsrc((void*)p, (short)0, (int)bytes, 0x00020000); }
__device__ __forceinline__ unsigned ld_sys(const unsigned* p) { return __hip_atomic_load(p, __ATOMIC_RELAXED, __HIP_MEMORY_SCOPE_SYSTEM); }
__device__ __forceinline__ u64 ld_sys64(const u64* p) { return __hip_atomic_load(p, __ATOMIC_RELAXED, __HIP_MEMORY_SCOPE_SYSTEM); }
__device__ __forceinline__ u64 ld_dev64(const u64* p) { return __hip_atomic_load(p, __ATOMIC_RELAXED, __HIP_MEMORY_SCOPE_AGENT); }
__device__ __forceinline__ void st_sys(unsigned* p, unsigned v) { __hip_atomic_store(p, v, __ATOMIC_RELAXED, __HIP_MEMORY_SCOPE_SYSTEM); }
__device__ __forceinline__ void st_sys64(u64* p, u64 v) { __hip_atomic_store(p, v, __ATOMIC_RELAXED, __HIP_MEMORY_SCOPE_SYSTEM); }
__device__ __forceinline__ void st_dev(unsigned* p, unsigned v) { __hip_atomic_store(p, v, __ATOMIC_RELAXED, __HIP_MEMORY_SCOPE_AGENT); }
__device__ __forceinline__ void st_dev64(u64* p, u64 v) { __hip_atomic_store(p, v, __ATOMIC_RELAXED, __HIP_MEMORY_SCOPE_AGENT); }
__device__ __forceinline__ int rfl(int v) { return __builtin_amdgcn_readfirstlane(v); }
__device__ __forceinline__ bool give_up(unsigned* tmo, unsigned call, u64 t0) {
    const unsigned tv = (unsigned)rfl((int)__hip_atomic_load(tmo, __ATOMIC_RELAXED, __HIP_MEMORY_SCOPE_AGENT));
    if (tv == call || __builtin_amdgcn_s_memrealtime() - t0 > DEADLINE) { st_dev(tmo, call); return true; }
    return false;
}
#define STAMP(i) do {} while (0)
#define BSTORE(v, rs, voff, soff, aux) do { __builtin_amdgcn_raw_buffer_store_b128(v, rs, voff, soff, aux); asm volatile("s_nop 1" :: "v"(v) : "memory"); } while (0)
#define TICKET_DRAW(dst, ptr) asm volatile("global_atomic_add %0, %1, %2, %3 sc0" : "=v"(dst) : "v"(0u), "v"(1u), "s"(ptr) : "memory")
#define TICKET_USE(v) asm volatile("" : "+v"(v))
#define VWAIT0 __builtin_amdgcn_s_waitcnt(0x0F70)
#define WAIT_V(n) asm volatile("s_waitcnt vmcnt(" #n ")" ::: "memory")
#define WAIT_L(n) asm volatile("s_waitcnt lgkmcnt(" #n ")" ::: "memory")
#define BAR do { asm volatile("" ::: "memory"); __builtin_amdgcn_s_barrier(); asm volatile("" ::: "memory"); } while (0)
#define RBAR __builtin_amdgcn_s_barrier()
#define SCHED __builtin_amdgcn_sched_barrier(0)

__device__ __forceinline__ unsigned unit_base(int src, unsigned e, unsigned g, int u) {
    return u < 4 ? (unsigned)XS_OFF + 2u * e * (unsigned)XS_T + ((unsigned)src * GM + g * 512u + (unsigned)u * 128u) * 16u
                 : (unsigned)WS_OFF + (2u * e + (unsigned)(u - 4)) * (unsigned)WS_T + ((unsigned)src * NS + g * 256u) * 16u;
}
__device__ __forceinline__ void pack_unit(u32x4 (&o)[4], unsigned& h0, unsigned& h1) {
    typedef unsigned short u16x2 __attribute__((ext_vector_type(2)));
    u16x2 m = {0, 0};
#pragma unroll
    for (int b = 0; b < 4; ++b)
#pragma unroll
        for (int i = 0; i < 4; ++i) m = __builtin_elementwise_max(m, __builtin_bit_cast(u16x2, o[b][i] & 0x7fff7fffu));
    const unsigned lm = (unsigned)(m.x > m.y ? m.x : m.y) >> 8;
    unsigned top = 0;
#pragma unroll
    for (int bit = 64; bit >= 1; bit >>= 1) if (__ballot(lm >= (top | (unsigned)bit)) != 0ull) top |= (unsigned)bit;
    const unsigned top4 = top * 0x01010101u;
    u32x4 p[3]; unsigned exc = 0; bool fit = true;
#pragma unroll
    for (int b = 0; b < 4; ++b) {
        unsigned dd[2], ov[2], cd[2];
#pragma unroll
        for (int h = 0; h < 2; ++h) {
            const unsigned hi = __builtin_amdgcn_perm(o[b][2 * h + 1], o[b][2 * h], 0x07050301u);
            p[b >> 1][(b & 1) * 2 + h] = __builtin_amdgcn_perm(o[b][2 * h + 1], o[b][2 * h], 0x06040200u);
            dd[h] = top4 - (hi & 0x7f7f7f7fu);
            ov[h] = (dd[h] + 0x79797979u) & 0x80808080u;
            cd[h] = ((dd[h] | ((ov[h] >> 4) - (ov[h] >> 7))) & 0x07070707u) | ((hi >> 4) & 0x08080808u);
        }
        p[2][b] = cd[0] | (cd[1] << 4);
        const u64 any = __ballot((ov[0] | ov[1]) != 0u);
        if (any != 0ull) {
            const int l = __builtin_ctzll(any);
            const unsigned o0 = (unsigned)__builtin_amdgcn_readlane((int)ov[0], l), o1 = (unsigned)__builtin_amdgcn_readlane((int)ov[1], l);
            const unsigned d0 = (unsigned)__builtin_amdgcn_readlane((int)dd[0], l), d1 = (unsigned)__builtin_amdgcn_readlane((int)dd[1], l);
            const unsigned ed = o0 ? (d0 >> (__builtin_ctz(o0) - 7)) & 0xffu : (d1 >> (__builtin_ctz(o1) - 7)) & 0xffu;
            const unsigned e4 = ed * 0x01010101u;
            const unsigned bad = ((dd[0] ^ e4) & (ov[0] - (ov[0] >> 7))) | ((dd[1] ^ e4) & (ov[1] - (ov[1] >> 7)));
            if (__ballot(bad != 0u) != 0ull) fit = false;
            exc |= (top - ed) << (8 * b);
        }
    }
    if (fit) { o[0] = p[0]; o[1] = p[1]; o[2] = p[2]; h0 = exc; h1 = top; }
}

__global__ __launch_bounds__(512, 2) void fused_kernel(Params P) {
    extern __shared__ __attribute__((aligned(16))) unsigned char shm_raw[];
    LAS unsigned char* lds = (LAS unsigned char*)shm_raw;
    const int tid = threadIdx.x, lane = tid & 63;
    const int wid = rfl(tid >> 6);
    const int bid = blockIdx.x, xcd = bid & 7, slot = bid >> 3;
    const int rank = P.rank;
    char* const myws = P.ws[rank];
    unsigned* const tmo = (unsigned*)(myws + TMO_OFF);
    STAMP(0);
    unsigned* const cntp = (unsigned*)(myws + CNT_OFF) + bid;
    const unsigned prev_v = __hip_atomic_load(cntp, __ATOMIC_RELAXED, __HIP_MEMORY_SCOPE_AGENT);

    if (slot < NP) {
        const int dst = (xcd + slot) & 7;
        char* const dws = P.ws[dst];
        LAS unsigned* ctl = (LAS unsigned*)(lds + 40960);
        LAS unsigned* ids1 = ctl + 16, * done1 = ids1 + 272, * ids2 = done1 + 272, * done2 = ids2 + 176;
        LAS unsigned char* hdrs = lds + 49152;
        for (int i = tid; i < 16 + 272 * 2 + 176 * 2; i += 512) ctl[i] = 0u;
        __syncthreads();
#define CLAIM(cw, ids, NSUB, k, sub, it) do { unsigned _s = 0; if (lane == 0) _s = __hip_atomic_fetch_add(ctl + (cw), 1u, __ATOMIC_RELAXED, __HIP_MEMORY_SCOPE_WORKGROUP); \
            _s = (unsigned)rfl((int)_s); k = (int)(_s / (unsigned)(NSUB)); sub = (int)(_s - (unsigned)k * (unsigned)(NSUB)); \
            if (k < 3) it = slot + k * NP; else { unsigned _v; while ((_v = (unsigned)rfl((int)*(volatile LAS unsigned*)(ids + k))) == 0u) __builtin_amdgcn_s_sleep(1); it = (int)(_v - 1u); } } while (0)
        unsigned call;
        {
            const __amdgpu_buffer_rsrc_t rs = mk_rsrc(dws, (unsigned)CTL_OFF);
            unsigned* const tk = (unsigned*)(myws + TK_OFF) + dst * 32;
            const float* xsrc = P.x + (size_t)(lane >> 2) * KS + (lane & 3) * 4;
            const float* wsrc = P.w + (size_t)dst * NS + lane * 4;
            LAS unsigned char* xb = lds + wid * 4352;
            const int xw = ((lane >> 1) & 1) * 1088 + (lane >> 2) * 16 + (lane & 1) * 8;
            const int ww = (((lane >> 1) & 1) * 2 + (lane >> 5)) * 1088 + (((lane >> 4) & 1) * 32 + (lane & 1) * 16 + ((lane >> 2) & 3) * 4) * 16;
            f32x4 v[8]; u32x4 o[4]; unsigned h0 = 0, h1 = 0;
#define P1_LOAD(it, sub) do { const int _e = (it) >> 3, _g = (it) & 7; \
            const float* _p = (sub) < 4 ? xsrc + (size_t)(_g * 512 + (sub) * 128) * KS + _e * 16 : wsrc + (size_t)(_e * 16 + ((sub) - 4) * 8) * GN + _g * 256; \
            const size_t _st = (sub) < 4 ? (size_t)16 * KS : (size_t)GN; \
            _Pragma("unroll") for (int j = 0; j < 8; ++j) v[j] = *(const f32x4*)(_p + j * _st); \
            asm volatile("" ::: "memory"); SCHED; } while (0)
#define P1_CVT(sub) do { \
            if ((sub) < 4) { _Pragma("unroll") for (int j = 0; j < 8; ++j) *(LAS u32x2*)(xb + xw + (j >> 2) * 2176 + (j & 3) * 256) = (u32x2){cvt_pk_bf16(v[j][0], v[j][1]), cvt_pk_bf16(v[j][2], v[j][3])}; } \
            else { _Pragma("unroll") for (int r = 0; r < 4; ++r) *(LAS u32x4*)(xb + ww + r * 16) = (u32x4){cvt_pk_bf16(v[0][r], v[1][r]), cvt_pk_bf16(v[2][r], v[3][r]), cvt_pk_bf16(v[4][r], v[5][r]), cvt_pk_bf16(v[6][r], v[7][r])}; } \
            WAIT_L(0); \
            _Pragma("unroll") for (int b = 0; b < 4; ++b) o[b] = *(const LAS u32x4*)(xb + b * 1088 + lane * 16); \
            WAIT_L(0); h1 = 0x80u; \
            if (dst != rank) pack_unit(o, h0, h1); \
            asm volatile("" : "+v"(o[0]), "+v"(o[1]), "+v"(o[2]), "+v"(o[3])); } while (0)
            int k0, s0, it0, k1, s1, it1;
            CLAIM(0, ids1, 6, k0, s0, it0);
            unsigned tkt = 0;
            if (it0 < NI1) { P1_LOAD(it0, s0); if (s0 == 0 && lane == 0) TICKET_DRAW(tkt, tk); }
            call = (unsigned)rfl((int)prev_v) + 1u;
            const unsigned tbase = (call - 1u) * (unsigned)NI1 - 3u * NP;
            P1_CVT(s0);
            if (s0 == 0 && it0 < NI1) { VWAIT0; asm volatile("s_waitcnt vmcnt(0)" : "+v"(tkt) :: "memory"); if (lane == 0) *(volatile LAS unsigned*)(ids1 + k0 + 3) = min(tkt - tbase, (unsigned)NI1) + 1u; }
            while (it0 < NI1) {
                CLAIM(0, ids1, 6, k1, s1, it1);
                if (it1 < NI1) { P1_LOAD(it1, s1); if (s1 == 0 && lane == 0) TICKET_DRAW(tkt, tk); }
                const unsigned e = (unsigned)it0 >> 3, g = (unsigned)it0 & 7u;
                if (h1 & 0x80u) {
                    const unsigned base = unit_base(rank, e, g, s0), st1 = s0 < 4 ? (unsigned)XS_T : 1024u, st2 = s0 < 4 ? 1024u : 2048u;
#pragma unroll
                    for (int b = 0; b < 4; ++b) BSTORE(o[b], rs, (unsigned)lane * 16u, base + (b & 1) * st1 + (b >> 1) * st2, AUX_SYS);
                } else {
                    const unsigned base = (unsigned)IB_OFF + (unsigned)((rank * NI1 + it0) * 6 + s0) * 3072u;
#pragma unroll
                    for (int b = 0; b < 3; ++b) BSTORE(o[b], rs, (unsigned)lane * 16u, base + b * 1024u, AUX_SYS);
                }
                if (lane == 0) { LAS unsigned char* hp = hdrs + k0 * 64 + (s0 >> 1) * 16; *(LAS unsigned*)(hp + 4 + (s0 & 1) * 4) = h0; hp[12 + (s0 & 1)] = (unsigned char)h1; }
                VWAIT0; asm volatile("s_waitcnt vmcnt(0) lgkmcnt(0)" : "+v"(tkt) :: "memory");
                unsigned dn = 0;
                if (lane == 0) {
                    if (s1 == 0 && it1 < NI1) *(volatile LAS unsigned*)(ids1 + k1 + 3) = min(tkt - tbase, (unsigned)NI1) + 1u;
                    dn = __hip_atomic_fetch_add(done1 + k0, 1u, __ATOMIC_RELAXED, __HIP_MEMORY_SCOPE_WORKGROUP);
                }
                if (rfl((int)dn) == 5 && lane < 3) {
                    asm volatile("" ::: "memory");
                    if (dst == rank) st_dev((unsigned*)(myws + INF_OFF) + (rank * 24 + (lane < 2 ? 2 * g + lane : 16 + g)) * 32 + e, call);
                    else { u32x4 hv = *(const LAS u32x4*)(hdrs + k0 * 64 + lane * 16); hv.x = call;
                           BSTORE(hv, mk_rsrc(dws + HDR_OFF, (unsigned)(NR * NI1 * 64)), (unsigned)lane * 16u, (unsigned)(rank * NI1 + it0) * 64u, AUX_SYS); }
                }
                if (it1 < NI1) P1_CVT(s1);
                k0 = k1; s0 = s1; it0 = it1;
            }
        }
        STAMP(1);
        if (dst != rank) {
            unsigned* const tk = (unsigned*)(myws + TK_OFF) + (8 + dst) * 32;
            const unsigned tbase = (call - 1u) * (unsigned)NI2 - 3u * NP;
            u64 rm0 = 0, rm1 = 0, rs0 = 0, rs1 = 0; bool dead = false;
            const __amdgpu_buffer_rsrc_t rr = mk_rsrc(myws + RDY_OFF, 1024u);
            u32x4 cur[8], nxt[8], rf; unsigned coff = 0, clen = 0, noff = 0, cpos = 0x3210u, npos = 0x3210u;
            unsigned mlist = 0;
            { const int j = (dst - rank - 1) & 7; int n = 0;
              for (int w = 0; w < 8; ++w) { bool direct = false; for (int i = 0; i < ND; ++i) direct |= (w < 7 && (w + 3 * i) % 7 == j); if (!direct) mlist |= (unsigned)w << (4 * n++); } }
            constexpr int MU = (8 - ND) / 2;
#define P2_KNOWN(it, sub) ((((it) < NMI ? (((sub) & 2) ? rm1 : rm0) >> (2 * (it) + ((sub) >> 2)) : ((((it) - NMI) & 1) ? rs1 : rs0) >> (((it) - NMI) >> 1)) & 1ull) != 0ull)
#define P2_RDY_LOAD() rf = __builtin_amdgcn_raw_buffer_load_b128(rr, (unsigned)lane * 16u, 0, AUX_DEV)
#define P2_RDY_USE() do { rm0 = __ballot(rf.x == call); rm1 = __ballot(rf.z == call); rs0 = __ballot(rf.y == call); rs1 = __ballot(rf.w == call); } while (0)
#define P2_WAIT(it, sub) do { const u64 _t0 = __builtin_amdgcn_s_memrealtime(); unsigned _n = 0; \
            while (!dead && !P2_KNOWN(it, sub)) { \
                asm volatile("" ::: "memory"); P2_RDY_LOAD(); P2_RDY_USE(); \
                if (P2_KNOWN(it, sub)) break; \
                if ((++_n & 31u) == 0u && give_up(tmo, call, _t0)) { dead = true; if (lane == 0) *(volatile LAS unsigned*)(ctl + 2) = 1u; } \
                __builtin_amdgcn_s_sleep(P2_SLEEP); } \
            asm volatile("" ::: "memory"); } while (0)
#define P2_LOAD(it, sub) do { if ((it) < NMI) { noff = (unsigned)MK_OFF + (unsigned)(rank * 128 + 4 * (it) + ((sub) >> 1)) * 8192u; \
                                               npos = ((mlist >> (4 * MU * ((sub) & 1))) & ((1u << (4 * MU)) - 1u)) | (0x8888u & ~((1u << (4 * MU)) - 1u)); } \
            else { noff = (unsigned)ST_OFF + (unsigned)((rank * 128 + (it) - NMI) * 8 + (sub)) * ST_SLOT; npos = 0x3210u; } \
            const __amdgpu_buffer_rsrc_t _r = mk_rsrc(myws + noff, 8192u); \
            _Pragma("unroll") for (int k = 0; k < 4; ++k) nxt[k] = __builtin_amdgcn_raw_buffer_load_b128(_r, (unsigned)lane * 16u + ((npos >> (4 * k)) & 15u) * 1024u, 0, AUX_DEV); \
            asm volatile("" ::: "memory"); } while (0)
#define P2_ROT(it) do { coff = (unsigned)rfl((int)noff); cpos = (unsigned)rfl((int)npos); clen = (it) < NMI ? 0u : min((unsigned)rfl((int)nxt[0].x), 8192u); \
            _Pragma("unroll") for (int k = 0; k < 4; ++k) cur[k] = nxt[k]; \
            if (clen > 4096u && (it) < NI2) { const __amdgpu_buffer_rsrc_t _r = mk_rsrc(myws + coff, clen); \
                _Pragma("unroll") for (int k = 4; k < 8; ++k) cur[k] = __builtin_amdgcn_raw_buffer_load_b128(_r, (unsigned)lane * 16u + k * 1024u, 0, AUX_DEV); } } while (0)
#define P2_STORE() do { const __amdgpu_buffer_rsrc_t _r = mk_rsrc(dws + coff, clen ? clen : 8192u); \
            _Pragma("unroll") for (int k = 0; k < 4; ++k) BSTORE(cur[k], _r, (unsigned)lane * 16u + ((cpos >> (4 * k)) & 15u) * 1024u, 0, AUX_SYS); \
            if (clen > 4096u) { _Pragma("unroll") for (int k = 4; k < 8; ++k) BSTORE(cur[k], _r, (unsigned)lane * 16u + k * 1024u, 0, AUX_SYS); } } while (0)
#pragma unroll
            for (int k = 0; k < 8; ++k) { nxt[k] = (u32x4){0u, 0u, 0u, 0u}; cur[k] = nxt[k]; }
            int k0, s0, it0, k1, s1, it1;
            CLAIM(1, ids2, 8, k0, s0, it0);
            unsigned tkt = 0;
            if (s0 == 0 && it0 < NI2) { if (lane == 0) TICKET_DRAW(tkt, tk); WAIT_V(0); asm volatile("" : "+v"(tkt) :: "memory"); if (lane == 0) *(volatile LAS unsigned*)(ids2 + k0 + 3) = min(tkt - tbase, (unsigned)NI2) + 1u; }
            if (it0 < NI2) { P2_WAIT(it0, s0); P2_LOAD(it0, s0); }
            STAMP(2);
            VWAIT0; P2_ROT(it0);
            while (it0 < NI2) {
                CLAIM(1, ids2, 8, k1, s1, it1);
                const bool early = it1 < NI2 && P2_KNOWN(it1, s1);
                if (early) P2_LOAD(it1, s1);
                const bool refresh = (rs0 & rs1) != ~0ull;
                if (refresh) P2_RDY_LOAD();
                if (s1 == 0 && it1 < NI2 && lane == 0) TICKET_DRAW(tkt, tk);
                P2_STORE();
                VWAIT0; asm volatile("s_waitcnt vmcnt(0)" : "+v"(tkt) :: "memory");
                if (lane == 0) {
                    if (s1 == 0 && it1 < NI2) *(volatile LAS unsigned*)(ids2 + k1 + 3) = min(tkt - tbase, (unsigned)NI2) + 1u;
                    if (__hip_atomic_fetch_add(done2 + k0, 1u, __ATOMIC_RELAXED, __HIP_MEMORY_SCOPE_WORKGROUP) == 7u && *(volatile LAS unsigned*)(ctl + 2) == 0u)
                        st_sys((unsigned*)(dws + OUF_OFF) + rank * 256 + it0, call);
                }
                if (refresh) P2_RDY_USE();
                if (!early && it1 < NI2) { P2_WAIT(it1, s1); P2_LOAD(it1, s1); VWAIT0; }
                P2_ROT(it1);
                k0 = k1; s0 = s1; it0 = it1;
            }
            STAMP(3);
        }
    } else if (slot < 16) {
        const unsigned call = (unsigned)rfl((int)prev_v) + 1u;
        const __amdgpu_buffer_rsrc_t rs = mk_rsrc(myws, (unsigned)CTL_OFF);
        const __amdgpu_buffer_rsrc_t rh = mk_rsrc(myws + HDR_OFF, (unsigned)(NR * NI1 * 64));
        constexpr int NEW = (16 - NP) * 64;
        for (int J = wid * (NEW / 8) + (bid - NP * 8); J < NI1 * 21; J += NEW) {
            const int it = J / 21, r = J - it * 21, sp = r / 3, j = r - sp * 3, src = (rank + 1 + sp) & 7;
            const unsigned e = (unsigned)it >> 3, g = (unsigned)it & 7u;
            u32x4 hv; bool dead = false;
            { const u64 t0 = __builtin_amdgcn_s_memrealtime(); unsigned n = 0;
              for (;;) {
                  asm volatile("" ::: "memory");
                  hv = __builtin_amdgcn_raw_buffer_load_b128(rh, 0u, (unsigned)((src * NI1 + it) * 64 + j * 16), AUX_SYS);
                  if ((unsigned)rfl((int)hv.x) == call) break;
                  if ((++n & 31u) == 0u && give_up(tmo, call, t0)) { dead = true; break; }
                  __builtin_amdgcn_s_sleep(EXP_SLEEP);
              }
              asm volatile("" ::: "memory"); }
            if (dead) break;
            const unsigned ibase = (unsigned)IB_OFF + (unsigned)((src * NI1 + it) * 6 + 2 * j) * 3072u;
            u32x4 p[6];
#pragma unroll
            for (int k = 0; k < 6; ++k) p[k] = __builtin_amdgcn_raw_buffer_load_b128(rs, (unsigned)lane * 16u, ibase + k * 1024u, AUX_DEV);
            VWAIT0;
            const unsigned tops = (unsigned)rfl((int)hv.w);
            const unsigned st1 = j < 2 ? (unsigned)XS_T : 1024u, st2 = j < 2 ? 1024u : 2048u;
#pragma unroll
            for (int u = 0; u < 2; ++u) {
                const unsigned top = (tops >> (8 * u)) & 0xffu, exc = (unsigned)rfl((int)(u ? hv.z : hv.y));
                if (top & 0x80u) continue;
                const unsigned t4 = top * 0x01010101u + 0x80808080u;
                const unsigned T0 = (t4 - 0x03020100u) & 0x7f7f7f7fu, T1 = (t4 - 0x00060504u) & 0x007f7f7fu;
                const unsigned base = unit_base(src, e, g, 2 * j + u);
#pragma unroll
                for (int b = 0; b < 4; ++b) { u32x4 d; const unsigned T1b = T1 | ((exc >> (8 * b)) << 24);
#pragma unroll
                    for (int h = 0; h < 2; ++h) { const unsigned c = p[3 * u + 2][b] >> (4 * h), lo = p[3 * u + (b >> 1)][(b & 1) * 2 + h];
                        const unsigned hi = __builtin_amdgcn_perm(T1b, T0, c & 0x07070707u) | ((c << 4) & 0x80808080u);
                        d[2 * h] = __builtin_amdgcn_perm(hi, lo, 0x05010400u); d[2 * h + 1] = __builtin_amdgcn_perm(hi, lo, 0x07030602u); }
                    BSTORE(d, rs, (unsigned)lane * 16u, base + (b & 1) * st1 + (b >> 1) * st2, AUX_DEV); }
            }
            WAIT_V(0);
            if (lane == 0) st_dev((unsigned*)(myws + INF_OFF) + (src * 24 + (j < 2 ? 2 * g + j : 16 + g)) * 32 + e, call);
        }
        STAMP(1);
    } else {
        const unsigned call = (unsigned)rfl((int)prev_v) + 1u;
        const int wr = wid >> 2, wc = wid & 3, fr = lane & 15, fq = lane >> 4;
        const __amdgpu_buffer_rsrc_t ro = mk_rsrc(P.out, (unsigned)((size_t)GM * GN));
        {
        const int ts = slot - 16;
        const int pm = (xcd >> 1) * 4 + (ts & 3), pn = (xcd & 1) * 4 + (ts >> 2);
        const int tile = pm * 8 + pn;
        const char* cA = myws + XS_OFF + ((size_t)wid * GM + pm * 256) * 16 + lane * 16;
        const char* cB = myws + WS_OFF + ((size_t)wid * NS + pn * 256) * 16 + lane * 16;
        const unsigned* fp = (const unsigned*)(myws + INF_OFF) + (wid * 24 + (lane < 32 ? pm : 16 + pn)) * 32 + (lane & 31);
        const unsigned ldsw = (unsigned)wid * 2048u;
        const int aoff = fq * 2048 + (wr * 64 + fr) * 16, boff = fq * 2048 + (wc * 32 + fr) * 16;
        unsigned known = 0;
        bool kdead = false;
#define SA(b, h) (((b) * 2 + (h)) * HTB)
#define SB(b, h) ((4 + (b) * 2 + (h)) * HTB)
#define STAGE(bufoff, g) do { _Pragma("unroll") for (int _i = 0; _i < 2; ++_i) \
        __builtin_amdgcn_global_load_lds((const unsigned*)((g) + _i * 1024), (LAS unsigned*)(lds + (bufoff) + ldsw + _i * 1024), 16, 0, AUX_DEV); } while (0)
#define LDA(dst, b, h) do { _Pragma("unroll") for (int m = 0; m < 4; ++m) _Pragma("unroll") for (int k = 0; k < 2; ++k) dst[m][k] = *(const LAS bf16x8*)(lds + SA(b, h) + aoff + m * 256 + k * 8192); } while (0)
#define LDB(dst, b, h) do { _Pragma("unroll") for (int n = 0; n < 2; ++n) _Pragma("unroll") for (int k = 0; k < 2; ++k) dst[n][k] = *(const LAS bf16x8*)(lds + SB(b, h) + boff + n * 256 + k * 8192); } while (0)
#define MMA(ai, bj, At, Bt) do { __builtin_amdgcn_s_setprio(1); _Pragma("unroll") for (int m = 0; m < 4; ++m) _Pragma("unroll") for (int n = 0; n < 2; ++n) _Pragma("unroll") for (int k = 0; k < 2; ++k) \
        acc[ai][bj][m][n] = __builtin_amdgcn_mfma_f32_16x16x32_bf16(Bt[n][k], At[m][k], acc[ai][bj][m][n], 0, 0, 0); __builtin_amdgcn_s_setprio(0); } while (0)
#define WAIT_EPOCH(e) do { if (!((known >> (e)) & 1u)) { const u64 _t0 = __builtin_amdgcn_s_memrealtime(); unsigned _n = 0; \
            for (;;) { const u64 _b = __ballot(ld_sys(fp) == call); known = (unsigned)_b & (unsigned)(_b >> 32); if ((known >> (e)) & 1u) break; \
                if ((++_n & 63u) == 0u && give_up(tmo, call, _t0)) { known = 0xffffffffu; kdead = true; break; } \
                __builtin_amdgcn_s_sleep(POLL_SLEEP); } \
            asm volatile("" ::: "memory"); } } while (0)
        f32x4 acc[2][2][4][2];
#pragma unroll
        for (int a = 0; a < 2; ++a)
#pragma unroll
            for (int b = 0; b < 2; ++b)
#pragma unroll
                for (int m = 0; m < 4; ++m)
#pragma unroll
                    for (int n = 0; n < 2; ++n) acc[a][b][m][n] = (f32x4){0.f, 0.f, 0.f, 0.f};
        bf16x8 At[4][2], B0[2][2], B1[2][2];
        WAIT_EPOCH(0);
        STAMP(1);
        STAGE(SB(0, 0), cB); STAGE(SA(0, 0), cA); STAGE(SB(0, 1), cB + 2048); STAGE(SA(0, 1), cA + 2048);
        if (wr == 1) RBAR;
        WAIT_V(4); RBAR;
        STAGE(SB(1, 0), cB + WS_T); STAGE(SA(1, 0), cA + XS_T); STAGE(SB(1, 1), cB + WS_T + 2048);
        WAIT_V(6); RBAR;
        for (int e = 0; e < NEP; ++e) {
            const bool last = (e == NEP - 1);
            const char* a1 = cA + (size_t)(2 * e + 1) * XS_T;
            const int t2 = last ? 2 * e : 2 * e + 2;
            const char* a2 = cA + (size_t)t2 * XS_T; const char* b2 = cB + (size_t)t2 * WS_T;
            const char* a3 = a2 + XS_T; const char* b3 = b2 + WS_T;
            if (!last) WAIT_EPOCH(e + 1);
            if (e == 15) STAMP(7); if (e == 27) STAMP(8); if (e == 29) STAMP(9); if (e == 30) STAMP(10);
            LDB(B0, 0, 0); SCHED; LDA(At, 0, 0); STAGE(SA(1, 1), a1 + 2048);
            WAIT_L(8); RBAR; WAIT_L(0); MMA(0, 0, At, B0); RBAR; SCHED;
            LDB(B1, 0, 1); STAGE(SB(0, 0), b2);
            RBAR; WAIT_L(0); MMA(0, 1, At, B1); RBAR;
            LDA(At, 0, 1); STAGE(SA(0, 0), a2);
            RBAR; WAIT_L(0); MMA(1, 0, At, B0); RBAR; SCHED;
            STAGE(SB(0, 1), b2 + 2048);
            WAIT_V(6); RBAR; MMA(1, 1, At, B1); RBAR;
            LDB(B0, 1, 0); SCHED; LDA(At, 1, 0); STAGE(SA(0, 1), a2 + 2048);
            WAIT_L(8); RBAR; WAIT_L(0); MMA(0, 0, At, B0); RBAR; SCHED;
            LDB(B1, 1, 1); STAGE(SB(1, 0), b3);
            RBAR; WAIT_L(0); MMA(0, 1, At, B1); RBAR;
            LDA(At, 1, 1); STAGE(SA(1, 0), a3);
            RBAR; WAIT_L(0); MMA(1, 0, At, B0); RBAR; SCHED;
            STAGE(SB(1, 1), b3 + 2048);
            WAIT_V(6); RBAR; MMA(1, 1, At, B1); RBAR;
        }
        if (wr == 0) RBAR;
        STAMP(2);
        u32x4 mk;
#pragma unroll
        for (int g2 = 0; g2 < 4; ++g2) { unsigned mm = 0;
#pragma unroll
            for (int e = 31; e >= 0; --e) { const int g = g2 * 2 + (e >> 4), d = (e >> 2) & 3; mm = __builtin_amdgcn_alignbit(mm, __float_as_uint(acc[g >> 2][d >> 1][g & 3][d & 1][e & 3]), 31); }
            mk[g2] = ~mm; }
        const unsigned reg_id = (unsigned)((rank * 128 + tile) * 8 + wid);
        BSTORE(mk, mk_rsrc(myws + MK_OFF, (unsigned)MK_BYTES), (unsigned)lane * 16u, reg_id * 1024u, AUX_DEV);
#pragma unroll
        for (int i = 0; i < ND; ++i) { const int j = (wid + 3 * i) % 7;
            BSTORE(mk, mk_rsrc(P.ws[(rank + 1 + j) & 7] + MK_OFF, wid < 7 ? (unsigned)MK_BYTES : 0u), (unsigned)lane * 16u, reg_id * 1024u, AUX_SYS); }
        SCHED;
        LAS float* red = (LAS float*)(lds + 8 * HTB);
        float mx = 0.f;
#pragma unroll
        for (int a = 0; a < 2; ++a)
#pragma unroll
            for (int b = 0; b < 2; ++b)
#pragma unroll
                for (int m = 0; m < 4; ++m)
#pragma unroll
                    for (int n = 0; n < 2; ++n) { const f32x4 v = acc[a][b][m][n]; mx = fmaxf(fmaxf(mx, fmaxf(v[0], v[1])), fmaxf(v[2], v[3])); }
#pragma unroll
        for (int o = 32; o >= 1; o >>= 1) mx = fmaxf(mx, __shfl_xor(mx, o));
        if (lane == 0) red[wid] = kdead ? -1.0f : mx;
        __builtin_amdgcn_s_waitcnt(0x0F70 | ND);
        WAIT_L(0); BAR;
        {
            float m8 = red[0], n8 = red[0];
#pragma unroll
            for (int i = 1; i < 8; ++i) { m8 = fmaxf(m8, red[i]); n8 = fminf(n8, red[i]); }
            if (lane == 0 && n8 >= 0.f) {
                if (wid == 0) st_dev((unsigned*)(myws + RDY_OFF) + tile * 2, call);
                st_sys64((u64*)(P.ws[wid] + AMX_OFF) + rank * 128 + tile, ((u64)call << 32) | (u64)__float_as_uint(m8));
            }
        }
        LAS unsigned char* sbuf = lds + wid * 8192;
#pragma unroll
        for (int i = 0; i < 8; ++i) *(LAS u32x4*)(sbuf + i * 1024 + lane * 16) = (u32x4){0u, 0u, 0u, 0u};
        const unsigned cntnz = __builtin_popcount(mk.x) + __builtin_popcount(mk.y) + __builtin_popcount(mk.z) + __builtin_popcount(mk.w);
        {
            const u64* gp = (const u64*)(myws + AMX_OFF) + wid * 128 + lane;
            const u64 t0 = __builtin_amdgcn_s_memrealtime(); unsigned n = 0; float gm; bool dead = false;
            for (;;) {
                const u64 g0 = ld_sys64(gp), g1 = ld_sys64(gp + 64);
                gm = fmaxf(__uint_as_float((unsigned)g0), __uint_as_float((unsigned)g1));
                if (__all((unsigned)(g0 >> 32) == call && (unsigned)(g1 >> 32) == call)) break;
                if ((++n & 63u) == 0u && give_up(tmo, call, t0)) { dead = true; break; }
                __builtin_amdgcn_s_sleep(1);
            }
#pragma unroll
            for (int o = 32; o >= 1; o >>= 1) gm = fmaxf(gm, __shfl_xor(gm, o));
            if (lane == 0) red[8 + wid] = dead ? -1.0f : gm;
        }
        WAIT_L(0); BAR;
        float amax = red[8], amin = red[8];
#pragma unroll
        for (int i = 1; i < 8; ++i) { amax = fmaxf(amax, red[8 + i]); amin = fminf(amin, red[8 + i]); }
        const bool bad = rfl(amin < 0.f ? 1 : 0) != 0;
        const float rinv = amax > 0.f ? 1.0f / amax : 0.f;
        STAMP(3);
        unsigned allor = 0;
        u32x4 qloc[8];
        {
            const float c127 = __uint_as_float(127u);
#pragma unroll
            for (int g = 0; g < 8; ++g) { u32x4& qb = qloc[g];
#pragma unroll
                for (int d = 0; d < 4; ++d) { f32x4& v = acc[g >> 2][d >> 1][g & 3][d & 1];
#pragma unroll
                    for (int j = 0; j < 4; ++j) { float t; asm("v_mul_f32_e64 %0, %1, %2 clamp" : "=v"(t) : "v"(v[j]), "v"(rinv)); asm("v_mul_f32_e32 %0, %1, %2" : "=v"(t) : "v"(c127), "v"(t)); v[j] = t; }
                    const unsigned w = __float_as_uint(v[0]) | (__float_as_uint(v[1]) << 8) | (__float_as_uint(v[2]) << 16) | (__float_as_uint(v[3]) << 24);
                    allor |= w; qb[d] = bad ? 0x80808080u : w; }
            }
        }
        allor |= allor >> 16; allor |= allor >> 8; allor &= 0xffu;
        const unsigned W = 31u - (unsigned)__builtin_clz(2u * allor + 1u);
        unsigned incl = cntnz * W;
#pragma unroll
        for (int o = 1; o < 64; o <<= 1) { const unsigned t = __shfl_up(incl, o); if (lane >= o) incl += t; }
        const unsigned total_bits = HDR_BITS + (unsigned)__builtin_amdgcn_readlane((int)incl, 63);
        const unsigned nbytes = (total_bits + 127u) / 128u * 16u;
        unsigned Pb = HDR_BITS + incl - cntnz * W;
        if (lane == 0) *(LAS unsigned*)sbuf = nbytes;
        { const u64 h = (u64)W << ((3u * lane) & 31u); LAS unsigned* hp = (LAS unsigned*)(sbuf + 4 + ((3u * lane) >> 5) * 4u);
          __hip_atomic_fetch_or(hp, (unsigned)h, __ATOMIC_RELAXED, __HIP_MEMORY_SCOPE_WORKGROUP); __hip_atomic_fetch_or(hp + 1, (unsigned)(h >> 32), __ATOMIC_RELAXED, __HIP_MEMORY_SCOPE_WORKGROUP); }
#pragma unroll
        for (int g = 0; g < 8; ++g)
#pragma unroll
            for (int d = 0; d < 4; ++d) { const f32x4 v = acc[g >> 2][d >> 1][g & 3][d & 1]; const unsigned mw = mk[g >> 1]; const int b0 = (g & 1) * 16 + d * 4;
                unsigned ch = __float_as_uint(v[0]); unsigned p = ((mw >> b0) & 1u) * W;
                ch |= __float_as_uint(v[1]) << p; p += ((mw >> (b0 + 1)) & 1u) * W;
                ch |= __float_as_uint(v[2]) << p; p += ((mw >> (b0 + 2)) & 1u) * W;
                ch |= __float_as_uint(v[3]) << p; p += ((mw >> (b0 + 3)) & 1u) * W;
                const u64 h = (u64)ch << (Pb & 31u); LAS unsigned* hp = (LAS unsigned*)(sbuf + ((Pb >> 3) & ~3u));
                __hip_atomic_fetch_or(hp, (unsigned)h, __ATOMIC_RELAXED, __HIP_MEMORY_SCOPE_WORKGROUP); __hip_atomic_fetch_or(hp + 1, (unsigned)(h >> 32), __ATOMIC_RELAXED, __HIP_MEMORY_SCOPE_WORKGROUP);
                Pb += p; }
        WAIT_L(0);
        {
            const __amdgpu_buffer_rsrc_t rd = mk_rsrc(myws + ST_OFF + (size_t)reg_id * ST_SLOT, nbytes);
#pragma unroll
            for (int k = 0; k < 8; ++k) { const u32x4 sv = *(const LAS u32x4*)(sbuf + k * 1024 + lane * 16); BSTORE(sv, rd, (unsigned)lane * 16u + k * 1024u, 0, AUX_DEV); }
            WAIT_V(0); WAIT_L(0); BAR;
            if (tid == 0 && !bad) st_dev((unsigned*)(myws + RDY_OFF) + tile * 2 + 1, call);
        }
        STAMP(4);
        {
            const unsigned voff = (unsigned)((pm * 256 + wr * 64 + fr) * GN + rank * NS + pn * 256 + wc * 64 + fq * 16);
#pragma unroll
            for (int g = 0; g < 8; ++g) BSTORE(qloc[g], ro, voff, (unsigned)(((g >> 2) * 128 + (g & 3) * 16) * GN), OUT_AUX);
        }
        }
        {
        LAS unsigned char* sbuf = lds + wid * 8448;
        const __amdgpu_buffer_rsrc_t rm = mk_rsrc(myws + MK_OFF, (unsigned)MK_BYTES);
        for (int i = 0; i < NR - 1; ++i) {
            const int J = (bid - 128) + 128 * i, tile = J / 7, src = (rank + 1 + (J - tile * 7)) & 7;
            const int pm = tile >> 3, pn = tile & 7;
            bool bad = false;
            {
                const unsigned* fp = (const unsigned*)(myws + OUF_OFF) + src * 256 + ((lane & 1) ? NMI + tile : (tile >> 2));
                const u64 t0 = __builtin_amdgcn_s_memrealtime(); unsigned n = 0;
                while (!__all(ld_sys(fp) == call)) {
                    if ((++n & 15u) == 0u && give_up(tmo, call, t0)) { bad = true; break; }
                    __builtin_amdgcn_s_sleep(4);
                }
                asm volatile("" ::: "memory");
            }
            if (i == 0) STAMP(5);
            if (i == 6) STAMP(11);
            const unsigned reg_id = (unsigned)((src * 128 + tile) * 8 + wid);
            const char* sp = myws + ST_OFF + (size_t)reg_id * ST_SLOT + lane * 16;
#pragma unroll
            for (int k = 0; k < 8; ++k) __builtin_amdgcn_global_load_lds((const unsigned*)(sp + k * 1024), (LAS unsigned*)(sbuf + k * 1024), 16, 0, AUX_DEV);
            const u32x4 mk = __builtin_amdgcn_raw_buffer_load_b128(rm, (unsigned)lane * 16u, reg_id * 1024u, AUX_DEV);
            const unsigned cntnz = __builtin_popcount(mk.x) + __builtin_popcount(mk.y) + __builtin_popcount(mk.z) + __builtin_popcount(mk.w);
            WAIT_V(0);
            unsigned W;
            { const LAS unsigned* hp = (const LAS unsigned*)(sbuf + 4 + ((3u * lane) >> 5) * 4u); W = __builtin_amdgcn_alignbit(hp[1], hp[0], (3u * lane) & 31u) & 7u; }
            unsigned incl = cntnz * W;
#pragma unroll
            for (int o = 1; o < 64; o <<= 1) { const unsigned t = __shfl_up(incl, o); if (lane >= o) incl += t; }
            unsigned Pb = HDR_BITS + incl - cntnz * W;
            const unsigned voff = (unsigned)((pm * 256 + wr * 64 + fr) * GN + src * NS + pn * 256 + wc * 64 + fq * 16);
#pragma unroll
            for (int g = 0; g < 8; ++g) { u32x4 qb;
#pragma unroll
                for (int d = 0; d < 4; ++d) { const unsigned mw = mk[g >> 1]; const int b0 = (g & 1) * 16 + d * 4;
                    const LAS unsigned* hp = (const LAS unsigned*)(sbuf + ((Pb >> 3) & ~3u));
                    const unsigned win = __builtin_amdgcn_alignbit(hp[1], hp[0], Pb & 31u);
                    const unsigned w0 = ((mw >> b0) & 1u) * W, w1 = ((mw >> (b0 + 1)) & 1u) * W, w2 = ((mw >> (b0 + 2)) & 1u) * W, w3 = ((mw >> (b0 + 3)) & 1u) * W;
                    const unsigned p1 = w0, p2 = p1 + w1, p3 = p2 + w2;
                    const unsigned o = __builtin_amdgcn_ubfe(win, 0u, w0) | (__builtin_amdgcn_ubfe(win, p1, w1) << 8) | (__builtin_amdgcn_ubfe(win, p2, w2) << 16) | (__builtin_amdgcn_ubfe(win, p3, w3) << 24);
                    qb[d] = bad ? 0x80808080u : o; Pb += p3 + w3; }
                BSTORE(qb, ro, voff, (unsigned)(((g >> 2) * 128 + (g & 3) * 16) * GN), OUT_AUX);
            }
            WAIT_L(0);
        }
        STAMP(12);
        }
    }
    __syncthreads();
    STAMP(6);
    if (tid == 0) st_dev(cntp, prev_v + 1u);
}
}

extern "C" void kernel_launch(int rank, int nranks, ncclComm_t comm, hipStream_t stream,
                              void* const* d_in, const int* in_sizes, int n_in,
                              void* d_out, int out_size, const int* in_dtypes, int out_dtype,
                              void* const* d_out_all, void* const* d_ws_all, size_t ws_bytes) {
    if (nranks != NR || ws_bytes < WS_NEED || (unsigned)rank >= (unsigned)NR) { fprintf(stderr, "unexpected configuration\n"); return; }
    HIP_CHECK(hipFuncSetAttribute((const void*)fused_kernel, hipFuncAttributeMaxDynamicSharedMemorySize, LDS_BYTES));
    Params P;
    P.x = (const float*)d_in[0]; P.w = (const float*)d_in[1]; P.rank = rank; P.out = (signed char*)d_out;
    for (int p = 0; p < NR; ++p) P.ws[p] = (char*)d_ws_all[p];
    hipLaunchKernelGGL(fused_kernel, dim3(256), dim3(512), LDS_BYTES, stream, P);
    HIP_CHECK(hipGetLastError());
}
```

```cpp
#include <hip/hip_runtime.h>
#include <rccl/rccl.h>
#include <cstdio>

#define HIP_CHECK(expr) do { hipError_t _e = (expr); if (_e != hipSuccess) { \
    fprintf(stderr, "HIP error %s at %s:%d\n", hipGetErrorString(_e), __FILE__, __LINE__); \
    return; } } while (0)

namespace {
#define LAS __attribute__((address_space(3)))
typedef short bf16x8 __attribute__((ext_vector_type(8)));
typedef float f32x4 __attribute__((ext_vector_type(4)));
typedef unsigned u32x4 __attribute__((ext_vector_type(4)));
typedef unsigned u32x2 __attribute__((ext_vector_type(2)));
typedef unsigned long long u64;

constexpr int NR = 8;
constexpr int GM = 4096, GN = 16384;
constexpr int KS = 512;
constexpr int NS = GN / NR;
constexpr int NT = 64;
constexpr int NEP = NT / 2;
#define NP 8
constexpr int NI1 = NEP * 8;
constexpr int NMI = 32;
constexpr int NI2 = NMI + 128;
constexpr size_t XS_T = (size_t)NR * GM * 16;
constexpr size_t WS_T = (size_t)NR * NS * 16;
constexpr size_t XS_OFF = 0, XS_BYTES = XS_T * NT;
constexpr size_t WS_OFF = XS_BYTES, WS_BYTES = WS_T * NT;
constexpr size_t MK_OFF = WS_OFF + WS_BYTES;
constexpr size_t MK_BYTES = (size_t)NR * 128 * 8 * 1024;
constexpr unsigned ST_SLOT = 59 * 1024 + 128;
constexpr size_t ST_OFF = MK_OFF + MK_BYTES;
constexpr size_t ST_BYTES = (size_t)NR * 128 * ST_SLOT;
constexpr size_t IB_OFF = ST_OFF + ST_BYTES;
constexpr size_t IB_BYTES = (size_t)NR * NI1 * 6 * 3072;
constexpr size_t CTL_OFF = IB_OFF + IB_BYTES;
constexpr size_t HDR_OFF = CTL_OFF;
constexpr size_t INF_OFF = HDR_OFF + (size_t)NR * NI1 * 64;
constexpr size_t AMX_OFF = INF_OFF + 24576;
constexpr size_t OUF_OFF = AMX_OFF + 8192;
constexpr size_t RDY_OFF = OUF_OFF + 8192;
constexpr size_t TK_OFF = RDY_OFF + 2048;
constexpr size_t CNT_OFF = TK_OFF + 2048;
constexpr size_t TMO_OFF = CNT_OFF + 2048;
constexpr size_t DBG_OFF = TMO_OFF + 2048;
constexpr size_t DB2_OFF = DBG_OFF + 256 * 16 * 8;
constexpr size_t WS_NEED = DB2_OFF + 3 * 8 * 1024 * 8;
constexpr u64 DEADLINE = 100000000ull;
constexpr int HTB = 16384;
constexpr int LDS_BYTES = 8 * HTB + 1024;
#define POLL_SLEEP 1
#define ND 2
#define P2_SLEEP 4
#define EXP_SLEEP 2
#define E_RAW 32
#define LA 2
#define LA2 3
#define TAPER 0
#define PW 8
#define DEC_SLEEP 2
#define OUT_AUX 0
constexpr unsigned HDR_BITS = 256;
constexpr unsigned RH_BITS = 2048 + 192;
constexpr unsigned RMAX_BITS = RH_BITS + 8192 * 7;
constexpr int AUX_SYS = 17;
constexpr int AUX_DEV = 16;

struct Params {
    const float* x; const float* w;
    char* ws[NR]; signed char* out;
    int rank;
};

__device__ __forceinline__ unsigned cvt_pk_bf16(float lo, float hi) { unsigned r; asm("v_cvt_pk_bf16_f32 %0, %1, %2" : "=v"(r) : "v"(lo), "v"(hi)); return r; }
__device__ __forceinline__ __amdgpu_buffer_rsrc_t mk_rsrc(const void* p, unsigned bytes) { return __builtin_amdgcn_make_buffer_rsrc((void*)p, (short)0, (int)bytes, 0x00020000); }
__device__ __forceinline__ unsigned ld_sys(const unsigned* p) { return __hip_atomic_load(p, __ATOMIC_RELAXED, __HIP_MEMORY_SCOPE_SYSTEM); }
__device__ __forceinline__ u64 ld_sys64(const u64* p) { return __hip_atomic_load(p, __ATOMIC_RELAXED, __HIP_MEMORY_SCOPE_SYSTEM); }
__device__ __forceinline__ u64 ld_dev64(const u64* p) { return __hip_atomic_load(p, __ATOMIC_RELAXED, __HIP_MEMORY_SCOPE_AGENT); }
__device__ __forceinline__ void st_sys(unsigned* p, unsigned v) { __hip_atomic_store(p, v, __ATOMIC_RELAXED, __HIP_MEMORY_SCOPE_SYSTEM); }
__device__ __forceinline__ void st_sys64(u64* p, u64 v) { __hip_atomic_store(p, v, __ATOMIC_RELAXED, __HIP_MEMORY_SCOPE_SYSTEM); }
__device__ __forceinline__ void st_dev(unsigned* p, unsigned v) { __hip_atomic_store(p, v, __ATOMIC_RELAXED, __HIP_MEMORY_SCOPE_AGENT); }
__device__ __forceinline__ void st_dev64(u64* p, u64 v) { __hip_atomic_store(p, v, __ATOMIC_RELAXED, __HIP_MEMORY_SCOPE_AGENT); }
__device__ __forceinline__ int rfl(int v) { return __builtin_amdgcn_readfirstlane(v); }
__device__ __forceinline__ unsigned wave_scan(unsigned v) {
    v += (unsigned)__builtin_amdgcn_update_dpp(0, (int)v, 0x111, 0xf, 0xf, false); v += (unsigned)__builtin_amdgcn_update_dpp(0, (int)v, 0x112, 0xf, 0xf, false);
    v += (unsigned)__builtin_amdgcn_update_dpp(0, (int)v, 0x114, 0xf, 0xf, false); v += (unsigned)__builtin_amdgcn_update_dpp(0, (int)v, 0x118, 0xf, 0xf, false);
    v += (unsigned)__builtin_amdgcn_update_dpp(0, (int)v, 0x142, 0xa, 0xf, false); v += (unsigned)__builtin_amdgcn_update_dpp(0, (int)v, 0x143, 0xc, 0xf, false);
    return v;
}
__device__ __forceinline__ unsigned wave_umax(unsigned v) {
    v = max(v, (unsigned)__builtin_amdgcn_update_dpp(0, (int)v, 0x111, 0xf, 0xf, false)); v = max(v, (unsigned)__builtin_amdgcn_update_dpp(0, (int)v, 0x112, 0xf, 0xf, false));
    v = max(v, (unsigned)__builtin_amdgcn_update_dpp(0, (int)v, 0x114, 0xf, 0xf, false)); v = max(v, (unsigned)__builtin_amdgcn_update_dpp(0, (int)v, 0x118, 0xf, 0xf, false));
    v = max(v, (unsigned)__builtin_amdgcn_update_dpp(0, (int)v, 0x142, 0xa, 0xf, false)); v = max(v, (unsigned)__builtin_amdgcn_update_dpp(0, (int)v, 0x143, 0xc, 0xf, false));
    return v;
}
__device__ __forceinline__ float max3f(float a, float b, float c) { float r; asm("v_max3_f32 %0, %1, %2, %3" : "=v"(r) : "v"(a), "v"(b), "v"(c)); return r; }
__device__ __forceinline__ float quant(float y, float rinv, float c127) { float t; asm("v_mul_f32_e64 %0, %1, %2 clamp" : "=v"(t) : "v"(y), "v"(rinv)); asm("v_mul_f32_e32 %0, %1, %2" : "=v"(t) : "v"(c127), "v"(t)); return t; }
__device__ __forceinline__ void lds_or_bits(LAS unsigned char* base, unsigned bit, unsigned val) {
    const u64 h = (u64)val << (bit & 31u); LAS unsigned* hp = (LAS unsigned*)(base + ((bit >> 3) & ~3u));
    __hip_atomic_fetch_or(hp, (unsigned)h, __ATOMIC_RELAXED, __HIP_MEMORY_SCOPE_WORKGROUP); __hip_atomic_fetch_or(hp + 1, (unsigned)(h >> 32), __ATOMIC_RELAXED, __HIP_MEMORY_SCOPE_WORKGROUP);
}
__device__ __forceinline__ unsigned lds_get_bits(const LAS unsigned char* base, unsigned bit) {
    const LAS unsigned* hp = (const LAS unsigned*)(base + ((bit >> 3) & ~3u)); return __builtin_amdgcn_alignbit(hp[1], hp[0], bit & 31u);
}
__device__ __forceinline__ bool give_up(unsigned* tmo, unsigned call, u64 t0) {
    const unsigned tv = (unsigned)rfl((int)__hip_atomic_load(tmo, __ATOMIC_RELAXED, __HIP_MEMORY_SCOPE_AGENT));
    if (tv == call || __builtin_amdgcn_s_memrealtime() - t0 > DEADLINE) { st_dev(tmo, call); return true; }
    return false;
}
#define STAMP(i) do {} while (0)
#define STAMP2(a, b, c) do {} while (0)
#define BSTORE(v, rs, voff, soff, aux) do { __builtin_amdgcn_raw_buffer_store_b128(v, rs, voff, soff, aux); asm volatile("s_nop 1" :: "v"(v) : "memory"); } while (0)
#define TICKET_DRAW(dst, ptr) asm volatile("global_atomic_add %0, %1, %2, %3 sc0" : "=v"(dst) : "v"(0u), "v"(1u), "s"(ptr) : "memory")
#define TICKET_USE(v) asm volatile("" : "+v"(v))
#define VWAIT0 __builtin_amdgcn_s_waitcnt(0x0F70)
#define WAIT_V(n) asm volatile("s_waitcnt vmcnt(" #n ")" ::: "memory")
#define WAIT_VX(n) WAIT_V(n)
#define WAIT_L(n) asm volatile("s_waitcnt lgkmcnt(" #n ")" ::: "memory")
#define BAR do { asm volatile("" ::: "memory"); __builtin_amdgcn_s_barrier(); asm volatile("" ::: "memory"); } while (0)
#define RBAR __builtin_amdgcn_s_barrier()
#define SCHED __builtin_amdgcn_sched_barrier(0)

__device__ __forceinline__ unsigned unit_base(int src, unsigned e, unsigned g, int u) {
    return u < 4 ? (unsigned)XS_OFF + 2u * e * (unsigned)XS_T + ((unsigned)src * GM + g * 512u + (unsigned)u * 128u) * 16u
                 : (unsigned)WS_OFF + (2u * e + (unsigned)(u - 4)) * (unsigned)WS_T + ((unsigned)src * NS + g * 256u) * 16u;
}
__device__ __forceinline__ void pack_unit(u32x4 (&o)[4], unsigned& h0, unsigned& h1) {
    typedef unsigned short u16x2 __attribute__((ext_vector_type(2)));
    u16x2 m = {0, 0};
#pragma unroll
    for (int b = 0; b < 4; ++b)
#pragma unroll
        for (int i = 0; i < 4; ++i) m = __builtin_elementwise_max(m, __builtin_bit_cast(u16x2, o[b][i] & 0x7fff7fffu));
    const unsigned lm = (unsigned)(m.x > m.y ? m.x : m.y) >> 8;
    unsigned top = 0;
#pragma unroll
    for (int bit = 64; bit >= 1; bit >>= 1) if (__ballot(lm >= (top | (unsigned)bit)) != 0ull) top |= (unsigned)bit;
    const unsigned top4 = top * 0x01010101u;
    u32x4 p[3]; unsigned exc = 0; bool fit = true;
#pragma unroll
    for (int b = 0; b < 4; ++b) {
        unsigned dd[2], ov[2], cd[2];
#pragma unroll
        for (int h = 0; h < 2; ++h) {
            const unsigned hi = __builtin_amdgcn_perm(o[b][2 * h + 1], o[b][2 * h], 0x07050301u);
            p[b >> 1][(b & 1) * 2 + h] = __builtin_amdgcn_perm(o[b][2 * h + 1], o[b][2 * h], 0x06040200u);
            dd[h] = top4 - (hi & 0x7f7f7f7fu);
            ov[h] = (dd[h] + 0x79797979u) & 0x80808080u;
            cd[h] = ((dd[h] | ((ov[h] >> 4) - (ov[h] >> 7))) & 0x07070707u) | ((hi >> 4) & 0x08080808u);
        }
        p[2][b] = cd[0] | (cd[1] << 4);
        const u64 any = __ballot((ov[0] | ov[1]) != 0u);
        if (any != 0ull) {
            const int l = __builtin_ctzll(any);
            const unsigned o0 = (unsigned)__builtin_amdgcn_readlane((int)ov[0], l), o1 = (unsigned)__builtin_amdgcn_readlane((int)ov[1], l);
            const unsigned d0 = (unsigned)__builtin_amdgcn_readlane((int)dd[0], l), d1 = (unsigned)__builtin_amdgcn_readlane((int)dd[1], l);
            const unsigned ed = o0 ? (d0 >> (__builtin_ctz(o0) - 7)) & 0xffu : (d1 >> (__builtin_ctz(o1) - 7)) & 0xffu;
            const unsigned e4 = ed * 0x01010101u;
            const unsigned bad = ((dd[0] ^ e4) & (ov[0] - (ov[0] >> 7))) | ((dd[1] ^ e4) & (ov[1] - (ov[1] >> 7)));
            if (__ballot(bad != 0u) != 0ull) fit = false;
            exc |= (top - ed) << (8 * b);
        }
    }
    if (fit) { o[0] = p[0]; o[1] = p[1]; o[2] = p[2]; h0 = exc; h1 = top; }
}

__global__ __launch_bounds__(512, 2) void fused_kernel(Params P) {
    extern __shared__ __attribute__((aligned(16))) unsigned char shm_raw[];
    LAS unsigned char* lds = (LAS unsigned char*)shm_raw;
    const int tid = threadIdx.x, lane = tid & 63;
    const int wid = rfl(tid >> 6);
    const int bid = blockIdx.x, xcd = bid & 7, slot = bid >> 3;
    const int rank = P.rank;
    char* const myws = P.ws[rank];
    unsigned* const tmo = (unsigned*)(myws + TMO_OFF);
    STAMP(0);
    unsigned* const cntp = (unsigned*)(myws + CNT_OFF) + bid;
    const unsigned prev_v = __hip_atomic_load(cntp, __ATOMIC_RELAXED, __HIP_MEMORY_SCOPE_AGENT);

    if (slot < NP) {
        const int dst = (xcd + slot) & 7;
        char* const dws = P.ws[dst];
        LAS unsigned* ctl = (LAS unsigned*)(lds + 40960);
        LAS unsigned* ids1 = ctl + 16, * done1 = ids1 + 288, * ids2 = done1 + 288, * done2 = ids2 + 192;
        LAS unsigned char* hdrs = lds + 49152;
        for (int i = tid; i < 16 + 288 * 2 + 192 * 2; i += 512) ctl[i] = 0u;
        __syncthreads();
#define CLAIM(cw, ids, NSUB, LAH, NI, SKIP, k, sub, it) for (;;) { unsigned _s = 0; if (lane == 0) _s = __hip_atomic_fetch_add(ctl + (cw), 1u, __ATOMIC_RELAXED, __HIP_MEMORY_SCOPE_WORKGROUP); \
            _s = (unsigned)rfl((int)_s); k = (int)(_s / (unsigned)(NSUB)); sub = (int)(_s - (unsigned)k * (unsigned)(NSUB)); \
            if (k < (LAH)) it = slot + k * NP; else { unsigned _v; while ((_v = (unsigned)rfl((int)*(volatile LAS unsigned*)(ids + k))) == 0u) __builtin_amdgcn_s_sleep(1); it = (int)(_v - 1u); } \
            if (it < (NI)) { if (SKIP) continue; break; } \
            if (sub == 0 && lane == 0) *(volatile LAS unsigned*)(ids + k + (LAH)) = (unsigned)(NI) + 1u; \
            bool _all = true; \
            _Pragma("unroll") for (int _j = 1; _j < (LAH); ++_j) _all &= (unsigned)rfl((int)*(volatile LAS unsigned*)(ids + k - _j)) == (unsigned)(NI) + 1u; \
            if (_all) break; }
        unsigned call;
        {
            const __amdgpu_buffer_rsrc_t rs = mk_rsrc(dws, (unsigned)CTL_OFF);
            unsigned* const tk = (unsigned*)(myws + TK_OFF) + dst * 32;
            const float* xsrc = P.x + (size_t)(lane >> 2) * KS + (lane & 3) * 4;
            const float* wsrc = P.w + (size_t)dst * NS + lane * 4;
            LAS unsigned char* xb = lds + wid * 4352;
            const int xw = ((lane >> 1) & 1) * 1088 + (lane >> 2) * 16 + (lane & 1) * 8;
            const int ww = (((lane >> 1) & 1) * 2 + (lane >> 5)) * 1088 + (((lane >> 4) & 1) * 32 + (lane & 1) * 16 + ((lane >> 2) & 3) * 4) * 16;
            f32x4 v[8]; u32x4 o[4]; unsigned h0 = 0, h1 = 0;
#define P1_LOAD(it, sub) do { const int _e = (it) >> 3, _g = (it) & 7; \
            const float* _p = (sub) < 4 ? xsrc + (size_t)(_g * 512 + (sub) * 128) * KS + _e * 16 : wsrc + (size_t)(_e * 16 + ((sub) - 4) * 8) * GN + _g * 256; \
            const size_t _st = (sub) < 4 ? (size_t)16 * KS : (size_t)GN; \
            _Pragma("unroll") for (int j = 0; j < 8; ++j) v[j] = *(const f32x4*)(_p + j * _st); \
            asm volatile("" ::: "memory"); SCHED; } while (0)
#define P1_CVT(it, sub) do { \
            if ((sub) < 4) { _Pragma("unroll") for (int j = 0; j < 8; ++j) *(LAS u32x2*)(xb + xw + (j >> 2) * 2176 + (j & 3) * 256) = (u32x2){cvt_pk_bf16(v[j][0], v[j][1]), cvt_pk_bf16(v[j][2], v[j][3])}; } \
            else { _Pragma("unroll") for (int r = 0; r < 4; ++r) *(LAS u32x4*)(xb + ww + r * 16) = (u32x4){cvt_pk_bf16(v[0][r], v[1][r]), cvt_pk_bf16(v[2][r], v[3][r]), cvt_pk_bf16(v[4][r], v[5][r]), cvt_pk_bf16(v[6][r], v[7][r])}; } \
            WAIT_L(0); \
            _Pragma("unroll") for (int b = 0; b < 4; ++b) o[b] = *(const LAS u32x4*)(xb + b * 1088 + lane * 16); \
            WAIT_L(0); h1 = 0x80u; \
            if (dst != rank && (it) < E_RAW * 8) pack_unit(o, h0, h1); \
            asm volatile("" : "+v"(o[0]), "+v"(o[1]), "+v"(o[2]), "+v"(o[3])); } while (0)
            int k0, s0, it0, k1, s1, it1;
            CLAIM(0, ids1, 6, LA, NI1, false, k0, s0, it0);
            unsigned tkt = 0;
            if (it0 < NI1) { P1_LOAD(it0, s0); if (s0 == 0 && lane == 0) TICKET_DRAW(tkt, tk); }
            call = (unsigned)rfl((int)prev_v) + 1u;
            const unsigned tbase = (call - 1u) * (unsigned)NI1 - (unsigned)(LA * NP);
            P1_CVT(it0, s0);
            if (s0 == 0 && it0 < NI1) { VWAIT0; asm volatile("s_waitcnt vmcnt(0)" : "+v"(tkt) :: "memory"); if (lane == 0) *(volatile LAS unsigned*)(ids1 + k0 + LA) = min(tkt - tbase, (unsigned)NI1) + 1u; }
            while (it0 < NI1) {
                CLAIM(0, ids1, 6, LA, NI1, false, k1, s1, it1);
                if (it1 < NI1) { P1_LOAD(it1, s1); if (s1 == 0 && lane == 0) TICKET_DRAW(tkt, tk); }
                const unsigned e = (unsigned)it0 >> 3, g = (unsigned)it0 & 7u;
                if (h1 & 0x80u) {
                    const unsigned base = unit_base(rank, e, g, s0), st1 = s0 < 4 ? (unsigned)XS_T : 1024u, st2 = s0 < 4 ? 1024u : 2048u;
#pragma unroll
                    for (int b = 0; b < 4; ++b) BSTORE(o[b], rs, (unsigned)lane * 16u, base + (b & 1) * st1 + (b >> 1) * st2, AUX_SYS);
                } else {
                    const unsigned base = (unsigned)IB_OFF + (unsigned)((rank * NI1 + it0) * 6 + s0) * 3072u;
#pragma unroll
                    for (int b = 0; b < 3; ++b) BSTORE(o[b], rs, (unsigned)lane * 16u, base + b * 1024u, AUX_SYS);
                }
                if (lane == 0) { LAS unsigned char* hp = hdrs + k0 * 64 + (s0 >> 1) * 16; *(LAS unsigned*)(hp + 4 + (s0 & 1) * 4) = h0; hp[12 + (s0 & 1)] = (unsigned char)h1; }
                VWAIT0; asm volatile("s_waitcnt vmcnt(0) lgkmcnt(0)" : "+v"(tkt) :: "memory");
                unsigned dn = 0;
                if (lane == 0) {
                    if (s1 == 0 && it1 < NI1) *(volatile LAS unsigned*)(ids1 + k1 + LA) = min(tkt - tbase, (unsigned)NI1) + 1u;
                    dn = __hip_atomic_fetch_add(done1 + k0, 1u, __ATOMIC_RELAXED, __HIP_MEMORY_SCOPE_WORKGROUP);
                }
                if (rfl((int)dn) == 5 && lane < 3) {
                    asm volatile("" ::: "memory"); STAMP2(0, dst, it0);
                    if (dst == rank || it0 >= E_RAW * 8) st_sys((unsigned*)(dws + INF_OFF) + (rank * 24 + (lane < 2 ? 2 * g + lane : 16 + g)) * 32 + e, call);
                    else { u32x4 hv = *(const LAS u32x4*)(hdrs + k0 * 64 + lane * 16); hv.x = call;
                           BSTORE(hv, mk_rsrc(dws + HDR_OFF, (unsigned)(NR * NI1 * 64)), (unsigned)lane * 16u, (unsigned)(rank * NI1 + it0) * 64u, AUX_SYS); }
                }
                if (it1 < NI1) P1_CVT(it1, s1);
                k0 = k1; s0 = s1; it0 = it1;
            }
        }
        STAMP(1);
        if (dst != rank) {
            unsigned* const tk = (unsigned*)(myws + TK_OFF) + (8 + dst) * 32;
            const unsigned tbase = (call - 1u) * (unsigned)NI2 - (unsigned)(LA2 * NP);
            u64 rm0 = 0, rm1 = 0, rs0 = 0, rs1 = 0; bool dead = false;
            const __amdgpu_buffer_rsrc_t rr = mk_rsrc(myws + RDY_OFF, 2048u);
            u32x4 cur[8], nxt[8], rf0 = {0u, 0u, 0u, 0u}, rf1 = rf0; unsigned len0 = 0, len1 = 0, coff = 0, clen = 0, cpos = 0, cnk = 0, noff = 0, nlen = 0, npos = 0, nnk = 0;
            unsigned mlist = 0x88888888u;
            { const int j = (dst - rank - 1) & 7; int n = 0;
              for (int w = 0; w < 8; ++w) { bool direct = false; for (int i = 0; i < ND; ++i) direct |= (w < 7 && (w + 3 * i) % 7 == j); if (!direct) { mlist = (mlist & ~(15u << (4 * n))) | ((unsigned)w << (4 * n)); ++n; } } }
#define P2_TILE(it, sub) ((it) < NMI ? 4 * (it) + ((sub) >> 1) : (it) - NMI)
#define P2_CLAIM(k, sub, it) CLAIM(1, ids2, 8, LA2, NI2, (it < NMI + TAPER && (sub & 1)), k, sub, it)
#define P2_KNOWN(it, sub) (((((it) < NMI ? (P2_TILE(it, sub) < 64 ? rm0 : rm1) : (P2_TILE(it, sub) < 64 ? rs0 : rs1)) >> (P2_TILE(it, sub) & 63)) & 1ull) != 0ull)
#define P2_RDY_LOAD() do { rf0 = __builtin_amdgcn_raw_buffer_load_b128(rr, (unsigned)lane * 16u, 0, AUX_DEV); rf1 = __builtin_amdgcn_raw_buffer_load_b128(rr, (unsigned)lane * 16u + 1024u, 0, AUX_DEV); } while (0)
#define P2_RDY_USE() do { rm0 = __ballot(rf0.x == call); rm1 = __ballot(rf1.x == call); rs0 = __ballot(rf0.w == call); rs1 = __ballot(rf1.w == call); len0 = rf0.z; len1 = rf1.z; } while (0)
#define P2_WAIT(it, sub) do { const u64 _t0 = __builtin_amdgcn_s_memrealtime(); unsigned _n = 0; \
            while (!dead && !P2_KNOWN(it, sub)) { \
                asm volatile("" ::: "memory"); P2_RDY_LOAD(); P2_RDY_USE(); \
                if (P2_KNOWN(it, sub)) break; \
                if ((++_n & 31u) == 0u && give_up(tmo, call, _t0)) { dead = true; if (lane == 0) *(volatile LAS unsigned*)(ctl + 2) = 1u; } \
                __builtin_amdgcn_s_sleep(P2_SLEEP); } \
            asm volatile("" ::: "memory"); } while (0)
#define P2_LOAD(it, sub) do { const int _t = P2_TILE(it, sub); \
            if ((it) < NMI) { noff = (unsigned)MK_OFF + (unsigned)(rank * 128 + _t) * 8192u; nlen = 8192u; npos = mlist; nnk = 8 - ND; } \
            else { const unsigned _l = min((unsigned)__builtin_amdgcn_readlane((int)(_t < 64 ? len0 : len1), _t & 63), ST_SLOT - 128u); \
                   const unsigned _q = _t < TAPER ? ((_l + 4095u) >> 12) << 9 : ((_l + 2047u) >> 11) << 8, _z = _t < TAPER ? 2u * _q : _q; \
                   const unsigned _o = min(_q * (unsigned)(sub), _l); nnk = (_z + 1023u) >> 10; \
                   noff = (unsigned)ST_OFF + (unsigned)(rank * 128 + _t) * ST_SLOT + _o; nlen = min(_l - _o, _z); npos = 0x76543210u; } \
            const __amdgpu_buffer_rsrc_t _r = mk_rsrc(myws + noff, nlen); \
            _Pragma("unroll") for (int k = 0; k < 8; ++k) if (k < 4 || nnk > 4u) nxt[k] = __builtin_amdgcn_raw_buffer_load_b128(_r, (unsigned)lane * 16u + ((npos >> (4 * k)) & 15u) * 1024u, 0, AUX_DEV); \
            asm volatile("" ::: "memory"); } while (0)
#define P2_ROT() do { coff = (unsigned)rfl((int)noff); cpos = (unsigned)rfl((int)npos); clen = (unsigned)rfl((int)nlen); cnk = (unsigned)rfl((int)nnk); \
            _Pragma("unroll") for (int k = 0; k < 8; ++k) cur[k] = nxt[k]; } while (0)
#define P2_STORE() do { const __amdgpu_buffer_rsrc_t _r = mk_rsrc(dws + coff, clen); \
            _Pragma("unroll") for (int k = 0; k < 8; ++k) if (k < 4 || cnk > 4u) { if (k > PW) WAIT_VX(PW); BSTORE(cur[k], _r, (unsigned)lane * 16u + ((cpos >> (4 * k)) & 15u) * 1024u, 0, AUX_SYS); } \
            if (cnk > 8u) { const __amdgpu_buffer_rsrc_t _s = mk_rsrc(myws + coff, clen); \
                _Pragma("unroll") for (int k = 0; k < 7; ++k) cur[k] = __builtin_amdgcn_raw_buffer_load_b128(_s, (unsigned)lane * 16u + (8 + k) * 1024u, 0, AUX_DEV); \
                _Pragma("unroll") for (int k = 0; k < 7; ++k) BSTORE(cur[k], _r, (unsigned)lane * 16u + (8 + k) * 1024u, 0, AUX_SYS); } } while (0)
#pragma unroll
            for (int k = 0; k < 8; ++k) { nxt[k] = (u32x4){0u, 0u, 0u, 0u}; cur[k] = nxt[k]; }
            int k0, s0, it0, k1, s1, it1;
            P2_CLAIM(k0, s0, it0);
            unsigned tkt = 0;
            if (s0 == 0 && it0 < NI2) { if (lane == 0) TICKET_DRAW(tkt, tk); WAIT_V(0); asm volatile("" : "+v"(tkt) :: "memory"); if (lane == 0) *(volatile LAS unsigned*)(ids2 + k0 + LA2) = min(tkt - tbase, (unsigned)NI2) + 1u; }
            if (it0 < NI2) { P2_WAIT(it0, s0); P2_LOAD(it0, s0); }
            STAMP(2);
            VWAIT0; P2_ROT();
            while (it0 < NI2) {
                P2_CLAIM(k1, s1, it1);
                const bool early = rfl((it1 < NI2 && P2_KNOWN(it1, s1)) ? 1 : 0) != 0;
                if (early) P2_LOAD(it1, s1);
                const bool refresh = rfl((rs0 & rs1) != ~0ull ? 1 : 0) != 0;
                if (refresh) P2_RDY_LOAD();
                if (s1 == 0 && it1 < NI2 && lane == 0) TICKET_DRAW(tkt, tk);
                P2_STORE();
                VWAIT0; asm volatile("s_waitcnt vmcnt(0)" : "+v"(tkt), "+v"(rf0), "+v"(rf1) :: "memory");
                if (lane == 0) {
                    if (s1 == 0 && it1 < NI2) *(volatile LAS unsigned*)(ids2 + k1 + LA2) = min(tkt - tbase, (unsigned)NI2) + 1u;
                    if (__hip_atomic_fetch_add(done2 + k0, 1u, __ATOMIC_RELAXED, __HIP_MEMORY_SCOPE_WORKGROUP) == (it0 < NMI + TAPER ? 3u : 7u) && *(volatile LAS unsigned*)(ctl + 2) == 0u)
                        st_sys((unsigned*)(dws + OUF_OFF) + rank * 256 + it0, call);
                }
                if (refresh) P2_RDY_USE();
                if (!early && it1 < NI2) { P2_WAIT(it1, s1); P2_LOAD(it1, s1); VWAIT0; }
                P2_ROT();
                k0 = k1; s0 = s1; it0 = it1;
            }
            STAMP(3);
        }
    } else {
        const unsigned call = (unsigned)rfl((int)prev_v) + 1u;
        const __amdgpu_buffer_rsrc_t ro = mk_rsrc(P.out, (unsigned)((size_t)GM * GN));
        if (slot < 16) {
        const __amdgpu_buffer_rsrc_t rs = mk_rsrc(myws, (unsigned)CTL_OFF);
        const __amdgpu_buffer_rsrc_t rh = mk_rsrc(myws + HDR_OFF, (unsigned)(NR * NI1 * 64));
        constexpr int NEW = (16 - NP) * 64;
        for (int J = wid * (NEW / 8) + (bid - NP * 8); J < E_RAW * 8 * 21; J += NEW) {
            const int it = J / 21, r = J - it * 21, sp = r / 3, j = r - sp * 3, src = (rank + 1 + sp) & 7;
            const unsigned e = (unsigned)it >> 3, g = (unsigned)it & 7u;
            u32x4 hv; bool dead = false;
            { const u64 t0 = __builtin_amdgcn_s_memrealtime(); unsigned n = 0;
              for (;;) {
                  asm volatile("" ::: "memory");
                  hv = __builtin_amdgcn_raw_buffer_load_b128(rh, 0u, (unsigned)((src * NI1 + it) * 64 + j * 16), AUX_SYS);
                  if ((unsigned)rfl((int)hv.x) == call) break;
                  if ((++n & 31u) == 0u && give_up(tmo, call, t0)) { dead = true; break; }
                  __builtin_amdgcn_s_sleep(EXP_SLEEP);
              }
              asm volatile("" ::: "memory"); }
            if (dead) break;
            STAMP2(1, src, it * 3 + j);
            const unsigned ibase = (unsigned)IB_OFF + (unsigned)((src * NI1 + it) * 6 + 2 * j) * 3072u;
            u32x4 p[6];
#pragma unroll
            for (int k = 0; k < 6; ++k) p[k] = __builtin_amdgcn_raw_buffer_load_b128(rs, (unsigned)lane * 16u, ibase + k * 1024u, AUX_DEV);
            VWAIT0;
            const unsigned tops = (unsigned)rfl((int)hv.w);
            const unsigned st1 = j < 2 ? (unsigned)XS_T : 1024u, st2 = j < 2 ? 1024u : 2048u;
#pragma unroll
            for (int u = 0; u < 2; ++u) {
                const unsigned top = (tops >> (8 * u)) & 0xffu, exc = (unsigned)rfl((int)(u ? hv.z : hv.y));
                if (top & 0x80u) continue;
                const unsigned t4 = top * 0x01010101u + 0x80808080u;
                const unsigned T0 = (t4 - 0x03020100u) & 0x7f7f7f7fu, T1 = (t4 - 0x00060504u) & 0x007f7f7fu;
                const unsigned base = unit_base(src, e, g, 2 * j + u);
#pragma unroll
                for (int b = 0; b < 4; ++b) { u32x4 d; const unsigned T1b = T1 | ((exc >> (8 * b)) << 24);
#pragma unroll
                    for (int h = 0; h < 2; ++h) { const unsigned c = p[3 * u + 2][b] >> (4 * h), lo = p[3 * u + (b >> 1)][(b & 1) * 2 + h];
                        const unsigned hi = __builtin_amdgcn_perm(T1b, T0, c & 0x07070707u) | ((c << 4) & 0x80808080u);
                        d[2 * h] = __builtin_amdgcn_perm(hi, lo, 0x05010400u); d[2 * h + 1] = __builtin_amdgcn_perm(hi, lo, 0x07030602u); }
                    BSTORE(d, rs, (unsigned)lane * 16u, base + (b & 1) * st1 + (b >> 1) * st2, AUX_DEV); }
            }
            WAIT_V(0);
            if (lane == 0) st_dev((unsigned*)(myws + INF_OFF) + (src * 24 + (j < 2 ? 2 * g + j : 16 + g)) * 32 + e, call);
            STAMP2(2, src, it * 3 + j);
        }
        STAMP(1);
        } else {
        const int wr = wid >> 2, wc = wid & 3, fr = lane & 15, fq = lane >> 4;
        const int ts = slot - 16;
        const int pm = (xcd >> 1) * 4 + (ts & 3), pn = (xcd & 1) * 4 + (ts >> 2);
        const int tile = pm * 8 + pn;
        const char* cA = myws + XS_OFF + ((size_t)wid * GM + pm * 256) * 16 + lane * 16;
        const char* cB = myws + WS_OFF + ((size_t)wid * NS + pn * 256) * 16 + lane * 16;
        const unsigned* fp = (const unsigned*)(myws + INF_OFF) + (wid * 24 + (lane < 32 ? pm : 16 + pn)) * 32 + (lane & 31);
        const unsigned ldsw = (unsigned)wid * 2048u;
        const int aoff = fq * 2048 + (wr * 64 + fr) * 16, boff = fq * 2048 + (wc * 32 + fr) * 16;
        unsigned fsamp = 0;
        unsigned used = 0; int ec, en;
        unsigned known = 0;
        bool kdead = false;
#define SA(b, h) (((b) * 2 + (h)) * HTB)
#define SB(b, h) ((4 + (b) * 2 + (h)) * HTB)
#define STAGE(bufoff, g) do { _Pragma("unroll") for (int _i = 0; _i < 2; ++_i) \
        __builtin_amdgcn_global_load_lds((const unsigned*)((g) + _i * 1024), (LAS unsigned*)(lds + (bufoff) + ldsw + _i * 1024), 16, 0, AUX_DEV); } while (0)
#define LDA(dst, b, h) do { _Pragma("unroll") for (int m = 0; m < 4; ++m) _Pragma("unroll") for (int k = 0; k < 2; ++k) dst[m][k] = *(const LAS bf16x8*)(lds + SA(b, h) + aoff + m * 256 + k * 8192); } while (0)
#define LDB(dst, b, h) do { _Pragma("unroll") for (int n = 0; n < 2; ++n) _Pragma("unroll") for (int k = 0; k < 2; ++k) dst[n][k] = *(const LAS bf16x8*)(lds + SB(b, h) + boff + n * 256 + k * 8192); } while (0)
#define MMA(ai, bj, At, Bt) do { __builtin_amdgcn_s_setprio(1); _Pragma("unroll") for (int m = 0; m < 4; ++m) _Pragma("unroll") for (int n = 0; n < 2; ++n) _Pragma("unroll") for (int k = 0; k < 2; ++k) \
        acc[ai][bj][m][n] = __builtin_amdgcn_mfma_f32_16x16x32_bf16(Bt[n][k], At[m][k], acc[ai][bj][m][n], 0, 0, 0); __builtin_amdgcn_s_setprio(0); } while (0)
#define NEXT_EPOCH(en) do { if ((known & ~used) == 0u) { const u64 _t0 = __builtin_amdgcn_s_memrealtime(); unsigned _n = 0; \
            for (;;) { const u64 _b = __ballot(ld_sys(fp) == call); known |= (unsigned)_b & (unsigned)(_b >> 32); if ((known & ~used) != 0u) break; \
                if ((++_n & 63u) == 0u && give_up(tmo, call, _t0)) { known = 0xffffffffu; kdead = true; break; } \
                __builtin_amdgcn_s_sleep(POLL_SLEEP); } \
            asm volatile("" ::: "memory"); } \
            en = __builtin_ctz(known & ~used); used |= 1u << en; } while (0)
        f32x4 acc[2][2][4][2];
#pragma unroll
        for (int a = 0; a < 2; ++a)
#pragma unroll
            for (int b = 0; b < 2; ++b)
#pragma unroll
                for (int m = 0; m < 4; ++m)
#pragma unroll
                    for (int n = 0; n < 2; ++n) acc[a][b][m][n] = (f32x4){0.f, 0.f, 0.f, 0.f};
        bf16x8 At[4][2], B0[2][2], B1[2][2];
        NEXT_EPOCH(en);
        STAMP(1);
        { const char* a0 = cA + (size_t)(2 * en) * XS_T; const char* b0 = cB + (size_t)(2 * en) * WS_T;
        STAGE(SB(0, 0), b0); STAGE(SA(0, 0), a0); STAGE(SB(0, 1), b0 + 2048); STAGE(SA(0, 1), a0 + 2048);
        if (wr == 1) RBAR;
        WAIT_V(4); RBAR;
        STAGE(SB(1, 0), b0 + WS_T); STAGE(SA(1, 0), a0 + XS_T); STAGE(SB(1, 1), b0 + WS_T + 2048);
        WAIT_V(6); RBAR; }
        for (int e = 0; e < NEP; ++e) {
            ec = en;
            { asm volatile("" : "+v"(fsamp)); const u64 _b = __ballot(fsamp == call); known |= (unsigned)_b & (unsigned)(_b >> 32); }
            if (e != NEP - 1) NEXT_EPOCH(en);
            const char* a1 = cA + (size_t)(2 * ec + 1) * XS_T;
            const char* a2 = cA + (size_t)(2 * en) * XS_T; const char* b2 = cB + (size_t)(2 * en) * WS_T;
            const char* a3 = a2 + XS_T; const char* b3 = b2 + WS_T;
            if (e == 15) STAMP(7); if (e == 27) STAMP(8); if (e == 29) STAMP(9); if (e == 30) STAMP(10);
            LDB(B0, 0, 0); SCHED; LDA(At, 0, 0); STAGE(SA(1, 1), a1 + 2048);
            WAIT_L(8); RBAR; WAIT_L(0); MMA(0, 0, At, B0); RBAR; SCHED;
            LDB(B1, 0, 1); STAGE(SB(0, 0), b2);
            RBAR; WAIT_L(0); MMA(0, 1, At, B1); RBAR;
            LDA(At, 0, 1); STAGE(SA(0, 0), a2);
            RBAR; WAIT_L(0); MMA(1, 0, At, B0); RBAR; SCHED;
            STAGE(SB(0, 1), b2 + 2048);
            WAIT_V(6); RBAR; MMA(1, 1, At, B1); RBAR;
            LDB(B0, 1, 0); SCHED; LDA(At, 1, 0);
            asm volatile("global_load_dword %0, %1, off sc0 sc1" : "=v"(fsamp) : "v"(fp) : "memory");
            STAGE(SA(0, 1), a2 + 2048);
            WAIT_L(8); RBAR; WAIT_L(0); MMA(0, 0, At, B0); RBAR; SCHED;
            LDB(B1, 1, 1); STAGE(SB(1, 0), b3);
            RBAR; WAIT_L(0); MMA(0, 1, At, B1); RBAR;
            LDA(At, 1, 1); STAGE(SA(1, 0), a3);
            RBAR; WAIT_L(0); MMA(1, 0, At, B0); RBAR; SCHED;
            STAGE(SB(1, 1), b3 + 2048);
            WAIT_V(6); RBAR; MMA(1, 1, At, B1); RBAR;
        }
        if (wr == 0) RBAR;
        STAMP(2);
        u32x4 mk;
#pragma unroll
        for (int g2 = 0; g2 < 4; ++g2) { unsigned mm = 0;
#pragma unroll
            for (int e = 31; e >= 0; --e) { const int g = g2 * 2 + (e >> 4), d = (e >> 2) & 3; mm = __builtin_amdgcn_alignbit(mm, __float_as_uint(acc[g >> 2][d >> 1][g & 3][d & 1][e & 3]), 31); }
            mk[g2] = ~mm; }
        const unsigned reg_id = (unsigned)((rank * 128 + tile) * 8 + wid);
        BSTORE(mk, mk_rsrc(myws + MK_OFF, (unsigned)MK_BYTES), (unsigned)lane * 16u, reg_id * 1024u, AUX_DEV);
#pragma unroll
        for (int i = 0; i < ND; ++i) { const int j = (wid + 3 * i) % 7;
            BSTORE(mk, mk_rsrc(P.ws[(rank + 1 + j) & 7] + MK_OFF, wid < 7 ? (unsigned)MK_BYTES : 0u), (unsigned)lane * 16u, reg_id * 1024u, AUX_SYS); }
        SCHED;
        LAS float* red = (LAS float*)(lds + 8 * HTB);
        float gmx[16];
#pragma unroll
        for (int i = 0; i < 16; ++i) { const f32x4 v0 = acc[i >> 3][i & 1][(i >> 1) & 3][0], v1 = acc[i >> 3][i & 1][(i >> 1) & 3][1];
            gmx[i] = max3f(max3f(v0[0], v0[1], 0.f), max3f(v0[2], v0[3], v1[0]), max3f(v1[1], v1[2], v1[3])); }
        float mx = max3f(gmx[0], gmx[1], gmx[15]);
#pragma unroll
        for (int i = 2; i < 15; i += 2) mx = max3f(mx, gmx[i], gmx[i + 1]);
        mx = __uint_as_float(wave_umax(__float_as_uint(mx) & 0x7fffffffu));
        if (lane == 63) red[wid] = kdead ? -1.0f : mx;
        __builtin_amdgcn_s_waitcnt(0x0F70 | ND);
        WAIT_L(0); BAR;
        {
            float m8 = red[0], n8 = red[0];
#pragma unroll
            for (int i = 1; i < 8; ++i) { m8 = fmaxf(m8, red[i]); n8 = fminf(n8, red[i]); }
            if (lane == 0 && n8 >= 0.f) {
                if (wid == 0) st_dev((unsigned*)(myws + RDY_OFF) + tile * 4, call);
                st_sys64((u64*)(P.ws[wid] + AMX_OFF) + rank * 128 + tile, ((u64)call << 32) | (u64)__float_as_uint(m8));
            }
        }
        LAS unsigned char* sbuf = lds + wid * 8192;
#pragma unroll
        for (int i = 0; i < 8; ++i) *(LAS u32x4*)(sbuf + i * 1024 + lane * 16) = (u32x4){0u, 0u, 0u, 0u};
        {
            const u64* gp = (const u64*)(myws + AMX_OFF) + wid * 128 + lane;
            const u64 t0 = __builtin_amdgcn_s_memrealtime(); unsigned n = 0; float gm; bool dead = false;
            for (;;) {
                const u64 g0 = ld_sys64(gp), g1 = ld_sys64(gp + 64);
                gm = fmaxf(__uint_as_float((unsigned)g0), __uint_as_float((unsigned)g1));
                if (__all((unsigned)(g0 >> 32) == call && (unsigned)(g1 >> 32) == call)) break;
                if ((++n & 63u) == 0u && give_up(tmo, call, t0)) { dead = true; break; }
                __builtin_amdgcn_s_sleep(1);
            }
            gm = __uint_as_float(wave_umax(__float_as_uint(gm)));
            if (lane == 63) red[8 + wid] = dead ? -1.0f : gm;
        }
        WAIT_L(0); BAR;
        float amax = red[8], amin = red[8];
#pragma unroll
        for (int i = 1; i < 8; ++i) { amax = fmaxf(amax, red[8 + i]); amin = fminf(amin, red[8 + i]); }
        const bool bad = rfl(amin < 0.f ? 1 : 0) != 0;
        const float rinv = amax > 0.f ? 1.0f / amax : 0.f;
        STAMP(3);
        const float c127 = __uint_as_float(127u);
        unsigned Wg[16], clmin = 31u, hdr = 0u, lbits = 0u;
#pragma unroll
        for (int i = 0; i < 16; ++i) { Wg[i] = (unsigned)__builtin_clz(2u * __float_as_uint(quant(gmx[i], rinv, c127)) + 1u); clmin = min(clmin, Wg[i]); }
#pragma unroll
        for (int i = 0; i < 16; ++i) { const unsigned dl = min(Wg[i] - clmin, 3u); hdr |= dl << (2 * i); Wg[i] = 31u - clmin - dl;
            lbits += (unsigned)__builtin_popcount((mk[i >> 2] >> (8 * (i & 3))) & 0xffu) * Wg[i]; }
        const unsigned incl = wave_scan(lbits);
        LAS unsigned* rbits = (LAS unsigned*)(red + 16);
        if (lane == 63) rbits[wid] = RH_BITS + incl;
        WAIT_L(0); BAR;
        unsigned rbase = HDR_BITS, total_bits = HDR_BITS;
#pragma unroll
        for (int i = 0; i < 8; ++i) { const unsigned t = rbits[i]; total_bits += t; if (i < wid) rbase += t; }
        const unsigned nbytes = (unsigned)rfl((int)((total_bits + 127u) / 128u * 16u));
        sbuf = lds;
        unsigned Pb = rbase + RH_BITS + incl - lbits;
        if (lane == 0) *(LAS unsigned*)(sbuf + wid * 4) = rbase;
        lds_or_bits(sbuf, rbase + 32u * lane, hdr);
        lds_or_bits(sbuf, rbase + 2048u + 3u * lane, 31u - clmin);
#pragma unroll
        for (int g = 0; g < 8; ++g)
#pragma unroll
            for (int d = 0; d < 4; ++d) { f32x4& v = acc[g >> 2][d >> 1][g & 3][d & 1]; const unsigned mw = mk[g >> 1], W = Wg[g * 2 + (d >> 1)]; const int b0 = (g & 1) * 16 + d * 4;
#pragma unroll
                for (int j = 0; j < 4; ++j) v[j] = quant(v[j], rinv, c127);
                unsigned ch = __float_as_uint(v[0]); unsigned p = ((mw >> b0) & 1u) * W;
                ch |= __float_as_uint(v[1]) << p; p += ((mw >> (b0 + 1)) & 1u) * W;
                ch |= __float_as_uint(v[2]) << p; p += ((mw >> (b0 + 2)) & 1u) * W;
                ch |= __float_as_uint(v[3]) << p; p += ((mw >> (b0 + 3)) & 1u) * W;
                lds_or_bits(sbuf, Pb, ch);
                Pb += p; }
        WAIT_L(0); BAR;
        {
            const __amdgpu_buffer_rsrc_t rd = mk_rsrc(myws + ST_OFF + (size_t)(rank * 128 + tile) * ST_SLOT, nbytes);
#pragma unroll
            for (int k = 0; k < 8; ++k) if (k < 4 || nbytes > 32768u) { const u32x4 sv = *(const LAS u32x4*)(sbuf + k * 8192 + tid * 16); BSTORE(sv, rd, (unsigned)tid * 16u + k * 8192u, 0, AUX_DEV); }
            WAIT_V(0); WAIT_L(0); BAR;
            if (tid == 0 && !bad) st_dev64((u64*)(myws + RDY_OFF) + tile * 2 + 1, ((u64)call << 32) | nbytes);
        }
        STAMP(4);
        SCHED;
        {
            const unsigned voff = (unsigned)((pm * 256 + wr * 64 + fr) * GN + rank * NS + pn * 256 + wc * 64 + fq * 16);
#pragma unroll
            for (int g = 0; g < 8; ++g) { u32x4 qb;
#pragma unroll
                for (int d = 0; d < 4; ++d) { const f32x4 v = acc[g >> 2][d >> 1][g & 3][d & 1];
                    const unsigned w = __float_as_uint(v[0]) | (__float_as_uint(v[1]) << 8) | (__float_as_uint(v[2]) << 16) | (__float_as_uint(v[3]) << 24); qb[d] = bad ? 0x80808080u : w; }
                BSTORE(qb, ro, voff, (unsigned)(((g >> 2) * 128 + (g & 3) * 16) * GN), OUT_AUX); }
        }
        }
        {
        const int fr = lane & 15, fq = lane >> 4;
        const __amdgpu_buffer_rsrc_t rm = mk_rsrc(myws + MK_OFF, (unsigned)MK_BYTES);
        constexpr int NDEC = (32 - NP) * 8;
        int par = 0;
        for (int J = 255 - bid; J < 128 * 14; J += NDEC, par ^= 1) {
            const int tile = J / 14, r = J - tile * 14, src = (rank + 1 + (r >> 1)) & 7, reg = (r & 1) * 4 + (wid >> 1), hf = wid & 1;
            const int pm = tile >> 3, pn = tile & 7;
            LAS unsigned char* sbuf = lds + par * 65536;
            bool bad = false;
            {
                const unsigned* fp = (const unsigned*)(myws + OUF_OFF) + src * 256 + ((lane & 1) ? NMI + tile : (tile >> 2));
                const u64 t0 = __builtin_amdgcn_s_memrealtime(); unsigned n = 0;
                while (!__all(ld_sys(fp) == call)) {
                    if ((++n & 15u) == 0u && give_up(tmo, call, t0)) { bad = true; break; }
                    __builtin_amdgcn_s_sleep(DEC_SLEEP);
                }
                asm volatile("" ::: "memory");
            }
            if (J < NDEC) STAMP(5);
            if (J + NDEC >= 128 * 14) STAMP(11);
            const char* sp = myws + ST_OFF + (size_t)(src * 128 + tile) * ST_SLOT + tid * 16;
#pragma unroll
            for (int k = 0; k < 4; ++k) __builtin_amdgcn_global_load_lds((const unsigned*)(sp + k * 8192), (LAS unsigned*)(sbuf + k * 8192 + wid * 1024), 16, 0, AUX_DEV);
            const u32x4 mk = __builtin_amdgcn_raw_buffer_load_b128(rm, (unsigned)lane * 16u, (unsigned)((src * 128 + tile) * 8 + reg) * 1024u, AUX_DEV);
            WAIT_V(0); BAR;
            if ((unsigned)rfl((int)*(const LAS unsigned*)(sbuf + 28)) + RMAX_BITS + 64u > 32768u * 8u) {
#pragma unroll
                for (int k = 4; k < 8; ++k) if (k < 7 || wid < 3) __builtin_amdgcn_global_load_lds((const unsigned*)(sp + k * 8192), (LAS unsigned*)(sbuf + k * 8192 + wid * 1024), 16, 0, AUX_DEV);
                WAIT_V(0); BAR;
            }
            const unsigned rbase = min(*(const LAS unsigned*)(sbuf + reg * 4), HDR_BITS + 7u * RMAX_BITS);
            const unsigned hdr = lds_get_bits(sbuf, rbase + 32u * lane), Wt = lds_get_bits(sbuf, rbase + 2048u + 3u * lane) & 7u;
            unsigned blo = 0u, bhi = 0u;
#pragma unroll
            for (int i = 0; i < 8; ++i) { blo += (unsigned)__builtin_popcount((mk[i >> 2] >> (8 * (i & 3))) & 0xffu) * ((Wt - ((hdr >> (2 * i)) & 3u)) & 7u);
                                          bhi += (unsigned)__builtin_popcount((mk[2 + (i >> 2)] >> (8 * (i & 3))) & 0xffu) * ((Wt - ((hdr >> (16 + 2 * i)) & 3u)) & 7u); }
            unsigned Pb = rbase + RH_BITS + wave_scan(blo + bhi) - (hf ? bhi : blo + bhi);
            const unsigned hh = hf ? hdr >> 16 : hdr;
            const unsigned m0 = hf ? mk.z : mk.x, m1 = hf ? mk.w : mk.y;
            const unsigned voff = (unsigned)((pm * 256 + hf * 128 + (reg >> 2) * 64 + fr) * GN + src * NS + pn * 256 + (reg & 3) * 64 + fq * 16);
#pragma unroll
            for (int g = 0; g < 4; ++g) { u32x4 qb;
#pragma unroll
                for (int d = 0; d < 4; ++d) { const unsigned mw = (g >> 1) ? m1 : m0, W = (Wt - ((hh >> (4 * g + 2 * (d >> 1))) & 3u)) & 7u; const int b0 = (g & 1) * 16 + d * 4;
                    const unsigned win = lds_get_bits(sbuf, Pb);
                    const unsigned w0 = ((mw >> b0) & 1u) * W, w1 = ((mw >> (b0 + 1)) & 1u) * W, w2 = ((mw >> (b0 + 2)) & 1u) * W, w3 = ((mw >> (b0 + 3)) & 1u) * W;
                    const unsigned p1 = w0, p2 = p1 + w1, p3 = p2 + w2;
                    const unsigned o = __builtin_amdgcn_ubfe(win, 0u, w0) | (__builtin_amdgcn_ubfe(win, p1, w1) << 8) | (__builtin_amdgcn_ubfe(win, p2, w2) << 16) | (__builtin_amdgcn_ubfe(win, p3, w3) << 24);
                    qb[d] = bad ? 0x80808080u : o; Pb += p3 + w3; }
                BSTORE(qb, ro, voff, (unsigned)(g * 16 * GN), OUT_AUX);
            }
            WAIT_L(0);
        }
        STAMP(12);
        }
    }
    __syncthreads();
    STAMP(6);
    if (tid == 0) st_dev(cntp, prev_v + 1u);
}
}

extern "C" void kernel_launch(int rank, int nranks, ncclComm_t comm, hipStream_t stream,
                              void* const* d_in, const int* in_sizes, int n_in,
                              void* d_out, int out_size, const int* in_dtypes, int out_dtype,
                              void* const* d_out_all, void* const* d_ws_all, size_t ws_bytes) {
    if (nranks != NR || ws_bytes < WS_NEED || (unsigned)rank >= (unsigned)NR) { fprintf(stderr, "unexpected configuration\n"); return; }
    HIP_CHECK(hipFuncSetAttribute((const void*)fused_kernel, hipFuncAttributeMaxDynamicSharedMemorySize, LDS_BYTES));
    Params P;
    P.x = (const float*)d_in[0]; P.w = (const float*)d_in[1]; P.rank = rank; P.out = (signed char*)d_out;
    for (int p = 0; p < NR; ++p) P.ws[p] = (char*)d_ws_all[p];
    hipLaunchKernelGGL(fused_kernel, dim3(256), dim3(512), LDS_BYTES, stream, P);
    HIP_CHECK(hipGetLastError());
}
```

```cpp
#include <hip/hip_runtime.h>
#include <rccl/rccl.h>
#include <cstdio>

#define HIP_CHECK(expr) do { hipError_t _e = (expr); if (_e != hipSuccess) { \
    fprintf(stderr, "HIP error %s at %s:%d\n", hipGetErrorString(_e), __FILE__, __LINE__); \
    return; } } while (0)

#define LAG0 12, 12, 12, 12, 14, 14, 1, 1, 1, 1, 2, 2, 2, 2, 28, 28, 0, 0, 0, 0, 0, 0, 0, 0
#define LAG1 0, 2, 4, 7, 10, 13, 8, 10, 13, 16, 19, 21, 22, 22, 42, 42, 0, 0, 0, 0, 0, 0, 0, 0
namespace {
#define LAS __attribute__((address_space(3)))
typedef short bf16x8 __attribute__((ext_vector_type(8)));
typedef float f32x4 __attribute__((ext_vector_type(4)));
typedef unsigned u32x4 __attribute__((ext_vector_type(4)));
typedef unsigned u32x2 __attribute__((ext_vector_type(2)));
typedef unsigned long long u64;

constexpr int NR = 8;
constexpr int GM = 4096, GN = 16384;
constexpr int KS = 512;
constexpr int NS = GN / NR;
constexpr int NT = 64;
constexpr int NEP = NT / 2;
#define PST 10240u
#define SST 16384u
#define PUSHE 4
constexpr unsigned J_MK = NEP * 24 - (PUSHE - 1) * 8, J_ST = J_MK + 128, NJ = J_ST + 512;
constexpr size_t XS_T = (size_t)NR * GM * 16;
constexpr size_t WS_T = (size_t)NR * NS * 16;
constexpr size_t ST_OFF = 0, ST_BYTES = (size_t)512 * 16384;
constexpr size_t MK_OFF = ST_OFF + ST_BYTES;
constexpr size_t MK_BYTES = (size_t)NR * 128 * 8192;
constexpr size_t XS_OFF = MK_OFF + MK_BYTES, XS_BYTES = XS_T * NT;
constexpr size_t WS_OFF = XS_OFF + XS_BYTES, WS_BYTES = WS_T * NT;
constexpr size_t OB_OFF = WS_OFF + WS_BYTES;
constexpr size_t OB_BYTES = (size_t)26 << 20;
constexpr size_t CTL_OFF = OB_OFF + OB_BYTES;
constexpr size_t HDR_OFF = CTL_OFF;
constexpr size_t RDY_OFF = HDR_OFF + (size_t)NR * 24 * NEP * 16;
constexpr size_t MKF_OFF = RDY_OFF + 16384;
constexpr size_t INF_OFF = MKF_OFF + 16384;
constexpr size_t AMX_OFF = INF_OFF + 49152;
constexpr size_t ACK_OFF = AMX_OFF + 16384;
constexpr size_t TK_OFF = ACK_OFF + 2048;
constexpr size_t CNT_OFF = TK_OFF + 2048;
constexpr size_t TMO_OFF = CNT_OFF + 2048;
constexpr size_t WS_NEED = TMO_OFF + 2048;
#define WS_BASE ((ws_bytes - WS_NEED) & ~(size_t)0x1fffff)
constexpr u64 DEADLINE = 100000000ull;
constexpr int HTB = 16384;
constexpr int TAB = 9 * HTB;
constexpr int ELUT = TAB + 768, DLUT = TAB + 896;
constexpr int OLUT = TAB + 1024;
constexpr int LDS_BYTES = OLUT + 5120;
constexpr int HCTL = 8 * 4352, SEQ_LDS = HCTL + 2048, RING = 38912, RSLOT = 10240, NRS = 10;
constexpr int NH = 8;
#define NJOIN 16
#define GEARLY 0
#define GLATE 1
#define POLL_SLEEP 1
#define PULL_SLEEP 2
#define DEC_SLEEP 1
#define AMX_SLEEP 12
#define WIN 7
#define NQ 2u
#define NF (6u + ((PUSHE - 1) & 1))
#define PW 0
#define W3N 11u
#define W2N 18u
#define LOOK 4
#define NT0 0
#define OUT_AUX 18
constexpr int AUX_SYS = 17;
constexpr int AUX_DEV = 16;

struct Params {
    const float* x; const float* w; char* me; signed char* out;
    int rank, pad;
    char* ws[NR];
};

__device__ __forceinline__ unsigned cvt_pk_bf16(float lo, float hi) { unsigned r; asm("v_cvt_pk_bf16_f32 %0, %1, %2" : "=v"(r) : "v"(lo), "v"(hi)); return r; }
__device__ __forceinline__ __amdgpu_buffer_rsrc_t mk_rsrc(const void* p, unsigned bytes) { return __builtin_amdgcn_make_buffer_rsrc((void*)p, (short)0, (int)bytes, 0x00020000); }
__device__ __forceinline__ u64 ld_sys64(const u64* p) { return __hip_atomic_load(p, __ATOMIC_RELAXED, __HIP_MEMORY_SCOPE_SYSTEM); }
__device__ __forceinline__ void st_sys(unsigned* p, unsigned v) { __hip_atomic_store(p, v, __ATOMIC_RELAXED, __HIP_MEMORY_SCOPE_SYSTEM); }
__device__ __forceinline__ void st_dev(unsigned* p, unsigned v) { __hip_atomic_store(p, v, __ATOMIC_RELAXED, __HIP_MEMORY_SCOPE_AGENT); }
__device__ __forceinline__ int rfl(int v) { return __builtin_amdgcn_readfirstlane(v); }
__device__ __forceinline__ unsigned wave_scan(unsigned v) {
    v += (unsigned)__builtin_amdgcn_update_dpp(0, (int)v, 0x111, 0xf, 0xf, false); v += (unsigned)__builtin_amdgcn_update_dpp(0, (int)v, 0x112, 0xf, 0xf, false);
    v += (unsigned)__builtin_amdgcn_update_dpp(0, (int)v, 0x114, 0xf, 0xf, false); v += (unsigned)__builtin_amdgcn_update_dpp(0, (int)v, 0x118, 0xf, 0xf, false);
    v += (unsigned)__builtin_amdgcn_update_dpp(0, (int)v, 0x142, 0xa, 0xf, false); v += (unsigned)__builtin_amdgcn_update_dpp(0, (int)v, 0x143, 0xc, 0xf, false);
    return v;
}
__device__ __forceinline__ void wave_scan2(const unsigned (&in)[2], unsigned (&v)[2]) {
#define SCAN_STEP(c, m) do { v[0] += (unsigned)__builtin_amdgcn_update_dpp(0, (int)v[0], c, m, 0xf, false); v[1] += (unsigned)__builtin_amdgcn_update_dpp(0, (int)v[1], c, m, 0xf, false); } while (0)
    v[0] = in[0]; v[1] = in[1];
    SCAN_STEP(0x111, 0xf); SCAN_STEP(0x112, 0xf); SCAN_STEP(0x114, 0xf); SCAN_STEP(0x118, 0xf); SCAN_STEP(0x142, 0xa); SCAN_STEP(0x143, 0xc);
}
__device__ __forceinline__ unsigned wave_umax(unsigned v) {
    v = max(v, (unsigned)__builtin_amdgcn_update_dpp(0, (int)v, 0x111, 0xf, 0xf, false)); v = max(v, (unsigned)__builtin_amdgcn_update_dpp(0, (int)v, 0x112, 0xf, 0xf, false));
    v = max(v, (unsigned)__builtin_amdgcn_update_dpp(0, (int)v, 0x114, 0xf, 0xf, false)); v = max(v, (unsigned)__builtin_amdgcn_update_dpp(0, (int)v, 0x118, 0xf, 0xf, false));
    v = max(v, (unsigned)__builtin_amdgcn_update_dpp(0, (int)v, 0x142, 0xa, 0xf, false)); v = max(v, (unsigned)__builtin_amdgcn_update_dpp(0, (int)v, 0x143, 0xc, 0xf, false));
    return v;
}
__device__ __forceinline__ float max3f(float a, float b, float c) { float r; asm("v_max3_f32 %0, %1, %2, %3" : "=v"(r) : "v"(a), "v"(b), "v"(c)); return r; }
__device__ __forceinline__ void lds_or_bits(LAS unsigned char* base, unsigned bit, unsigned val) {
    const u64 h = (u64)val << (bit & 31u); LAS unsigned* hp = (LAS unsigned*)(base + ((bit >> 3) & ~3u));
    __hip_atomic_fetch_or(hp, (unsigned)h, __ATOMIC_RELAXED, __HIP_MEMORY_SCOPE_WORKGROUP); __hip_atomic_fetch_or(hp + 1, (unsigned)(h >> 32), __ATOMIC_RELAXED, __HIP_MEMORY_SCOPE_WORKGROUP);
}
__device__ __forceinline__ void lds_or_neg(unsigned b4, int nb, unsigned val) {
    unsigned a; asm("v_mad_i32_i24 %0, %1, -4, %2" : "=v"(a) : "v"(nb >> 5), "s"(b4)); LAS unsigned* hp = (LAS unsigned*)(size_t)a;
    __hip_atomic_fetch_or(hp, __builtin_amdgcn_alignbit(val, 0u, (unsigned)nb), __ATOMIC_RELAXED, __HIP_MEMORY_SCOPE_WORKGROUP); __hip_atomic_fetch_or(hp + 1, val >> (nb & 31), __ATOMIC_RELAXED, __HIP_MEMORY_SCOPE_WORKGROUP);
}
__device__ __forceinline__ unsigned lds_get_bits(const LAS unsigned char* base, unsigned bit) {
    const LAS unsigned* hp = (const LAS unsigned*)(base + ((bit >> 3) & ~3u)); return __builtin_amdgcn_alignbit(hp[1], hp[0], bit & 31u);
}
__device__ __forceinline__ unsigned lds_add_rtn(LAS unsigned* p, unsigned v) { unsigned r, t;
    asm volatile("s_mov_b64 exec, 1\n ds_add_rtn_u32 %1, %2, %3\n s_mov_b64 exec, -1\n s_waitcnt lgkmcnt(0)\n v_readfirstlane_b32 %0, %1" : "=s"(r), "=&v"(t) : "v"(p), "v"(v) : "memory"); return r; }
__device__ __forceinline__ void lds_add(LAS unsigned* p, unsigned v) { asm volatile("s_mov_b64 exec, 1\n ds_add_u32 %0, %1\n s_mov_b64 exec, -1" :: "v"(p), "v"(v) : "memory"); }
__device__ __forceinline__ void lds_put(LAS unsigned* p, unsigned v) { asm volatile("s_mov_b64 exec, 1\n ds_write_b32 %0, %1\n s_mov_b64 exec, -1" :: "v"(p), "v"(v) : "memory"); }
__device__ __forceinline__ void lds_put4(LAS unsigned* p, u32x4 v) { asm volatile("s_mov_b64 exec, 1\n ds_write_b128 %0, %1\n s_mov_b64 exec, -1" :: "v"(p), "v"(v) : "memory"); }
__device__ __forceinline__ bool give_up(unsigned* tmo, unsigned call, u64 t0) {
    const unsigned tv = (unsigned)rfl((int)__hip_atomic_load(tmo, __ATOMIC_RELAXED, __HIP_MEMORY_SCOPE_AGENT));
    if (tv == call || __builtin_amdgcn_s_memrealtime() - t0 > DEADLINE) { st_dev(tmo, call); return true; }
    return false;
}
#define BSTORE(v, rs, voff, soff, aux) do { __builtin_amdgcn_raw_buffer_store_b128(v, rs, voff, soff, aux); asm volatile("s_nop 1" :: "v"(v) : "memory"); } while (0)
#define VWAIT0 __builtin_amdgcn_s_waitcnt(0x0F70)
#define WAIT_V(n) asm volatile("s_waitcnt vmcnt(" #n ")" ::: "memory")
#define WAIT_L(n) do { asm volatile("" ::: "memory"); __builtin_amdgcn_s_waitcnt(0xC07F | ((n) << 8)); asm volatile("" ::: "memory"); } while (0)
#define BAR do { asm volatile("" ::: "memory"); __builtin_amdgcn_s_barrier(); asm volatile("" ::: "memory"); } while (0)
#define RBAR __builtin_amdgcn_s_barrier()
#define SCHED __builtin_amdgcn_sched_barrier(0)

__device__ __forceinline__ unsigned inf_idx(int src, unsigned blk, unsigned e) { return (((((unsigned)src + e) & 7u) * 24u + blk) * 32u + e) * 2u; }
__device__ __forceinline__ unsigned unit_base(int src, unsigned e, unsigned blk, unsigned h) {
    return blk < 16u ? (unsigned)XS_OFF + 2u * e * (unsigned)XS_T + ((unsigned)src * GM + blk * 256u + h * 128u) * 16u
                     : (unsigned)WS_OFF + (2u * e + h) * (unsigned)WS_T + ((unsigned)src * NS + (blk - 16u) * 256u) * 16u;
}
#define IMM 1u
#define PMT 0xdcba9876fe543210ull
#define SPM 63u
#define RG 2
#define LOOKS 2
#define WB0 6u
#define W0LAG 7
#define TQUICK 72
#define SJ 1u
#define LATED 64u
#define TQ0 32
#define DLYW 8u
#define DLYE 10u
#define DLYL 18u
constexpr int lag0[24] = {LAG0}, lag1[24] = {LAG1};
struct alignas(128) Order { unsigned short v[1024]; };
constexpr int order_key(int b, int e) { return (e * (496 + 4 * (lag1[b] - lag0[b])) + 124 * (lag0[b] + (b >= 16 && e == 0 ? W0LAG : 0))) * 32 + b; }
constexpr Order make_order() { Order t = {};
    int n = 0;
    for (int e = 0; e < NEP; ++e) for (int b = 0; b < (e > 0 && e < PUSHE ? 16 : 24); ++b) { int i = n++;
        for (; i > 0 && order_key(t.v[i - 1] & 31, t.v[i - 1] >> 5) > order_key(b, e); --i) t.v[i] = t.v[i - 1];
        t.v[i] = (unsigned short)(b | (e << 5)); }
    return t; }
__device__ constexpr Order pull_order = make_order();
__device__ __forceinline__ unsigned bfi(unsigned m, unsigned a, unsigned b) { return (a & m) | (b & ~m); }
__device__ __forceinline__ unsigned perm(unsigned hi, unsigned lo, unsigned sel) { return __builtin_amdgcn_perm(hi, lo, sel); }
__device__ __forceinline__ unsigned dot4(unsigned a, unsigned b, unsigned c) { return __builtin_amdgcn_udot4(a, b, c, false); }
template <int NG>
__device__ __forceinline__ unsigned code_level(const unsigned* d, unsigned T1, unsigned T0, const LAS unsigned char* lds0, unsigned elut, unsigned* cw, unsigned* ch, unsigned* nb, unsigned& total) {
    unsigned c[NG], n[NG], t[NG]; u32x2 w[NG];
#pragma unroll
    for (int i = 0; i < NG; ++i) { c[i] = perm(T1, T0, (0x86868686u - d[i]) ^ 0x80808080u); t[i] = dot4(c[i] & 0x08080808u, 0x08040201u, elut); n[i] = (unsigned)__builtin_popcount(c[i] & 0x3c3c3c3cu); }
    SCHED;
#pragma unroll
    for (int i = 0; i < NG; ++i) w[i] = *(const LAS u32x2*)(lds0 + t[i]);
    SCHED;
    unsigned lb = 0u;
#pragma unroll
    for (int i = 0; i < NG; i += 2) { nb[i >> 1] = n[i] + n[i + 1]; lb += nb[i >> 1]; }
    const unsigned incl = wave_scan(lb);
#pragma unroll
    for (int i = 0; i < NG; i += 4) cw[i >> 2] = bfi(0x0f0f0f0fu, bfi(0x03030303u, c[i], c[i + 1] << 2), bfi(0x03030303u, c[i + 2], c[i + 3] << 2) << 4);
#pragma unroll
    for (int i = 0; i < NG; ++i) c[i] = d[i] & 0x0f0f0f0fu;
    SCHED;
#pragma unroll
    for (int i = 0; i < NG; ++i) { t[i] = dot4(c[i], w[i].y, 0u); c[i] = dot4(c[i], w[i].x, 0u); }
    SCHED;
#pragma unroll
    for (int i = 0; i < NG; i += 2) ch[i >> 1] = (c[i] | (t[i] << 8)) | ((c[i + 1] | (t[i + 1] << 8)) << n[i]);
    total = (unsigned)__builtin_amdgcn_readlane((int)incl, 63);
    return incl - lb;
}
__device__ __forceinline__ void pack_unit(u32x4 (&o)[4], unsigned& h0, unsigned& h1, LAS unsigned char* lds0, unsigned xbo, unsigned elut, int lane) {
    LAS unsigned char* xb = lds0 + xbo;
    *(LAS u32x4*)(xb + lane * 16) = (u32x4){0u, 0u, 0u, 0u}; *(LAS u32x4*)(xb + 1536 + lane * 16) = (u32x4){0u, 0u, 0u, 0u};
    u32x4 p[2]; unsigned d[8], big = 0u;
#pragma unroll
    for (int i = 0; i < 8; ++i) {
        const unsigned hi = perm(o[i >> 1][(i & 1) * 2 + 1], o[i >> 1][(i & 1) * 2], 0x07050301u), lo = perm(o[i >> 1][(i & 1) * 2 + 1], o[i >> 1][(i & 1) * 2], 0x06040200u);
        p[i >> 2][i & 3] = bfi(0x80808080u, hi, lo);
        d[i] = bfi(0x01010101u, lo >> 7, hi << 1);
    }
    const unsigned top = (unsigned)min(max((__builtin_amdgcn_readlane((int)wave_scan(__builtin_amdgcn_sad_u8(d[0], 0u, __builtin_amdgcn_sad_u8(d[1], 0u, 0u))), 63) + 1956) >> 9, 15), 255);
    const unsigned top4 = top * 0x01010101u;
#pragma unroll
    for (int i = 0; i < 8; ++i) { d[i] = top4 - d[i]; big |= d[i]; }
    unsigned exc = 0u, hx = 0u;
    if (__ballot((big & 0xf0f0f0f0u) != 0u) != 0ull) {
        unsigned x[8]; hx = 1u;
#pragma unroll
        for (int i = 0; i < 8; ++i) { const unsigned e = top4 - d[i]; d[i] = ((top4 | 0x80808080u) - (e & 0x7f7f7f7fu)) ^ (~(top4 ^ e) & 0x80808080u);
            const unsigned z = (((d[i] & 0x70707070u) + 0x70707070u) | d[i]) & 0x80808080u; x[i] = d[i] & (z | (z - (z >> 7))); }
        for (unsigned j = 0;; ++j) {
            const unsigned xm = max(max(max(x[0], x[1]), max(x[2], x[3])), max(max(x[4], x[5]), max(x[6], x[7])));
            const u64 any = __ballot(xm != 0u);
            if (any == 0ull) break;
            if (j == 3u) return;
            const unsigned xw = (unsigned)__builtin_amdgcn_readlane((int)xm, __builtin_ctzll(any)), v = xw >> (24 - (__builtin_clz(xw) & 24));
            exc |= ((top - v) & 255u) << (8u * j);
#pragma unroll
            for (int i = 0; i < 8; ++i) { const unsigned t = x[i] ^ (v * 0x01010101u), z = ~(((t & 0x7f7f7f7fu) + 0x7f7f7f7fu) | t) & 0x80808080u, f = z | (z - (z >> 7));
                d[i] = bfi(f, (18u + j) * 0x01010101u, d[i]); x[i] &= ~f; }
        }
    }
    unsigned cw[2], ch[4], nb[4], bits1, bits2;
    unsigned Pb = code_level<8>(d, 0xffffff80u, 0x8182ffffu, lds0, elut, cw, ch, nb, bits1);
    if (bits1 > 4096u) return;
    const unsigned n1 = (bits1 + 63u) >> 6;
#pragma unroll
    for (int i = 0; i < 4; ++i) { lds_or_bits(xb, Pb, ch[i]); Pb += nb[i]; }
    asm volatile("" ::: "memory");
    u64 xs = *(const volatile LAS u64*)(xb + lane * 8);
    *(LAS u32x2*)(xb + 1024 + lane * 8) = (u32x2){cw[0], cw[1]};
    { const int left = (int)bits1 - lane * 64; xs |= left < 64 ? 0x5555555555555555ull << max(left, 0) : 0ull; }
    const unsigned x0 = (unsigned)xs, x1 = (unsigned)(xs >> 32);
    d[0] = x0 & 0x0f0f0f0fu; d[1] = (x0 >> 4) & 0x0f0f0f0fu; d[2] = x1 & 0x0f0f0f0fu; d[3] = (x1 >> 4) & 0x0f0f0f0fu;
    Pb = code_level<4>(d, 0xffff80ffu, 0xffff8182u, lds0, elut, cw, ch, nb, bits2) + n1 * 32u;
    if (bits2 > 1920u) return;
    if ((unsigned)lane < n1) { *(LAS unsigned*)(xb + 1536 + lane * 4) = cw[0]; lds_or_bits(xb + 1536, Pb, ch[0]); lds_or_bits(xb + 1536, Pb + nb[0], ch[1]); }
    asm volatile("" ::: "memory");
    o[2] = *(const volatile LAS u32x4*)(xb + 1024 + lane * 16);
    o[0] = p[0]; o[1] = p[1]; h0 = exc | (top << 24); h1 = hx | (n1 << 4) | (((max(n1 * 32u + bits2, 1536u) - 513u) >> 10) << 1);
}
template <int NW, int NU>
__device__ __forceinline__ void decode_level(const unsigned (&cw)[NU][NW], const unsigned (&ew)[NU][NW], const unsigned (&T)[NU], unsigned (&pos)[NU], const LAS unsigned char* lds0, unsigned (&g)[NU][8]) {
    unsigned sel[NU][4 * NW], lo[NU][4 * NW], hi[NU][4 * NW], ps[NU][4 * NW];
#pragma unroll
    for (int i = 0; i < 4 * NW; ++i)
#pragma unroll
        for (int u = 0; u < NU; ++u) { const unsigned e = ((i & 2) ? ew[u][i >> 2] >> 4 : ew[u][i >> 2]) & ((i & 1) ? 0x04040404u : 0x01010101u);
            sel[u][i] = dot4(e, (i & 1) ? 0x08040201u : 0x20100804u, (unsigned)DLUT); ps[u][i] = pos[u]; pos[u] += (unsigned)__builtin_popcount(e); }
#pragma unroll
    for (int u = 0; u < NU; ++u)
#pragma unroll
        for (int i = 0; i < 4 * NW; ++i) { const LAS unsigned* q = (const LAS unsigned*)(lds0 + (ps[u][i] & ~3u)); lo[u][i] = q[0]; hi[u][i] = q[1]; sel[u][i] = *(const LAS unsigned*)(lds0 + sel[u][i]); }
    SCHED;
#pragma unroll
    for (int u = 0; u < NU; ++u)
#pragma unroll
        for (int i = 0; i < 4 * NW; ++i) g[u][i] = perm(0u, T[u], (cw[u][i >> 2] >> (2 * (i & 3))) & 0x03030303u);
    SCHED;
#pragma unroll
    for (int u = 0; u < NU; ++u)
#pragma unroll
        for (int i = 0; i < 4 * NW; ++i) g[u][i] = perm(__builtin_amdgcn_alignbyte(hi[u][i], lo[u][i], ps[u][i]), g[u][i], sel[u][i]);
}
__device__ __forceinline__ unsigned rot4(unsigned e) { return bfi(0x80808080u, e << 7, e >> 1); }
template <int NU>
__device__ __forceinline__ void unpack_piece(const u32x4 (&p)[NU][2], const u32x2 (&cw)[NU], const unsigned (&c2)[NU][1], const unsigned (&n3)[NU], const unsigned (&h0)[NU], unsigned h1, LAS unsigned char* lds0, unsigned wb, int lane, u32x4 (&o)[NU][4]) {
    unsigned top4[NU], e2[NU][1], c1[NU][2], e1[NU][2], cn[2] = {0u, 0u}, ex[2], g[NU][8], T[NU], pos[NU]; u32x2 v[NU], sy[NU];
#pragma unroll
    for (int u = 0; u < NU; ++u) { top4[u] = (h0[u] >> 24) * 0x01010101u;
        const unsigned lo = n3[u] & 0x0f0f0f0fu, hi = (n3[u] >> 4) & 0x0f0f0f0fu; sy[u] = (u32x2){perm(hi, lo, 0x05010400u), perm(hi, lo, 0x07030602u)};
        v[u] = (u32x2){rot4(top4[u] - sy[u].x), rot4(top4[u] - sy[u].y)};
        e2[u][0] = c2[u][0] & (c2[u][0] >> 1) & 0x55555555u; c1[u][0] = cw[u].x; c1[u][1] = cw[u].y; e1[u][0] = cw[u].x & (cw[u].x >> 1) & 0x55555555u; e1[u][1] = cw[u].y & (cw[u].y >> 1) & 0x55555555u;
        cn[u] = (unsigned)__builtin_popcount(e2[u][0]) | ((unsigned)(__builtin_popcount(e1[u][0]) + __builtin_popcount(e1[u][1])) << 16); }
    if (h1 & 0x10001u) {
#pragma unroll
        for (int u = 0; u < NU; ++u)
#pragma unroll
            for (int i = 0; i < 2; ++i) { const unsigned c = perm(0xffffff80u, 0x8182ffffu, (0x86868686u - sy[u][i]) ^ 0x80808080u), f = c & 0x10101010u;
                v[u][i] = bfi((f << 4) - (f >> 4), v[u][i], perm(0u, rot4(h0[u]), c & 0x03030303u)); } }
#pragma unroll
    for (int u = 0; u < NU; ++u) *(LAS u32x2*)(lds0 + wb + u * 2176 + lane * 8) = v[u];
    if (NU == 2) wave_scan2(cn, ex); else ex[0] = wave_scan(cn[0]);
#pragma unroll
    for (int u = 0; u < NU; ++u) { ex[u] -= cn[u]; T[u] = rot4(top4[u] - 0x00060501u); pos[u] = wb + u * 2176u + (ex[u] & 0xffffu); }
    asm volatile("" ::: "memory");
    decode_level<1, NU>(c2, e2, T, pos, lds0, g);
#pragma unroll
    for (int u = 0; u < NU; ++u) { *(LAS u32x4*)(lds0 + wb + u * 2176 + 1024 + lane * 16) = (u32x4){perm(g[u][1], g[u][0], 0x05010400u), perm(g[u][1], g[u][0], 0x07030602u), perm(g[u][3], g[u][2], 0x05010400u), perm(g[u][3], g[u][2], 0x07030602u)};
        T[u] = rot4(top4[u] - 0x00040302u); pos[u] = wb + u * 2176u + 1024u + min(ex[u] >> 16, 1024u); }
    asm volatile("" ::: "memory");
    decode_level<2, NU>(c1, e1, T, pos, lds0, g);
#pragma unroll
    for (int u = 0; u < NU; ++u)
#pragma unroll
        for (int i = 0; i < 8; ++i) { const unsigned mm = p[u][i >> 2][i & 3], hi = bfi(0x80808080u, mm, g[u][i]), lo = bfi(0x80808080u, g[u][i], mm);
            o[u][i >> 1][(i & 1) * 2] = perm(hi, lo, 0x05010400u); o[u][i >> 1][(i & 1) * 2 + 1] = perm(hi, lo, 0x07030602u); }
}
__device__ __forceinline__ void make_dlut(LAS unsigned char* dlut, int lane) {
    if (lane < 16) { unsigned w = 0u, n = 4u; for (int b = 0; b < 4; ++b) w |= (((lane >> b) & 1) ? n++ : (unsigned)b) << (8 * b); *(LAS unsigned*)(dlut + lane * 4) = w; }
}
__device__ __forceinline__ void make_elut(LAS unsigned char* elut, int lane) {
    if (lane < 16) { u32x2 w = {0u, 0u}; int n = 0;
        for (int b = 0; b < 4; ++b) if ((lane >> b) & 1) { const unsigned v = ((n & 1) ? 16u : 1u) << (8 * b); if (n < 2) w.x |= v; else w.y |= v; ++n; }
        *(LAS u32x2*)(elut + lane * 8) = w; }
}
__device__ __forceinline__ void make_olut(LAS unsigned char* lut, int tid) {
    if (tid < 256) { u64 ga = 0x0c0c0c0c0c0c0c0cull, sc = ga; unsigned ru = 0u, n = 0u;
        for (unsigned b = 0; b < 8u; ++b) {
            if ((tid >> b) & 1) { const unsigned s = (n & 1u) * 4u + (n >> 1); ga ^= (u64)(b ^ 12u) << (8u * s); sc ^= (u64)(s ^ 12u) << (8u * b); ++n; }
            else ru += 1u << (4u * n); }
        *(LAS u64*)(lut + tid * 8) = ga; *(LAS u64*)(lut + 2048 + tid * 8) = sc; *(LAS unsigned*)(lut + 4096 + tid * 4) = ru; }
}
__device__ __forceinline__ void cvt_unit(bool isx, const f32x4 (&v)[8], u32x4 (&o)[4], LAS unsigned char* xb, int lane) {
    if (isx) { const int xw = ((lane >> 1) & 1) * 1088 + (lane >> 2) * 16 + (lane & 1) * 8;
#pragma unroll
        for (int j = 0; j < 8; ++j) *(LAS u32x2*)(xb + xw + (j >> 2) * 2176 + (j & 3) * 256) = (u32x2){cvt_pk_bf16(v[j][0], v[j][1]), cvt_pk_bf16(v[j][2], v[j][3])};
        asm volatile("" ::: "memory");
#pragma unroll
        for (int b = 0; b < 4; ++b) o[b] = *(const volatile LAS u32x4*)(xb + b * 1088 + lane * 16);
        asm volatile("" ::: "memory");
    } else {
#pragma unroll
        for (int r = 0; r < 4; ++r) o[r] = (u32x4){cvt_pk_bf16(v[0][r], v[1][r]), cvt_pk_bf16(v[2][r], v[3][r]), cvt_pk_bf16(v[4][r], v[5][r]), cvt_pk_bf16(v[6][r], v[7][r])};
    }
}

__device__ __forceinline__ unsigned piece_off(unsigned bk, unsigned e, unsigned dst) {
    return (unsigned)OB_OFF + (bk < 16u ? e * 16u + bk : 496u + (dst * 32u + e) * 8u + bk) * PST;
}
__device__ __forceinline__ void store_unit(const u32x4 (&o)[4], unsigned h1, __amdgpu_buffer_rsrc_t rs, unsigned base, unsigned hf, int lane) {
#pragma unroll
    for (int b = 0; b < 2; ++b) BSTORE(o[b], rs, (unsigned)lane * 16u, base + hf * 2048u + b * 1024u, AUX_SYS);
    if (h1 & 0x8000u) {
#pragma unroll
        for (int b = 2; b < 4; ++b) BSTORE(o[b], rs, (unsigned)lane * 16u, base + 4096u + hf * 2048u + b * 1024u, AUX_SYS);
    } else BSTORE(o[2], rs, lane < 44 || (h1 & 6u) ? (unsigned)lane * 16u + (lane < 32 ? 4096u + hf * 512u : lane < 44 ? 4608u + hf * 192u : 4800u + hf * 320u) : 0xfffffff0u, base, AUX_SYS);
}

__global__ __launch_bounds__(512, 2) void fused_kernel(Params P) {
    extern __shared__ __attribute__((aligned(16))) unsigned char shm_raw[];
    LAS unsigned char* lds = (LAS unsigned char*)shm_raw;
    u64 code; unsigned tch;
    asm volatile("s_getpc_b64 %1\n .Lfk_first:\n s_mov_b64 exec, 3\n v_lshrrev_b32 %0, 5, %2\n v_and_or_b32 %0, %2, 1, %0\n v_lshl_add_u32 %0, %0, 7, %3\n v_min_u32 %0, .Lfk_last-.Lfk_first, %0\n global_load_dword %0, %0, %1\n s_mov_b64 exec, -1"
        : "=&v"(tch), "=&s"(code) : "v"(threadIdx.x), "s"((blockIdx.x >> 3) * 2048u) : "memory");
    const int tid = threadIdx.x, lane = tid & 63;
    const int wid = rfl(tid >> 6);
    const int bid = blockIdx.x, xcd = bid & 7, slot = bid >> 3;
    const int rank = P.rank;
    char* const dws = P.ws[(xcd + slot) & 7];
    asm volatile("" :: "s"(P.x), "s"(P.w), "s"(P.me), "s"(rank), "s"(dws));
    char* const myws = P.me;
    unsigned* const tmo = (unsigned*)(myws + TMO_OFF);
    unsigned* const cntp = (unsigned*)(myws + CNT_OFF) + bid;
    const unsigned prev_v = __hip_atomic_load(cntp, __ATOMIC_RELAXED, __HIP_MEMORY_SCOPE_AGENT);

    const int peer = (xcd + slot) & 7;
    unsigned call;
    const __amdgpu_buffer_rsrc_t ro = mk_rsrc(P.out, (unsigned)((size_t)GM * GN));
    const __amdgpu_buffer_rsrc_t rs = mk_rsrc(myws, (unsigned)CNT_OFF);
    const unsigned l0off = lane == 0 ? 0u : 0xfffffff0u;
#define FLAG(off) __builtin_amdgcn_raw_buffer_store_b32(call, rs, l0off, off, AUX_DEV)
    const unsigned l16 = (unsigned)lane * 16u, lbase = (unsigned)(size_t)(lds + RING);
#define DMA(g, l, off, aux) asm volatile("s_mov_b32 m0, %0\n s_nop 0\n global_load_lds_dwordx4 %1, %2 offset:" #off " " aux :: "s"(l), "v"(l16), "s"(g) : "memory")
#define WAIT_VN(n) asm volatile("s_waitcnt vmcnt(%0)" :: "n"(n) : "memory")
    const int gs = slot - NH, ts = gs < 16 ? gs : (gs & 3) * 4 + (gs < 20 ? GEARLY : GLATE), grp = ts & 3;
    const bool half = grp == GEARLY || grp == GLATE;
    const unsigned wb = (unsigned)wid * 4352u;
    const u64 wsv = (u64)P.ws[lane & 7];
    LAS unsigned* ctl = (LAS unsigned*)(lds + HCTL);
    const bool hs = gs < 0 && peer != rank;
    {
        const unsigned i64 = (unsigned)(half ? (ts >> 2) + (gs & 16) / 4 : (ts >> 2) + (grp > (GEARLY > 0 && GLATE > 0 ? 0 : GEARLY != 1 && GLATE != 1 ? 1 : 2) ? 4 : 0)) * 8u + xcd;
        unsigned nb = i64 + (half ? (grp == GLATE ? 128u : 64u) : 0u), bk, eg, to;
#define BATCH(n, bk, eg, to) do { const unsigned _x = (n) < 32u ? ((((n) & 7u) + ((n) & 16u) / 2u + WB0) & 15u) + ((n) & 8u) * 2u : (n) < 160u ? (n) - 96u : (n) - 120u, _o = (n) < 64u ? (n) - 32u : (n) < 128u ? (n) - 88u : (n) - 120u, _w = (n) < 120u ? (n) - 8u : (n) - 136u; \
            if ((n) < 32u || ((n) >= 128u && (n) < 160u) || ((n) >= 184u && (n) < 248u)) { bk = _x & 15u; eg = _x >> 4; to = 0xffu; } \
            else if (((n) >= 64u && (n) < 120u) || (n) >= 248u) { bk = 16u + (_w & 7u); eg = _w / 56u; to = 1u << ((rank + 1u + ((_w % 56u) >> 3)) & 7u); } \
            else { bk = 16u + (_o & 7u); eg = _o >> 3; to = 1u << rank; } } while (0)
        if (gs < 0) nb = 544u + slot;
        BATCH(nb, bk, eg, to);
        if (hs) { bk = 16u + slot; eg = 0u; to = 1u << peer; }
        constexpr unsigned NBAT = 584u; unsigned bk0 = 0u, eg0 = 0u, to0 = 0u;
        nb = gs < 0 ? (hs ? ~0u : nb + 8u) : !half ? ~0u : 192u + (i64 >> 3) * 4u + (xcd & 3) + (~xcd & 4) * 16u + (grp == GLATE ? 0u : 32u);
        const unsigned hf = (unsigned)wid & 1u, ej = (unsigned)wid >> 1;
        f32x4 v[8];
#define U_LOAD(bk, eg, to, nt) do { const unsigned _e = (eg) * 4u + ej; \
            const float* _p = bk < 16u ? P.x + (size_t)(bk * 256u + hf * 128u + (unsigned)(lane >> 2)) * KS + _e * 16u + (lane & 3) * 4 : P.w + (size_t)(_e * 16u + hf * 8u) * GN + __builtin_ctz(to) * NS + (bk - 16u) * 256u + lane * 4; \
            const size_t _st = bk < 16u ? (size_t)16 * KS : (size_t)GN; \
            _Pragma("unroll") for (int j = 0; j < 8; ++j) v[j] = (nt) ? __builtin_nontemporal_load((const f32x4*)(_p + j * _st)) : *(const f32x4*)(_p + j * _st); } while (0)
        if (gs < 0 && !hs && wid == 0) { const unsigned k = ((unsigned)lane >> 3) & 3u;
            const u64 a = wsv + (k == 0u ? (unsigned)HDR_OFF : k == 1u ? (unsigned)OB_OFF : k == 2u ? piece_off(16u, 0u, rank) : (unsigned)WS_OFF);
            unsigned t; asm volatile("global_load_dword %0, %1, off sc0 sc1\n s_waitcnt vmcnt(0)" : "=&v"(t) : "v"(a) : "memory");
        }
        for (unsigned i = 0; i < (gs < 0 ? (hs ? 0u : DLYL) : !half ? (i64 < 16u ? 0u : DLYW) : grp == GLATE ? DLYL : DLYE); ++i) __builtin_amdgcn_s_sleep(8);
        U_LOAD(bk, eg, to, NT0);
        if (gs < 0 && wid == 1) { DMA((const char*)pull_order.v, (unsigned)(size_t)(lds + SEQ_LDS), 0, ""); DMA((const char*)pull_order.v, (unsigned)(size_t)(lds + SEQ_LDS), 1024, ""); }
        asm volatile("" ::: "memory"); SCHED;
        if (tid < 8) *(LAS unsigned*)(lds + TAB + 128 + tid * 4) = 0u;
        if (wid == 7) make_elut(lds + ELUT, lane);
        if (gs < 0) { if (tid < 128) ctl[tid] = tid == 1 || tid == 2 ? ~0u : 0u; if (wid == 6) make_dlut(lds + DLUT, lane); }
        WAIT_L(0); BAR;
        call = (unsigned)rfl((int)prev_v) + 1u;
        SCHED; VWAIT0; asm volatile("s_waitcnt vmcnt(0)" :: "v"(tch));
        for (unsigned r = 0;; ++r) {
            u32x4 o[4]; unsigned h0 = 0u, h1 = 0x8000u; const unsigned e = eg * 4u + ej;
            if (r == 1u) { U_LOAD(bk, eg, to, 1); asm volatile("" ::: "memory"); SCHED; VWAIT0; }
            cvt_unit(bk < 16u, v, o, lds + wb, lane);
            asm volatile("" : "+v"(o[0]), "+v"(o[1]), "+v"(o[2]), "+v"(o[3]) :: "memory");
            unsigned bk1, eg1, to1; BATCH(nb, bk1, eg1, to1);
            if (nb < NBAT && r) U_LOAD(bk1, eg1, to1, 1);
            asm volatile("" ::: "memory"); SCHED;
            const bool send = hs && e - 1u < PUSHE - 1u, stage = send || ((to >> rank) & 1u) != 0u;
            const __amdgpu_buffer_rsrc_t rt = mk_rsrc(send ? dws : myws, (unsigned)CNT_OFF);
            if (stage) { const unsigned base = unit_base(rank, e, bk, hf), st1 = bk < 16u ? (unsigned)XS_T : 1024u, st2 = bk < 16u ? 1024u : 2048u;
#pragma unroll
                for (int b = 0; b < 4; ++b) { if (send) WAIT_VN(PW); BSTORE(o[b], rt, (unsigned)lane * 16u, base + (b & 1) * st1 + (b >> 1) * st2, AUX_SYS); } }
            LAS unsigned char* hi = lds + TAB + 256 + r * 64u;
            const bool pk = to != 1u << rank && !send;
            if (pk) pack_unit(o, h0, h1, lds, wb, (unsigned)ELUT, lane);
#define TELL(r, bk, eg, to) do { const unsigned dn = lds_add_rtn((LAS unsigned*)(lds + TAB + 128) + (r), 1u << (4u * ej)) + (1u << (4u * ej)); \
            if ((hs || (r) == 0u ? ((dn >> (4u * ej)) & 3u) == 2u && (unsigned)(lane >> 3) == ej : dn == 0x2222u && lane < 32) && (lane & 7) != rank && ((to >> (lane & 7)) & 1u)) { \
                u32x4 hv = *(const volatile LAS u32x4*)(lds + TAB + 256 + (r) * 64u + (lane >> 3) * 16); hv.x = call; \
                asm volatile("global_store_dwordx4 %0, %1, off sc0 sc1\n s_nop 1" :: "v"(wsv + HDR_OFF + ((rank * 24u + bk) * 32u + eg * 4u + (lane >> 3)) * 16u), "v"(hv) : "memory"); \
            } asm volatile("" ::: "memory"); SCHED; } while (0)
            asm volatile("" : "+v"(o[0]), "+v"(o[1]), "+v"(o[2]) :: "memory"); VWAIT0; WAIT_V(0);
            if (stage) __builtin_amdgcn_raw_buffer_store_b32(call, rt, l0off, (unsigned)INF_OFF + (inf_idx(rank, bk, e) + hf) * 4u, AUX_SYS);
            if (send) break;
            if (pk) {
                store_unit(o, h1, rs, piece_off(bk, e, __builtin_ctz(to)), hf, lane);
                if (lane == 0) { *(LAS unsigned*)(hi + ej * 16u + 4u + hf * 4u) = h0; *(LAS unsigned short*)(hi + ej * 16u + 12u + hf * 2u) = (unsigned short)h1; }
            }
            if (r > IMM + 1u) TELL(r - 1u, bk0, eg0, to0);
            if (r <= IMM || nb >= NBAT) { VWAIT0; WAIT_V(0); TELL(r, bk, eg, to); }
            if (nb >= NBAT) break;
            bk0 = bk; eg0 = eg; to0 = to; bk = bk1; eg = eg1; to = to1; nb = nb < 384u ? nb + 128u : nb >= 544u ? nb + 8u : nb - 448u < 32u ? nb + 64u : ~0u;
        }
        if (lds_add_rtn((LAS unsigned*)(lds + TAB + 156), 1u) == 7u) __builtin_amdgcn_raw_buffer_store_b32(call, mk_rsrc(cntp, 4u), l0off, 0, AUX_DEV);
    }
    if (gs >= 0) {
        const int wr = wid >> 2, wc = wid & 3, fr = lane & 15, fq = lane >> 4;
        const int pm = (int)(PMT >> (grp * 16 + (xcd >> 1) * 4)) & 15, pn = (xcd & 1) * 4 + (ts >> 2);
        const int tile = pm * 8 + pn, a0 = gs >> 4;
        make_olut(lds + OLUT, tid);
        BAR;
#define SRC_OF(e) ((wid - (e)) & 7)
        const char* cA = myws + XS_OFF + (size_t)pm * 4096;
        const char* cB = myws + WS_OFF + (size_t)pn * 4096;
#define A_OF(e) (cA + (size_t)(2 * (e)) * XS_T + (size_t)SRC_OF(e) * (GM * 16))
#define B_OF(e) (cB + (size_t)(2 * (e)) * WS_T + (size_t)SRC_OF(e) * (NS * 16))
        const u64* fp = (const u64*)(myws + INF_OFF) + (wid * 24 + (lane < 32 ? pm : 16 + pn)) * 32 + (lane & 31);
        const u64 call2 = (u64)call * 0x100000001ull;
        const unsigned ldsw = (unsigned)wid * 2048u;
        const int aoff = fq * 2048 + (wr * 64 + fr) * 16, boff = fq * 2048 + (wc * 16 + fr) * 16;
        u64 fsamp = 0;
        unsigned used = 0; int en;
        unsigned known = 0; u64 seen = 0;
        const unsigned own = 0x01010101u << ((wid - rank) & 7);
        bool kdead = false;
#define SA(b, h) (((b) * 2 + (h)) * HTB)
#define SB(b, h) ((4 + (b) * 2 + (h)) * HTB)
        const unsigned lw = (unsigned)(size_t)lds + ldsw;
#define STAGE(bufoff, g, h) asm volatile("s_mov_b32 m0, %0\n s_nop 0\n global_load_lds_dwordx4 %1, %2 offset:%3 sc1\n global_load_lds_dwordx4 %1, %2 offset:%4 sc1" \
            :: "s"(lw + (bufoff) - (h) * 2048), "v"(l16), "s"(g), "n"((h) * 2048), "n"((h) * 2048 + 1024) : "memory")
#define LDA(dst, b, h) do { _Pragma("unroll") for (int m = 0; m < 4; ++m) _Pragma("unroll") for (int k = 0; k < 2; ++k) dst[m][k] = *(const LAS bf16x8*)(lds + SA(b, h) + aoff + m * 256 + k * 8192); } while (0)
#define LDB(dst, b, h) do { _Pragma("unroll") for (int n = 0; n < 2; ++n) _Pragma("unroll") for (int k = 0; k < 2; ++k) dst[n][k] = *(const LAS bf16x8*)(lds + SB(b, h) + boff + n * 1024 + k * 8192); } while (0)
#define NEXT_EPOCH(en) do { if ((known & ~used) == 0u) { const u64 _t0 = __builtin_amdgcn_s_memrealtime(); unsigned _n = 0; \
            const unsigned _w = (0xffffu << (__builtin_ctz(~known) & 24)) | own;        \
            for (;;) { const u64 _m = (((u64)_w << 32) | _w) & ~seen; u64 _v = 0; if ((_m >> lane) & 1ull) _v = ld_sys64(fp); \
                seen |= __ballot(_v == call2); known = (unsigned)seen & (unsigned)(seen >> 32); if ((known & ~used) != 0u) break; \
                if ((++_n & 63u) == 0u && give_up(tmo, call, _t0)) { seen = ~0ull; known = 0xffffffffu; kdead = true; break; } \
                __builtin_amdgcn_s_sleep(POLL_SLEEP); } \
            asm volatile("" ::: "memory"); } \
            en = __builtin_ctz(known & ~used); used |= 1u << en; } while (0)
#define NEXT(x) do { if (e != NEP - 1) { x; } } while (0)
#define LASTW do { if (e == NEP - 1) WAIT_V(0); } while (0)
        f32x4 acc[2][2][4][2];
#pragma unroll
        for (int a = 0; a < 2; ++a)
#pragma unroll
            for (int b = 0; b < 2; ++b)
#pragma unroll
                for (int m = 0; m < 4; ++m)
#pragma unroll
                    for (int n = 0; n < 2; ++n) acc[a][b][m][n] = (f32x4){0.f, 0.f, 0.f, 0.f};
        bf16x8 At[4][2], B0[2][2], B1[2][2];
        NEXT_EPOCH(en);
        bf16x8 B2[2][2];
#define EL(g, c) acc[(g) >> 2][((c) >> 1) & 1][(g) & 3][(c) & 1][(c) >> 2]
#define MASK_WORD(g2) do { unsigned _m = 0; _Pragma("unroll") for (int _e = 31; _e >= 0; --_e) _m = __builtin_amdgcn_alignbit(_m, __float_as_uint(EL((g2) * 2 + (_e >> 4), _e & 15)), 31); mk[g2] = ~_m; } while (0)
        u32x4 mk = {0u, 0u, 0u, 0u};
#define MEET1 do { asm volatile("s_setprio 1\n s_cmp_eq_u32 %0, 0\n s_cbranch_scc1 .Lm%=\n s_barrier\n s_setprio 2\n .Lm%=:" :: "s"(wr) : "scc", "memory"); } while (0)
#define MEET0 do { asm volatile("s_cmp_eq_u32 %0, 1\n s_cbranch_scc1 .Lm%=\n s_barrier\n .Lm%=:" :: "s"(wr) : "scc", "memory"); } while (0)
#define MMA2(ai, Bx, By) do { \
        _Pragma("unroll") for (int i = 0; i < 32; ++i) { const int bj = i >> 4, k = (i >> 3) & 1, m = (i >> 1) & 3, n = i & 1; \
            acc[ai][bj][m][n] = __builtin_amdgcn_mfma_f32_16x16x32_bf16(bj ? By[n][k] : Bx[n][k], At[m][k], acc[ai][bj][m][n], 0, 0, 0); } \
        SCHED; __builtin_amdgcn_s_setprio(0); } while (0)
        if (half) {
            const char* hA = myws + XS_OFF + (size_t)pm * 4096 + a0 * 2048; const char* hB = myws + WS_OFF + (size_t)pn * 4096;
            unsigned rs_ = 0u, rr = 0u;
#define HSTAGE(kt) do { const char* _a = hA + (size_t)(2 * en + (kt)) * XS_T + (size_t)SRC_OF(en) * (GM * 16); const char* _b = hB + (size_t)(2 * en + (kt)) * WS_T + (size_t)SRC_OF(en) * (NS * 16); \
            STAGE(rs_ + HTB, _b, 0); STAGE(rs_ + 2 * HTB, _b, 1); STAGE(rs_, _a, 0); rs_ = rs_ == 6 * HTB ? 0u : rs_ + 3 * HTB; } while (0)
#define HREAD() do { const LAS unsigned char* _a = lds + rr + aoff; const LAS unsigned char* _b = lds + rr + HTB + boff; \
            _Pragma("unroll") for (int n = 0; n < 2; ++n) _Pragma("unroll") for (int k = 0; k < 2; ++k) B0[n][k] = *(const LAS bf16x8*)(_b + n * 1024 + k * 8192); \
            _Pragma("unroll") for (int m = 0; m < 4; ++m) _Pragma("unroll") for (int k = 0; k < 2; ++k) At[m][k] = *(const LAS bf16x8*)(_a + m * 256 + k * 8192); \
            _Pragma("unroll") for (int n = 0; n < 2; ++n) _Pragma("unroll") for (int k = 0; k < 2; ++k) B1[n][k] = *(const LAS bf16x8*)(_b + HTB + n * 1024 + k * 8192); \
            rr = rr == 6 * HTB ? 0u : rr + 3 * HTB; } while (0)
#define SAMPLE() asm volatile("global_load_dwordx2 %0, %1, off sc0 sc1" : "+v"(fsamp) : "v"(fp) : "memory")
#define LOOK_AT() do { asm volatile("" : "+v"(fsamp)); seen |= __ballot(fsamp == call2); known = (unsigned)seen & (unsigned)(seen >> 32); } while (0)
            HSTAGE(0); HSTAGE(1);
            WAIT_V(6); RBAR;
            for (int e = 0; e < NEP; ++e) {
                HREAD();
                LOOK_AT();
                bool later = (known & ~used) == 0u;
                if (!later) { NEXT_EPOCH(en); HSTAGE(0); } else SAMPLE();
                if (wr == 1) { if (later) WAIT_V(1); WAIT_V(6); }
                WAIT_L(0); MEET1; SCHED;
                MMA2(0, B0, B1);
                if (wr == 0) { if (later) WAIT_V(1); WAIT_V(6); }
                MEET0; SCHED;
                HREAD();
                if (later) WAIT_V(0);
                if (e == NEP - 1) later = false;
                else {
                    if (later) { LOOK_AT(); if (wr == 1 || (known & ~used) != 0u) { NEXT_EPOCH(en); HSTAGE(0); later = false; } }
                    if (!later) { SAMPLE(); HSTAGE(1); }
                }
                if (wr == 1) WAIT_V(7);
                WAIT_L(0); MEET1; SCHED;
                MMA2(0, B0, B1);
                if (later) { NEXT_EPOCH(en); HSTAGE(0); HSTAGE(1); }
                WAIT_V(6);
                MEET0;
                SCHED;
            }
            MASK_WORD(0); MASK_WORD(1);
        } else {
        { const char* ax = A_OF(en); const char* b0 = B_OF(en);
        STAGE(SB(0, 1), b0, 1); STAGE(SB(0, 0), b0, 0); STAGE(SA(0, 0), ax, 0); STAGE(SB(1, 1), b0 + WS_T, 1); STAGE(SA(0, 1), ax, 1);
        STAGE(SA(1, 0), ax + XS_T, 0); STAGE(SB(1, 0), b0 + WS_T, 0);
        WAIT_V(8); RBAR;
        LDB(B1, 0, 1); WAIT_L(0); RBAR; }
        const char* a3 = A_OF(en) + XS_T;
        for (int e = 0; e < NEP; ++e) {
            LDB(B0, 0, 0); LDA(At, 0, 0); STAGE(SA(1, 1), a3, 1);
            if (e != NEP - 1) { LOOK_AT(); NEXT_EPOCH(en); }
            const char* a2 = A_OF(en); const char* b2 = B_OF(en); const char* b3 = b2 + WS_T; a3 = a2 + XS_T;
            NEXT(STAGE(SB(0, 1), b2, 1));
            if (e == NEP - 1) WAIT_V(6);
            WAIT_V(8); WAIT_L(0); MEET1; SCHED; MMA2(0, B0, B1); MEET0; SCHED;
            LDA(At, 0, 1); LDB(B2, 1, 1); NEXT(STAGE(SA(0, 0), a2, 0); STAGE(SB(0, 0), b2, 0));
            if (e == NEP - 1) WAIT_V(2);
            WAIT_V(8); WAIT_L(0); MEET1; SCHED; MMA2(1, B0, B1); MEET0; SCHED;
            LDB(B0, 1, 0); LDA(At, 1, 0);
            NEXT(SAMPLE(); STAGE(SA(0, 1), a2, 1); STAGE(SB(1, 1), b3, 1));
            LASTW; WAIT_V(9); WAIT_L(0); MEET1; SCHED; MMA2(0, B0, B2); MEET0; SCHED;
            LDA(At, 1, 1); LDB(B1, 0, 1); NEXT(STAGE(SA(1, 0), a3, 0); STAGE(SB(1, 0), b3, 0));
            if (e == NEP - 1) { MASK_WORD(0); MASK_WORD(1); }
            WAIT_V(8); WAIT_L(0); MEET1; SCHED; MMA2(1, B0, B2); MEET0; SCHED;
        }
        MASK_WORD(2); MASK_WORD(3);
        }
#pragma unroll
        for (int q = 0; q < 4; ++q) if (q < 2 || !half) __builtin_amdgcn_raw_buffer_store_b32(mk[q], rs, (unsigned)tid * 4u + q * 2048u, (unsigned)MK_OFF + (unsigned)((rank * 128 + tile) * 2 + a0) * 4096u, AUX_SYS);
        u32x4 pub = {call * 4u, 0u, call * 4u, 0u};
#define SEND(off, v) do { const u64 _p = wsv + (off) + (unsigned)((rank * 128 + pn * 16 + pm) * 2 + a0) * 8u; \
            if (half) { const u32x2 _h = {v.x, v.y}; asm volatile("global_store_dwordx2 %0, %1, off sc0 sc1" :: "v"(_p), "v"(_h) : "memory"); } \
            else asm volatile("global_store_dwordx4 %0, %1, off sc0 sc1\n s_nop 1" :: "v"(_p), "v"(v) : "memory"); } while (0)
        SCHED;
        LAS float* red = (LAS float*)(lds + TAB);
        float mx = 0.f;
#pragma unroll
        for (int a = 0; a < 2; ++a) if (a == 0 || !half) { float gmx[4];
#pragma unroll
            for (int g = 0; g < 4; ++g) { gmx[g] = max3f(EL(4 * a + g, 0), EL(4 * a + g, 1), mx);
#pragma unroll
                for (int c = 2; c < 16; c += 2) gmx[g] = max3f(gmx[g], EL(4 * a + g, c), EL(4 * a + g, c + 1)); }
            mx = fmaxf(max3f(gmx[0], gmx[1], gmx[2]), gmx[3]); }
        mx = __uint_as_float(wave_umax(__float_as_uint(mx) & 0x7fffffffu));
        if (lane == 63) red[wid] = kdead ? -1.0f : mx;
        WAIT_L(0); BAR;
        {
            float m8 = red[0], n8 = red[0];
#pragma unroll
            for (int i = 1; i < 8; ++i) { m8 = fmaxf(m8, red[i]); n8 = fminf(n8, red[i]); }
            if (wid == 7 && lane < 8 && n8 >= 0.f) { const u32x4 gr = {__float_as_uint(m8), call, __float_as_uint(m8), call}; SEND(AMX_OFF, gr); }
            if (wid == 7) WAIT_V(1); else WAIT_V(0);
            BAR;
            if (tid < 8 && n8 >= 0.f) SEND(RDY_OFF, pub);
        }
        LAS unsigned char* sbuf;
#pragma unroll
        for (int i = 0; i < 8; ++i) if (i < 4 || !half) *(LAS u32x4*)(lds + i * 8192 + tid * 16) = (u32x4){0u, 0u, 0u, 0u};
        LAS unsigned* alc = (LAS unsigned*)(red + 16);
        if (tid < 5) alc[tid] = tid < 4 ? 32u : 0u;
        const unsigned cn[2] = {(unsigned)__builtin_popcount(mk[0]) | ((unsigned)__builtin_popcount(mk[1]) << 16), (unsigned)__builtin_popcount(mk[2]) | ((unsigned)__builtin_popcount(mk[3]) << 16)};
        const unsigned inc[2] = {wave_scan(cn[0]), wave_scan(cn[1])};
        {
            const __amdgpu_buffer_rsrc_t ra = mk_rsrc(myws + AMX_OFF, 16384u);
            const u64 t0 = __builtin_amdgcn_s_memrealtime(); unsigned n = 0; bool dead = false; u32x4 gv = {0u, 0u, 0u, 0u}, gw = gv;
            for (;;) {
                asm volatile("" ::: "memory");
                if (gv.y != call || gv.w != call) gv = __builtin_amdgcn_raw_buffer_load_b128(ra, (unsigned)lane * 16u, (unsigned)wid * 2048u, AUX_SYS);
                if (gw.y != call || gw.w != call) gw = __builtin_amdgcn_raw_buffer_load_b128(ra, (unsigned)lane * 16u + 1024u, (unsigned)wid * 2048u, AUX_SYS);
                if (__all(gv.y == call && gv.w == call && gw.y == call && gw.w == call)) break;
                if ((++n & 63u) == 0u && give_up(tmo, call, t0)) { dead = true; break; }
                __builtin_amdgcn_s_sleep(AMX_SLEEP);
            }
            float gm = fmaxf(fmaxf(__uint_as_float(gv.x), __uint_as_float(gv.z)), fmaxf(__uint_as_float(gw.x), __uint_as_float(gw.z)));
            gm = __uint_as_float(wave_umax(__float_as_uint(gm)));
            if (lane == 63) red[8 + wid] = dead ? -1.0f : gm;
        }
        WAIT_L(0); BAR;
        float amax = red[8], amin = red[8];
#pragma unroll
        for (int i = 1; i < 8; ++i) { amax = fmaxf(amax, red[8 + i]); amin = fminf(amin, red[8 + i]); }
        const bool bad = rfl(amin < 0.f ? 1 : 0) != 0;
        const float rinv = amax > 0.f ? 1.0f / amax : 0.f;
        u32x4 wq[8];
#define TOLD() do { WAIT_V(0); pub[(q - 1) & 2] += 1u; const unsigned _n = lds_add_rtn(alc + 4, 1u); if (_n == 8u * q - 1u && lane < 8 && !bad) SEND(RDY_OFF, pub); SCHED; } while (0)
#pragma unroll
        for (int q = 0; q < 4; ++q) if (q < 2 || !half) {
            sbuf = lds + q * 16384;
            const unsigned mw = mk[q];
            u32x2 ga[4]; unsigned lo[4], run[4], ln[4], b5 = 0u;
            const bool two = q == 2 || (unsigned)((a0 * 2 + q) * 128 + tile - TQ0) < 384u + TQUICK - TQ0;
#pragma unroll
            for (int d = 0; d < 4; ++d) ga[d] = *(const LAS u32x2*)(lds + OLUT + ((mw >> (8 * d)) & 255u) * 8u);
            SCHED;
#pragma unroll
            for (int d = 0; d < 8; ++d) { const int g = 2 * q + (d >> 2), c = (d & 3) * 4;
                unsigned w = 0u;
#pragma unroll
                for (int j = 0; j < 4; ++j) w = __builtin_amdgcn_cvt_pk_u8_f32(EL(g, c + j) * (rinv * 127.f), j, w);
                wq[g][d & 3] = w;
            }
            SCHED;
            if (q > 0) TOLD();
#pragma unroll
            for (int d = 0; d < 4; ++d) { const unsigned one = two ? 0u : 0x10101010u, w0 = wq[2 * q + (d >> 1)][(d & 1) * 2] + one, w1 = wq[2 * q + (d >> 1)][(d & 1) * 2 + 1] + one;
                const unsigned ev = perm(w1, w0, ga[d].x), od = perm(w1, w0, ga[d].y); lo[d] = bfi(0x0f0f0f0fu, ev, od << 4); run[d] = bfi(0x0f0f0f0fu, ev >> 4, od); }
            if (two) {
#pragma unroll
                for (int d = 3; d >= 0; --d) { const unsigned l = lo[d], h = run[d], f = (h + 0x77777777u) & 0x88888888u;
                    run[d] = (bfi(0x11111111u, l >> 3, h << 1) & ((f << 1) - (f >> 3))) - (f >> 2); lo[d] = l | ((h + 0x66666666u) & 0x88888888u);
                    b5 = (b5 << __builtin_popcount((mw >> (8 * d)) & 255u)) | (__builtin_amdgcn_udot8(f, 0x84210000u, 0u, false) * 2u + __builtin_amdgcn_udot8(f >> 3, 0x8421u, 0u, false)); }
            }
#pragma unroll
            for (int d = 0; d < 4; ++d) ln[d] = __builtin_amdgcn_udot8(run[d], 0x11111111u, 0u, false);
            const unsigned ul = (ln[0] + ln[1]) + (ln[2] + ln[3]), ui = wave_scan(ul);
            const unsigned nq = ((unsigned)__builtin_amdgcn_readlane((int)inc[q >> 1], 63) >> (16 * (q & 1))) & 0xffffu;
            const unsigned need = __ballot(max(max(ln[0], ln[1]), max(ln[2], ln[3])) > 32u) == 0ull ? nq + (two ? (nq + 3u) >> 2 : 0u) + (((unsigned)__builtin_amdgcn_readlane((int)ui, 63) + 4u) >> 2) : 32768u;
            unsigned at = 0u;
            if (lane == 0) asm volatile("ds_add_rtn_u32 %0, %1, %2" : "=v"(at) : "v"(alc + q), "v"(need) : "memory");
            SCHED;
#pragma unroll
            for (int d = 0; d < 4; ++d) { unsigned a = 1u;
#pragma unroll
                for (int j = 6; j >= 0; --j) a = (a << ((run[d] >> (4 * j)) & 15u)) | 1u;
                run[d] = a; }
            asm volatile("s_waitcnt lgkmcnt(0)" : "+v"(at), "+v"(run[0]), "+v"(run[1]), "+v"(run[2]), "+v"(run[3]) :: "memory");
            at = (unsigned)rfl((int)at);
            if (at + need <= 32760u) {
                if (two) at += (nq + 3u) >> 2;
                const unsigned bf = ((inc[q >> 1] - cn[q >> 1]) >> (16 * (q & 1))) & 0xffffu, b4 = (unsigned)(size_t)sbuf - 4u;
                int pn = -4 * (int)(at + bf), pu = (int)(ul - ui - (at + nq) * 4u);
                if (two) lds_or_neg(b4, (int)(nq - at * 4u - bf), b5);
                if (lane == 0) *(LAS unsigned short*)(sbuf + wid * 2) = (unsigned short)((at + nq) | 0x8000u);
#pragma unroll
                for (int d = 0; d < 4; ++d) { lds_or_neg(b4, pn, lo[d]); lds_or_neg(b4, pu, run[d]); pn -= __builtin_popcount((mw >> (8 * d)) & 255u) * 4; pu -= (int)ln[d]; }
                if (lane == 63) lds_or_neg(b4, pu, 1u);
            }
            WAIT_V(0); WAIT_L(0); BAR;
            const unsigned took = (unsigned)rfl((int)*(const volatile LAS unsigned*)(alc + q)), nbytes = took > 32760u ? 16384u : ((took + 255u) >> 8) << 7;
            pub[1 + (q & 2)] |= nbytes << (16 * (q & 1));
            const __amdgpu_buffer_rsrc_t rd = mk_rsrc(myws + ST_OFF + (size_t)(tile * 4 + a0 * 2 + q) * SST, nbytes);
#pragma unroll
            for (int k = 0; k < 2; ++k) if (k == 0 || nbytes > 8192u) { const u32x4 sv = took > 32760u ? wq[2 * q + k] : *(const LAS u32x4*)(sbuf + k * 8192 + tid * 16); BSTORE(sv, rd, (unsigned)tid * 16u + k * 8192u, 0, AUX_SYS); }
            SCHED;
        }
        {
            const unsigned voff = (unsigned)((pm * 256 + a0 * 128 + wr * 64 + fr) * GN + rank * NS + pn * 256 + wc * 64 + fq * 16);
#pragma unroll
            for (int g = 0; g < 8; ++g) if (g < 4 || !half) { const u32x4 qb = bad ? (u32x4){0x80808080u, 0x80808080u, 0x80808080u, 0x80808080u} : wq[g];
                BSTORE(qb, ro, voff, (unsigned)(((g >> 2) * 128 + (g & 3) * 16) * GN), OUT_AUX); }
        }
        if (half) WAIT_V(4);
        WAIT_V(8); BAR;
        pub.x = pub.z = call * 4u + 2u;
        if (tid < 8 && !bad) SEND(RDY_OFF, pub);
            if (slot >= 32 - NJOIN) { if (tid < 128) ctl[tid] = tid == 1 || tid == 2 ? ~0u : tid == 7 ? 4u : 0u; WAIT_L(0); BAR;  }
    }
    if (gs < 0 || slot >= 32 - NJOIN) {
        if (peer == rank) {
            if (slot == 0 && wid == 0) {
                const u64 t0 = __builtin_amdgcn_s_memrealtime(); unsigned k = 0;
                for (;;) {
                    asm volatile("" ::: "memory");
                    const u32x4 a = __builtin_amdgcn_raw_buffer_load_b128(mk_rsrc(myws + ACK_OFF, 1024u), l16, 0, AUX_SYS);
                    if (__all((a.x == call && a.y == call && a.z == call && a.w == call) || (lane >> 3) == rank || ((lane & 7) >= NH / 4 && (lane & 7) < 8 - NJOIN / 4))) break;
                    if ((++k & 15u) == 0u && give_up(tmo, call, t0)) break;
                    if (__builtin_amdgcn_s_memrealtime() - t0 < 10000ull) __builtin_amdgcn_s_sleep(100); else __builtin_amdgcn_s_sleep(2);
                }
            }
        } else {
        if (wid == 0) {
            unsigned s = 0u, sl = 0u, ends = 0u;
            u32x4 dv = {0u, 0u, 0u, 0u};
            __builtin_amdgcn_s_setprio(3);
            WAIT_V(0);
            for (;;) {
                if ((unsigned)rfl((int)dv.x) != s + 1u) {
                    dv = *(const volatile LAS u32x4*)(ctl + 80 + sl * 4);
                    if ((unsigned)rfl((int)dv.x) != s + 1u) {
                        WAIT_V(0); ends = 0u; lds_put(ctl, s);
                        do { __builtin_amdgcn_s_sleep(1); dv = *(const volatile LAS u32x4*)(ctl + 80 + sl * 4); } while ((unsigned)rfl((int)dv.x) != s + 1u);
                    }
                }
                const char* g = dws + (unsigned)rfl((int)dv.y); unsigned land = (unsigned)rfl((int)dv.z); int left = rfl((int)dv.w);
                if (left < 0) break;
                const unsigned win = land >> 24; land &= 0xffffffu;
                if (++sl == NRS) sl = 0u;
                dv = *(const volatile LAS u32x4*)(ctl + 80 + sl * 4);
#define PULL(w, off) do { asm volatile("s_waitcnt vmcnt(%2)\n global_load_lds_dwordx4 %0, %1 offset:" #off " sc0 sc1" :: "v"(l16), "s"(g), "n"(w) : "memory"); \
                ends &= (1u << (w)) - 1u; lds_put(ctl, s - (unsigned)__builtin_popcount(ends)); ends <<= 1; } while (0)
#define PULL_END(w, off) do { asm volatile("s_waitcnt vmcnt(%2)\n s_mov_b64 exec, %3\n global_load_lds_dwordx4 %0, %1 offset:" #off " sc0 sc1\n s_mov_b64 exec, -1" :: "v"(l16), "s"(g), "n"(w), "s"(mask) : "memory"); \
                ends &= (1u << (w)) - 1u; lds_put(ctl, s - (unsigned)__builtin_popcount(ends)); ends <<= 1; } while (0)
#define PULL_JOB(w) do { \
                for (;;) { asm volatile("s_mov_b32 m0, %0" :: "s"(land) : "memory"); \
                    if (left <= 4096) break; \
                    PULL(w, 0); PULL(w, 1024); PULL(w, 2048); PULL(w, 3072); left -= 4096; g += 4096; land += 4096u; } \
                const u64 mask = ~0ull >> (63 - (((left - 1) & 1023) >> 4)); \
                if (left > 2048) { PULL(w, 0); PULL(w, 1024); left -= 2048; g += 2048; land += 2048u; asm volatile("s_mov_b32 m0, %0" :: "s"(land) : "memory"); } \
                if (left > 1024) { PULL(w, 0); PULL_END(w, 1024); } else PULL_END(w, 0); } while (0)
                if (win > 5u) PULL_JOB(WIN - 1); else if (win == 3u) PULL_JOB(3); else if (win == 5u) PULL_JOB(5); else if (win == 4u) PULL_JOB(4); else if (win == 2u) PULL_JOB(2); else PULL_JOB(1);
                ends |= 1u; ++s;
            }
            WAIT_V(0); lds_put(ctl, s);
            if (lane == 0) st_sys((unsigned*)(dws + ACK_OFF) + rank * 32 + slot, call);
        } else if (wid == 1) {
            constexpr unsigned P_ST = NH + (J_MK - NH * NF) / NQ, NTK = P_ST + 512u / SJ, P_MK = ~0u, NMK = 128 / NH;
            unsigned* const tk = (unsigned*)(myws + TK_OFF) + peer * 32;
            const unsigned tbase = (call - 1u) * (NTK + 2u * NJOIN) - 2u * NH;
            unsigned pA = (unsigned)slot, pB = NH + (unsigned)slot, s = 0u, sl = 0u, rel = 0u, idA, idB, hoA, hoB, tkt = 0u;
            u32x4 hA = {0u, 0u, 0u, 0u}, hB; bool dead = false, first = true;
            __builtin_amdgcn_s_setprio(2);
            if (gs >= 0) { unsigned t = 0u; if (lane < 2) t = __hip_atomic_fetch_add(tk + lane * 256, 2u - lane, __ATOMIC_RELAXED, __HIP_MEMORY_SCOPE_AGENT); pA = (unsigned)rfl((int)t) - tbase; pB = pA + 1u; dead = pA < P_ST; }
#define JOB_OF(p, id, ho) do { const unsigned _t = (p) < NH ? (p) + (lane & 15) * 8u : (p) < P_ST ? ((p) >> 3) * (8u * NQ) + 8u * (NF - NQ) + ((p) & 7u) + (lane & (NQ - 1)) * 8u : (p) == P_MK ? J_MK + slot + (lane & 15) * NH : SJ == 1u ? J_ST + (p) - P_ST : J_ST + (((p) - P_ST) >> 4) * 32u + (((p) - P_ST) & 15u) + (lane & 1) * 16u; \
                id = (p) < P_ST ? *(const volatile LAS unsigned short*)(lds + SEQ_LDS + _t * 2u) : (_t - J_ST) & 127u; \
                ho = (p) < P_ST ? (unsigned)HDR_OFF + ((peer * 24u + (id & 31u)) * 32u + (id >> 5)) * 16u : (unsigned)RDY_OFF + (peer * 128u + (id & 7u) * 16u + (id >> 3)) * 16u + ((p) == P_MK ? 0u : (lane & 2) * 8192u); \
                if ((unsigned)lane >= ((p) < NH ? NF : (p) < P_ST ? NQ : (p) == P_MK ? NMK : 4u)) ho = lane == 16 && (p) >= P_ST ? (unsigned)TK_OFF + 1024u + peer * 128u : 0xfffffff0u; \
                id = (id << 16) | (_t & 0x3fffu); } while (0)
#define NEXT_SLOT() do { ++s; if (++sl == NRS) { sl = 0u; rel += 8u; } } while (0)
#define SLOT_FREE() do { for (;;) { const unsigned _l = *(const volatile LAS unsigned*)ctl, _r = *(const volatile LAS unsigned*)(ctl + 16 + sl); \
                if (s - (unsigned)rfl((int)_l) < LOOK && (unsigned)rfl((int)_r) == rel) break; __builtin_amdgcn_s_sleep(1); } } while (0)
#define POST_NIL(lo) do { lds_put(ctl + 32 + sl * 4, 0x4000u); lds_put4(ctl + 80 + sl * 4, (u32x4){s + 1u, 0u, (lbase + sl * RSLOT + (lo)) | (win << 24), 16u}); NEXT_SLOT(); } while (0)
            JOB_OF(pA, idA, hoA);
            unsigned pC = 0u; int held = gs < 0 ? 0 : 3;
            while (pA < NTK || pA == P_MK) {
                if (held == 0 && pB >= P_ST) { held = 1; pC = pB; pB = P_MK; }
                JOB_OF(pB, idB, hoB);
                asm volatile("buffer_load_dwordx4 %0, %1, %2, 0 offen sc0 sc1" : "=&v"(hB) : "v"(hoB), "s"(rs) : "memory");
#define DRAW() asm volatile("s_mov_b64 exec, 1\n global_atomic_add %0, %1, %2, %3 sc0\n s_mov_b64 exec, -1" : "+v"(tkt) : "v"(0u), "v"(1u), "s"(tk) : "memory")
                if (pA < P_ST + LATED) DRAW();
                if (pA >= P_ST && first) { first = false; lds_put(ctl + 1, s); }
                const bool str = pA >= P_ST && pA != P_MK;
                const unsigned id = idA >> 16, q = (pA - P_ST) * SJ >> 7, nj = pA < NH ? NF : pA < P_ST ? NQ : str ? SJ : NMK;
                const unsigned off = pA < P_ST ? piece_off(id & 31u, id >> 5, rank)
                                   : !str ? (unsigned)MK_OFF + (peer * 128u + id) * 8192u : (unsigned)ST_OFF + (id * 4u + q) * SST;
                unsigned j = 0u, k = 0u; u64 t0 = 0;
                for (;;) {
                    bool ok; unsigned len, win = WIN - 1;
                    if (pA < P_ST) { ok = hA.x == call; len = hA.w & 0x80068006u ? (hA.w & 0x80000000u ? 10240u : hA.w & 0x8000u ? 8192u : hA.w & 0x60000u ? min(5888u + ((hA.w >> 16) & 6u) * 64u, 6144u) : 5504u + (hA.w & 6u) * 64u) : 5504u; }
                    else {
                        const unsigned n = (unsigned)__builtin_amdgcn_readlane((int)hA.x, 16) - (call - 1u) * NJOIN; win = n < 2u ? WIN - 1 : n < 4u ? 5u : n < 7u ? 4u : n < W3N ? 3u : n < W2N ? 2u : 1u;
                        if (!str) { ok = hA.x - call * 4u < 4u && hA.z - call * 4u < 4u; len = 8192u; }
                        else { const unsigned mf = (unsigned)__ballot(hA.x == call) >> 2; ok = (q & 2u ? hA.z : hA.x) - call * 4u - (q & 1u) - 1u < 2u && ((mf >> (lane & 1)) & 1u); len = min(max(((q & 2u ? hA.w : hA.y) >> (16u * (q & 1u))) & 0xfff0u, 16u), 16384u); }
                    }
                    if (dead) { ok = true; len = 16u; }
                    const unsigned nok = (unsigned)__builtin_ctz(~(unsigned)__ballot(ok || (unsigned)lane < j) | (1u << nj)), big = (unsigned)__ballot(len > 8192u && str) & (2u * SJ - 1u);
                    while (j < nok) {
                        unsigned lim = min(nok, j + 4u);
                        if (big >> j) {
                            lim = j + 1u;
                            if ((big >> j) & 1u) { if (sl == NRS - 1) { SLOT_FREE(); POST_NIL(0u); }
                                while ((unsigned)rfl((int)*(const volatile LAS unsigned*)(ctl + 17 + sl)) != rel) __builtin_amdgcn_s_sleep(1); }
                        }
                        const unsigned d = ((unsigned)lane - j) & 3u, sj = sl + d >= NRS ? sl + d - NRS : sl + d;
                        const unsigned ld = *(const volatile LAS unsigned*)ctl, rv = *(const volatile LAS unsigned*)(ctl + 16 + sj);
                        const unsigned j1 = (unsigned)__builtin_ctz(~(unsigned)__ballot(((unsigned)lane < j) | ((s + d - ld < (win > 5u ? LOOK : LOOKS)) & (rv == (sl + d >= NRS ? rel + 8u : rel)))) | (1u << lim));
                        if (j1 == j) { if (nok < nj) break; __builtin_amdgcn_s_sleep(1); continue; }
                        const u32x4 da = {idA | (dead ? 0x8000u : 0u), hA.y, hA.z, hA.w}, db = {s + d + 1u, dead ? 0u : off, (lbase + sj * RSLOT + (pA < P_ST ? 0u : 2048u)) | (win << 24), len};
                        asm volatile("s_mov_b64 exec, %3\n ds_write_b128 %0, %1 offset:128\n ds_write_b128 %0, %2 offset:320\n s_mov_b64 exec, -1" :: "v"(ctl + sj * 4), "v"(da), "v"(db), "s"((u64)((1u << j1) - (1u << j))) : "memory");
                        if (!str) { s += j1 - j; sl += j1 - j; if (sl >= NRS) { sl -= NRS; rel += 8u; } if (pA == P_MK) lds_put(ctl + 3, s); }
                        else for (unsigned i = j; i < j1; ++i) {
                            const char* m = myws + ((unsigned)MK_OFF + ((peer * 128u + (unsigned)__builtin_amdgcn_readlane((int)id, i)) * 4u + q) * 2048u);
                            DMA(m, lbase + sl * RSLOT, 0, "sc1"); DMA(m, lbase + sl * RSLOT, 1024, "sc1"); NEXT_SLOT(); }
                        if ((big >> j) & 1u) POST_NIL(RSLOT - 1024u);
                        j = j1;
                    }
                    if (j == nj) break;
                    if (k == 0u) t0 = __builtin_amdgcn_s_memrealtime(); else __builtin_amdgcn_s_sleep(PULL_SLEEP);
                    if ((++k & SPM) == 0u) {
                        if (give_up(tmo, call, t0)) dead = true;
                        asm volatile("buffer_load_dwordx4 %0, %2, %4, 0 offen sc0 sc1\n buffer_load_dwordx4 %1, %3, %4, 0 offen sc0 sc1\n s_waitcnt vmcnt(0)" : "=&v"(hA), "+v"(hB) : "v"(hoA), "v"(hoB), "s"(rs) : "memory");
                    } else {
                        const unsigned l[2] = {nok, str ? nok + 2u : min(nok + 1u, nj - 1u)}; u32x4 sv[2];
                        asm volatile("s_nop 4\n s_load_dwordx4 %0, %2, %3 glc\n s_load_dwordx4 %1, %2, %4 glc\n s_waitcnt lgkmcnt(0)"
                            : "=&s"(sv[0]), "=&s"(sv[1])
                            : "s"(myws), "s"(__builtin_amdgcn_readlane((int)hoA, l[0])), "s"(__builtin_amdgcn_readlane((int)hoA, l[1])) : "memory");
#pragma unroll
                        for (unsigned i = 0; i < 2u; ++i) hA = (unsigned)lane == l[i] ? sv[i] : hA;
                        if (str && j && k > 1u) { WAIT_V(0); lds_put(ctl + 3, s); }
                    }
                }
                if (str && pA >= P_ST + LATED) DRAW();
                unsigned tn; asm volatile("s_waitcnt vmcnt(0)\n v_readfirstlane_b32 %0, %2" : "=s"(tn), "+v"(hB) : "v"(tkt) : "memory");
                lds_put(ctl + 3, s);
                pA = pB; pB = held == 1 ? pC : tn - tbase; if (held == 1) held = 2;
                hA = hB; idA = idB; hoA = hoB;
            }
            asm volatile(".Lfk_last:");
            if (first) lds_put(ctl + 1, s);
            lds_put(ctl + 2, s);
            SLOT_FREE();
            lds_put4(ctl + 80 + sl * 4, (u32x4){s + 1u, 0u, 0u, ~0u});
        } else {
            if (wid < 6 && gs < 0) { make_olut(lds + OLUT, tid - 128); lds_add(ctl + 7, 1u); }
            unsigned s0, owed = 0u;
#define PAY() do { if (owed) { VWAIT0; WAIT_V(0); FLAG(owed); owed = 0u;  } } while (0)
            for (;;) {
                const unsigned c = lds_add_rtn(ctl + 4, 1u), s = c >> 1, u = c & 1u, sl = s % NRS;
                const LAS unsigned char* land = lds + RING + sl * RSLOT;
                u32x4 ds;
                for (;;) { const u32x2 m = *(const volatile LAS u32x2*)ctl; ds = *(const volatile LAS u32x4*)(ctl + 32 + sl * 4);
                    s0 = (unsigned)rfl((int)m.y); if (s >= s0 || (unsigned)rfl((int)m.x) > s) break;
                    PAY(); __builtin_amdgcn_s_sleep(DEC_SLEEP); }
                if (s >= s0) break;
                asm volatile("" ::: "memory");
                const unsigned id = (unsigned)rfl((int)ds.x), h0[1] = {(unsigned)rfl((int)(u ? ds.z : ds.y))}, h1 = ((unsigned)rfl((int)ds.w) >> (16u * u)) & 0xffffu, bk = (id >> 16) & 31u, e = id >> 21;
                u32x4 p[1][2], o[1][4]; u32x2 cw[1]; unsigned c2[1][1], n3[1];
                const unsigned l3 = (unsigned)lane + ((h1 >> 4) & 127u);
                n3[0] = *(const volatile LAS unsigned*)(land + l3 * 4u + (l3 < 48u ? 5120u + u * 192u : 5312u + u * 320u));
                c2[0][0] = *(const volatile LAS unsigned*)(land + lane * 4 + (lane < 48 ? 5120u + u * 192u : 5312u + u * 320u));
                cw[0] = *(const volatile LAS u32x2*)(land + 4096 + u * 512 + lane * 8);
#pragma unroll
                for (int k = 0; k < 2; ++k) p[0][k] = *(const volatile LAS u32x4*)(land + u * 2048u + k * 1024 + lane * 16);
                if (h1 & 0x8000u) {
#pragma unroll
                    for (int k = 0; k < 2; ++k) { o[0][k] = p[0][k]; o[0][2 + k] = *(const volatile LAS u32x4*)(land + 6144 + u * 2048u + k * 1024 + lane * 16); }
                }
                lds_add(ctl + 16 + sl, 4u);
                if (id & 0x8000u) continue;
                if (!(h1 & 0x8000u)) unpack_piece<1>(p, cw, c2, n3, h0, h1, lds, wb, lane, o);
                PAY();
                const unsigned st1 = bk < 16u ? (unsigned)XS_T : 1024u, st2 = bk < 16u ? 1024u : 2048u, base = unit_base(peer, e, bk, u);
#pragma unroll
                for (int b = 0; b < 4; ++b) BSTORE(o[0][b], rs, (unsigned)lane * 16u, base + (b & 1) * st1 + (b >> 1) * st2, AUX_DEV);
                owed = (unsigned)INF_OFF + (inf_idx(peer, bk, e) + u) * 4u;
            }
            PAY();
        }
        {
            __builtin_amdgcn_s_setprio(0);
            const unsigned s0 = (unsigned)rfl((int)*(const volatile LAS unsigned*)(ctl + 1));
            while (rfl((int)*(const volatile LAS unsigned*)(ctl + 7)) != 4) __builtin_amdgcn_s_sleep(1);
            for (;;) {
#pragma unroll
                for (int u = 0; u < RG; ++u) { *(LAS u32x4*)(lds + wb + u * 2176 + lane * 16) = (u32x4){0u, 0u, 0u, 0u}; *(LAS unsigned*)(lds + wb + u * 2176 + 1024 + lane * 4) = 0u; }
                const unsigned c = lds_add_rtn(ctl + 5, 1u), s = s0 + c / (8u / RG), r = (c * RG) & 7u, sl = s % NRS; unsigned end, idv, rh, mw[2];
                const LAS unsigned char* sbuf = lds + RING + 2048 + sl * RSLOT;
                for (;;) { const u32x4 m = *(const volatile LAS u32x4*)ctl; idv = *(const volatile LAS unsigned*)(ctl + 32 + sl * 4); rh = *(const volatile LAS unsigned*)(sbuf + (r & 6u) * 2u);
                    mw[0] = *(const volatile LAS unsigned*)(sbuf - 2048 + r * 256u + lane * 4); if (RG == 2) mw[1] = *(const volatile LAS unsigned*)(sbuf - 1792 + r * 256u + lane * 4);
                    end = (unsigned)rfl((int)m.z); if (s >= end || (unsigned)rfl((int)min(m.x, m.w)) > s) break; __builtin_amdgcn_s_sleep(DEC_SLEEP); }
                if (s >= end) break;
                asm volatile("" ::: "memory");
                const unsigned id = (unsigned)rfl((int)idv), t = id & 0x3fffu, tile = id >> 16; const bool bad = (id & 0x8000u) != 0u;
                if (id & 0x4000u) { lds_add(ctl + 16 + sl, RG); continue; }
                if (t < J_ST) {
#pragma unroll
                    for (int u = 0; u < RG; ++u) { const u32x4 m = *(const volatile LAS u32x4*)(sbuf + (r + u) * 1024u + lane * 16);
                        BSTORE(m, rs, (unsigned)lane * 16u, (unsigned)MK_OFF + (peer * 128u + tile) * 8192u + (r + u) * 1024u, AUX_DEV); }
                    VWAIT0; WAIT_V(0);
                    if (((lds_add_rtn(ctl + 16 + sl, RG) + RG) & 7u) == 0u && !bad) FLAG((unsigned)MKF_OFF + (peer * 128u + (tile & 7u) * 16u + (tile >> 3)) * 16u);
                    continue;
                }
                const unsigned q = (t - J_ST) >> 7, mid2 = bad ? 0u : (unsigned)rfl((int)rh) >> ((r & 1u) * 16u);
                const unsigned voff = ((tile >> 3) * 256u + (q >> 1) * 128u + (r >> 2) * 64u + (q & 1u) * 32u + (lane & 15)) * GN + peer * NS + (tile & 7u) * 256u + (r & 3u) * 64u + (lane >> 4) * 16;
                u32x4 qb[2][2];
                const unsigned mid[2] = {mid2 & 0x7fffu, (mid2 >> 16) & 0x7fffu};
                if ((mid2 & 0x8000u) && t - (J_ST + TQ0) < 384u + TQUICK - TQ0) {
                    u32x2 sc[2][4]; unsigned cn[2], pe[2], at[2] = {7u, 7u}, last[2], ne[2][4];
#pragma unroll
                    for (int u = 0; u < RG; ++u) cn[u] = (unsigned)__builtin_popcount(mw[u]) << 16;
#pragma unroll 1
                    for (unsigned pass = 0; pass < 9u; ++pass) {
                        unsigned w[2], ru[2][2], inc[2], p[2], pn[2], pb[2];
#pragma unroll
                        for (int u = 0; u < RG; ++u) w[u] = lds_get_bits(sbuf, mid[u] * 4u + pass * 1024u + lane * 16u) & 0xffffu;
#pragma unroll
                        for (int u = 0; u < RG; ++u) {
#pragma unroll
                            for (int d = 0; d < 2; ++d) ru[u][d] = *(const LAS unsigned*)(lds + OLUT + 4096 + ((w[u] >> (8 * d)) & 255u) * 4u);
                            cn[u] |= (unsigned)__builtin_popcount(w[u]); }
                        if (RG == 2) wave_scan2(cn, inc); else inc[0] = wave_scan(cn[0]);
#pragma unroll
                        for (int u = 0; u < RG; ++u) { const unsigned tot = (unsigned)__builtin_amdgcn_readlane((int)inc[u], 63), st = ((mid[u] - (tot >> 16)) & 0x7fffu) * 4u, bf = (inc[u] - cn[u]) >> 16;
                            pn[u] = st + bf * 4u; pb[u] = ((st - (tot >> 16)) & 0x1ffffu) + bf;
                            p[u] = at[u] + ((inc[u] - cn[u]) & 0xffffu); at[u] += tot & 0xffffu; cn[u] = 0u; }
                        unsigned l0[2][4], l1[2][4], ps[2][4], b5[2];
                        if (pass == 0u) {
#pragma unroll
                            for (int u = 0; u < RG; ++u) { b5[u] = lds_get_bits(sbuf, pb[u]);
#pragma unroll
                                for (int d = 0; d < 4; ++d) { const LAS unsigned* lp = (const LAS unsigned*)(sbuf + ((pn[u] >> 3) & ~3u));
                                    l0[u][d] = lp[0]; l1[u][d] = lp[1]; ps[u][d] = pn[u]; pn[u] += (unsigned)__builtin_popcount((mw[u] >> (8 * d)) & 255u) * 4u;
                                    sc[u][d] = *(const LAS u32x2*)(lds + OLUT + 2048 + ((mw[u] >> (8 * d)) & 255u) * 8u); } }
                        }
#pragma unroll
                        for (int d = 0; d < 2; ++d)
#pragma unroll
                            for (int u = 0; u < RG; ++u) { const unsigned pc = min(p[u], 2072u); const u64 h = (u64)ru[u][d] << ((pc & 7u) * 4u); LAS unsigned* hp = (LAS unsigned*)(lds + wb + u * 2176 + (pc >> 3) * 4u);
                                __hip_atomic_fetch_add(hp, (unsigned)h, __ATOMIC_RELAXED, __HIP_MEMORY_SCOPE_WORKGROUP); __hip_atomic_fetch_add(hp + 1, (unsigned)(h >> 32), __ATOMIC_RELAXED, __HIP_MEMORY_SCOPE_WORKGROUP);
                                p[u] += (unsigned)__builtin_popcount((w[u] >> (8 * d)) & 255u); }
                        if (pass == 0u) { unsigned ce[2] = {0u, 0u};
#pragma unroll
                            for (int u = 0; u < RG; ++u)
#pragma unroll
                                for (int d = 0; d < 4; ++d) { const unsigned l = __builtin_amdgcn_alignbit(l1[u][d], l0[u][d], ps[u][d]);
                                    unsigned h = (b5[u] >> ((ps[u][d] - ps[u][0]) >> 2)) & 255u; h = (h | (h << 12)) & 0x000f000fu; h = (h | (h << 6)) & 0x03030303u; h = (h | (h << 3)) & 0x11111111u;
                                    const unsigned ev = bfi(0x0f0f0f0fu, l, h << 4), od = bfi(0x0f0f0f0fu, l >> 4, h), v0 = perm(od, ev, sc[u][d].x), v1 = perm(od, ev, sc[u][d].y);
                                    const unsigned t0 = v0 & (v0 >> 1) & 0x08080808u, t1 = v1 & (v1 >> 1) & 0x08080808u;
                                    qb[u][d >> 1][(d & 1) * 2] = v0; qb[u][d >> 1][(d & 1) * 2 + 1] = v1;
                                    sc[u][d] = *(const LAS u32x2*)(lds + dot4(t0, 0x08040201u, dot4(t1, 0x80402010u, (unsigned)OLUT + 2048u)));
                                    ne[u][d] = ce[u]; ce[u] += (unsigned)(__builtin_popcount(t0) + __builtin_popcount(t1)); }
                            if (RG == 2) wave_scan2(ce, pe); else pe[0] = wave_scan(ce[0]);
#pragma unroll
                            for (int u = 0; u < RG; ++u) { last[u] = 7u + (unsigned)__builtin_amdgcn_readlane((int)pe[u], 63); pe[u] += 8u - ce[u]; }
                        }
                        if (at[0] > last[0] && at[RG - 1] > last[RG - 1]) break;
                    }
                    lds_add(ctl + 16 + sl, RG);
                    unsigned h0[2][4], h1[2][4];
#pragma unroll
                    for (int u = 0; u < RG; ++u)
#pragma unroll
                        for (int d = 0; d < 4; ++d) { ne[u][d] += pe[u]; const LAS unsigned* hp = (const LAS unsigned*)(lds + wb + u * 2176 + (ne[u][d] >> 3) * 4u); h0[u][d] = hp[0]; h1[u][d] = hp[1]; }
                    SCHED;
#pragma unroll
                    for (int u = 0; u < RG; ++u)
#pragma unroll
                        for (int d = 0; d < 4; ++d) { const unsigned h = __builtin_amdgcn_alignbit(h1[u][d], h0[u][d], ne[u][d] * 4u), ev = (h << 3) & 0x78787878u, od = (h >> 1) & 0x78787878u;
                            qb[u][d >> 1][(d & 1) * 2] += perm(od, ev, sc[u][d].x); qb[u][d >> 1][(d & 1) * 2 + 1] += perm(od, ev, sc[u][d].y); }
                } else if (mid2 & 0x8000u) {
                    u32x2 sc[2][4]; unsigned cn[2], pn[2], at[2], last[2], hofs[2];
#pragma unroll
                    for (int u = 0; u < 2; ++u) cn[u] = (unsigned)__builtin_popcount(mw[u]) << 16;
#pragma unroll 1
                    for (unsigned pass = 0; pass < 5u; ++pass) {
                        unsigned w[2], ru[2][4], inc[2], p[2];
#pragma unroll
                        for (int u = 0; u < 2; ++u) w[u] = lds_get_bits(sbuf, mid[u] * 4u + pass * 2048u + lane * 32u);
#pragma unroll
                        for (int u = 0; u < 2; ++u) {
#pragma unroll
                            for (int d = 0; d < 4; ++d) ru[u][d] = *(const LAS unsigned*)(lds + OLUT + 4096 + ((w[u] >> (8 * d)) & 255u) * 4u);
                            cn[u] |= (unsigned)__builtin_popcount(w[u]); }
                        wave_scan2(cn, inc);
#pragma unroll
                        for (int u = 0; u < 2; ++u) { const unsigned tot = (unsigned)__builtin_amdgcn_readlane((int)inc[u], 63);
                            if (pass == 0u) { const unsigned st = (mid[u] - (tot >> 16)) & 0x7fffu; pn[u] = (st + ((inc[u] - cn[u]) >> 16)) * 4u; at[u] = (st & 7u) + 7u; last[u] = at[u] + (tot >> 16); hofs[u] = wb + u * 2176u + 4u - (st >> 3) * 4u - (RING + 2048 + sl * RSLOT); }
                            p[u] = at[u] + ((inc[u] - cn[u]) & 0xffffu); at[u] += tot & 0xffffu; cn[u] = 0u; }
#pragma unroll
                        for (int d = 0; d < 4; ++d)
#pragma unroll
                            for (int u = 0; u < 2; ++u) { const unsigned pc = min(p[u], 2072u); const u64 h = (u64)ru[u][d] << ((pc & 7u) * 4u); LAS unsigned* hp = (LAS unsigned*)(lds + wb + u * 2176 + (pc >> 3) * 4u);
                                __hip_atomic_fetch_add(hp, (unsigned)h, __ATOMIC_RELAXED, __HIP_MEMORY_SCOPE_WORKGROUP); __hip_atomic_fetch_add(hp + 1, (unsigned)(h >> 32), __ATOMIC_RELAXED, __HIP_MEMORY_SCOPE_WORKGROUP);
                                p[u] += (unsigned)__builtin_popcount((w[u] >> (8 * d)) & 255u); }
                        if (at[0] > last[0] && at[1] > last[1]) break;
                    }
                    asm volatile("" ::: "memory");
                    unsigned l0[2][4], l1[2][4], h0[2][4], h1[2][4], ps[2][4];
#pragma unroll
                    for (int u = 0; u < 2; ++u)
#pragma unroll
                        for (int d = 0; d < 4; ++d) { const unsigned la = (unsigned)(size_t)sbuf + ((pn[u] >> 3) & ~3u); const LAS unsigned* lp = (const LAS unsigned*)(size_t)la; const LAS unsigned* hp = (const LAS unsigned*)(size_t)(la + hofs[u]);
                            l0[u][d] = lp[0]; l1[u][d] = lp[1]; h0[u][d] = hp[0]; h1[u][d] = hp[1]; ps[u][d] = pn[u]; pn[u] += (unsigned)__builtin_popcount((mw[u] >> (8 * d)) & 255u) * 4u;
                            sc[u][d] = *(const LAS u32x2*)(lds + OLUT + 2048 + ((mw[u] >> (8 * d)) & 255u) * 8u); }
                    lds_add(ctl + 16 + sl, RG);
                    SCHED;
#pragma unroll
                    for (int u = 0; u < 2; ++u)
#pragma unroll
                        for (int d = 0; d < 4; ++d) { const unsigned l = __builtin_amdgcn_alignbit(l1[u][d], l0[u][d], ps[u][d]), h = __builtin_amdgcn_alignbit(h1[u][d], h0[u][d], ps[u][d]);
                            const unsigned ev = bfi(0x0f0f0f0fu, l, h << 4), od = bfi(0x0f0f0f0fu, l >> 4, h);
                            qb[u][d >> 1][(d & 1) * 2] = perm(od, ev, sc[u][d].x); qb[u][d >> 1][(d & 1) * 2 + 1] = perm(od, ev, sc[u][d].y); }
                } else {
#pragma unroll
                    for (int k = 0; k < 2 * RG; ++k) { qb[k >> 1][k & 1] = *(const volatile LAS u32x4*)(sbuf + (k & 1) * 8192 + (r + (k >> 1)) * 1024u + lane * 16);
                        if (bad) qb[k >> 1][k & 1] = (u32x4){0x80808080u, 0x80808080u, 0x80808080u, 0x80808080u}; }
                    lds_add(ctl + 16 + sl, RG);
                }
#pragma unroll
                for (int k = 0; k < 2 * RG; ++k) BSTORE(qb[k % RG][k / RG], ro, voff + (k % RG) * 64u, (unsigned)((k / RG) * 16 * GN), OUT_AUX);
            }
        }
        }
    }
}
}

extern "C" void kernel_launch(int rank, int nranks, ncclComm_t comm, hipStream_t stream,
                              void* const* d_in, const int* in_sizes, int n_in,
                              void* d_out, int out_size, const int* in_dtypes, int out_dtype,
                              void* const* d_out_all, void* const* d_ws_all, size_t ws_bytes) {
    if (nranks != NR || ws_bytes < WS_NEED || (unsigned)rank >= (unsigned)NR) { fprintf(stderr, "unexpected configuration\n"); return; }
    HIP_CHECK(hipFuncSetAttribute((const void*)fused_kernel, hipFuncAttributeMaxDynamicSharedMemorySize, LDS_BYTES));
    Params P;
    P.x = (const float*)d_in[0]; P.w = (const float*)d_in[1]; P.rank = rank; P.out = (signed char*)d_out; P.me = (char*)d_ws_all[rank] + WS_BASE; P.pad = 0;
    for (int p = 0; p < NR; ++p) P.ws[p] = (char*)d_ws_all[p] + WS_BASE;
    hipLaunchKernelGGL(fused_kernel, dim3(256), dim3(512), LDS_BYTES, stream, P);
    HIP_CHECK(hipGetLastError());
}
```
